```python
import numpy as np

kernel_name = "gpu_gemm_a2a_m4096_k4096_n4096_bf16_gelu"


ABSMAX_RTOL = 0.008

COMPUTE_DTYPE = "bfloat16"
EPILOGUE = "gelu"
IN_QUANT = None
OUT_QUANT = None
IN_DTYPES = {"x": "bfloat16", "w_mat": "bfloat16"}
WORLD_SIZE = 8

def _to_bf16(x):
    u = np.ascontiguousarray(x, dtype=np.float32).view(np.uint32)
    return ((u + 0x7FFF + ((u >> 16) & 1)) & 0xFFFF0000).view(np.float32)

def setup_inputs():

    x = _to_bf16(np.random.randn(4096, 4096).astype(np.float32))
    w_mat = _to_bf16((np.random.randn(4096, 4096) / np.sqrt(4096)).astype(np.float32))
    return {"x": x, "w_mat": w_mat}

def _epilogue(y):
    c = 0.7978845608028654
    return (0.5 * y * (1.0 + np.tanh(c * (y + 0.044715 * y ** 3)))).astype(np.float32)

def reference(x, w_mat):
    y = (x @ w_mat).astype(np.float32)
    return _epilogue(y)
```

```cpp
#include <hip/hip_runtime.h>
#include <rccl/rccl.h>
#include <cstdio>

#define HIP_CHECK(expr) do { hipError_t _e = (expr); if (_e != hipSuccess) { \
    fprintf(stderr, "HIP error %s at %s:%d\n", hipGetErrorString(_e), __FILE__, __LINE__); \
    return; } } while (0)

namespace {
#define LAS __attribute__((address_space(3)))
#define DI __device__ __forceinline__
typedef __bf16 bf16x8 __attribute__((ext_vector_type(8)));
typedef __bf16 bf16x2 __attribute__((ext_vector_type(2)));
typedef short s16x4 __attribute__((ext_vector_type(4)));
typedef short s16x8 __attribute__((ext_vector_type(8)));
typedef float f32x4 __attribute__((ext_vector_type(4)));
typedef float f32x2 __attribute__((ext_vector_type(2)));
typedef unsigned u32x4 __attribute__((ext_vector_type(4)));

constexpr int NR = 8;
constexpr int ML = 512, K = 4096, N = 4096, NS = 512;
constexpr int BM = 128, BN = 64, BK = 64, NT = K / BK;
constexpr int XT = BM * BK * 2, WT = BK * BN * 2, STG = XT + WT, NSTAGE = 6;
constexpr int LDS_BYTES = STG * NSTAGE;
constexpr int NWG = 256;
constexpr unsigned WS_FLAG = 0;
constexpr unsigned WS_CTR = 64 * 1024;
constexpr unsigned WS_TMO = 128 * 1024;
constexpr unsigned long long DEADLINE = 50000000ull;

struct Args { const void* x; const void* w; void* out[NR]; void* ws[NR]; int rank; int pad; };

DI void glds(unsigned m0v, unsigned voff, __amdgpu_buffer_rsrc_t srd, unsigned soff) {
  asm volatile("s_mov_b32 m0, %0\n\ts_nop 0\n\tbuffer_load_dwordx4 %1, %2, %3 offen lds" :: "s"(m0v), "v"(voff), "s"(srd), "s"(soff) : "memory");
}
DI unsigned cvtpk(float lo, float hi) { f32x2 v = {lo, hi}; bf16x2 b = __builtin_convertvector(v, bf16x2); return __builtin_bit_cast(unsigned, b); }
DI float gelu(float y) {
  const float k1 = -2.0f * 0.7978845608028654f * 1.4426950408889634f, k3 = k1 * 0.044715f;
  const float t = y * __builtin_fmaf(k3, y * y, k1);
  return y * __builtin_amdgcn_rcpf(1.0f + __builtin_amdgcn_exp2f(t));
}
template <int Nn> DI void waitv() { asm volatile("s_waitcnt vmcnt(%0)" :: "i"(Nn) : "memory"); }

__global__ __launch_bounds__(256, 1) void gemm_a2a(Args a) {
  extern __shared__ __attribute__((aligned(16))) char smem[];
  LAS char* lds = (LAS char*)smem;
  const int tid = threadIdx.x, lane = tid & 63;
  const int wid = __builtin_amdgcn_readfirstlane(tid >> 6);
  const int b = blockIdx.x, xcd = b & 7, j = b >> 3, rg = j & 3;
  const int rank = a.rank;
  const unsigned peer = (unsigned)((j >> 2) + rank) & 7u;
  const int col0 = peer * NS + xcd * BN, row0 = rg * BM;
  const int slot = xcd * 4 + rg;
  unsigned* wsl = (unsigned*)a.ws[rank];
  unsigned call = 0;
  if (tid == 0) call = __hip_atomic_fetch_add(wsl + (WS_CTR + b * 128) / 4, 1u, __ATOMIC_RELAXED, __HIP_MEMORY_SCOPE_AGENT) + 1u;

  __amdgpu_buffer_rsrc_t sx = __builtin_amdgcn_make_buffer_rsrc((void*)((const char*)a.x + (size_t)row0 * K * 2), (short)0, BM * K * 2, 0x00020000);
  __amdgpu_buffer_rsrc_t sw = __builtin_amdgcn_make_buffer_rsrc((void*)((const char*)a.w + (size_t)col0 * 2), (short)0, K * N * 2 - col0 * 2, 0x00020000);
  unsigned xo[4], wo[2];
#pragma unroll
  for (int i = 0; i < 4; ++i) { const int r = 32 * wid + 8 * i + (lane >> 3), c = (lane & 7) ^ ((r >> 1) & 7); xo[i] = r * (K * 2) + c * 16; }
#pragma unroll
  for (int i = 0; i < 2; ++i) { const int r = 16 * wid + 8 * i + (lane >> 3), c = (lane & 7) ^ ((r & 2) | ((r >> 1) & 4)); wo[i] = r * (N * 2) + c * 16; }
  const unsigned ldsbase = (unsigned)(size_t)lds;
  auto stage = [&](int t, int s) {
    const unsigned lb = ldsbase + s * STG;
#pragma unroll
    for (int i = 0; i < 4; ++i) glds(lb + (32 * wid + 8 * i) * 128, xo[i], sx, t * (BK * 2));
#pragma unroll
    for (int i = 0; i < 2; ++i) glds(lb + XT + (16 * wid + 8 * i) * 128, wo[i], sw, t * (BK * N * 2));
  };
  const int wm = wid >> 1, wn = wid & 1, fr = lane & 15, g = lane >> 4, q = (lane & 15) >> 2, p = lane & 3;
  int xr[2];
#pragma unroll
  for (int s = 0; s < 2; ++s) { const int r = 64 * wm + fr; xr[s] = r * 128 + (((4 * s + g) ^ ((r >> 1) & 7)) * 16); }
  int wr[2][2];
#pragma unroll
  for (int nb = 0; nb < 2; ++nb)
#pragma unroll
    for (int t = 0; t < 2; ++t) { const int r = 8 * g + 4 * t + q, c = 4 * wn + 2 * nb + (p >> 1); wr[nb][t] = XT + r * 128 + ((c ^ ((r & 2) | ((r >> 1) & 4))) * 16) + 8 * (p & 1); }
  f32x4 acc[4][2];
#pragma unroll
  for (int m = 0; m < 4; ++m) for (int n = 0; n < 2; ++n) acc[m][n] = (f32x4){0.f, 0.f, 0.f, 0.f};

  auto compute = [&](int s) {
    LAS char* sb = lds + s * STG;
    bf16x8 xf[4][2], wf[2][2];
#pragma unroll
    for (int ks = 0; ks < 2; ++ks) {
#pragma unroll
      for (int nb = 0; nb < 2; ++nb) {
        s16x4 lo = __builtin_amdgcn_ds_read_tr16_b64_v4i16((LAS s16x4*)(sb + wr[nb][0] + ks * 4096));
        s16x4 hi = __builtin_amdgcn_ds_read_tr16_b64_v4i16((LAS s16x4*)(sb + wr[nb][1] + ks * 4096));
        wf[nb][ks] = __builtin_bit_cast(bf16x8, __builtin_shufflevector(lo, hi, 0, 1, 2, 3, 4, 5, 6, 7));
      }
#pragma unroll
      for (int mb = 0; mb < 4; ++mb) xf[mb][ks] = *(LAS bf16x8*)(sb + xr[ks] + mb * 2048);
    }
#pragma unroll
    for (int ks = 0; ks < 2; ++ks)
#pragma unroll
      for (int mb = 0; mb < 4; ++mb)
#pragma unroll
        for (int nb = 0; nb < 2; ++nb) acc[mb][nb] = __builtin_amdgcn_mfma_f32_16x16x32_bf16(wf[nb][ks], xf[mb][ks], acc[mb][nb], 0, 0, 0);
  };

#pragma unroll
  for (int s = 0; s < NSTAGE - 1; ++s) stage(s, s);
  int sc = 0, sp = NSTAGE - 1;
  for (int t = 0; t < NT - (NSTAGE - 1); ++t) {
    waitv<6 * (NSTAGE - 2)>();
    __builtin_amdgcn_s_barrier();
    stage(t + NSTAGE - 1, sp);
    compute(sc);
    sc = sc == NSTAGE - 1 ? 0 : sc + 1; sp = sp == NSTAGE - 1 ? 0 : sp + 1;
  }
#define TAIL(i) { waitv<6 * (i)>(); __builtin_amdgcn_s_barrier(); compute(sc); sc = sc == NSTAGE - 1 ? 0 : sc + 1; }
  TAIL(4) TAIL(3) TAIL(2) TAIL(1) TAIL(0)

  char* ob = (char*)a.out[peer] + ((size_t)(rank * ML + row0 + 64 * wm + fr) * NS + xcd * BN + 32 * wn + 16 * (g & 1) + 8 * (g >> 1)) * 2;
#pragma unroll
  for (int mb = 0; mb < 4; ++mb) {
    unsigned d0 = cvtpk(gelu(acc[mb][0][0]), gelu(acc[mb][0][1])), d1 = cvtpk(gelu(acc[mb][0][2]), gelu(acc[mb][0][3]));
    unsigned e0 = cvtpk(gelu(acc[mb][1][0]), gelu(acc[mb][1][1])), e1 = cvtpk(gelu(acc[mb][1][2]), gelu(acc[mb][1][3]));
    auto r0 = __builtin_amdgcn_permlane16_swap(d0, e0, false, false);
    auto r1 = __builtin_amdgcn_permlane16_swap(d1, e1, false, false);
    u32x4 v = {r0[0], r1[0], r0[1], r1[1]};
    asm volatile("global_store_dwordx4 %0, %1, off sc0 sc1\n\ts_nop 1" :: "v"(ob + (size_t)mb * 16 * NS * 2), "v"(v) : "memory");
  }
  asm volatile("s_waitcnt vmcnt(0)" ::: "memory");
  __syncthreads();
  if (wid == 0) {
    call = __builtin_amdgcn_readfirstlane(call);
    if (peer != (unsigned)rank) {
      if (lane == 0) __hip_atomic_store((unsigned*)a.ws[peer] + (WS_FLAG + (rank * 32 + slot) * 128) / 4, call, __ATOMIC_RELAXED, __HIP_MEMORY_SCOPE_SYSTEM);
    } else if (lane < NR && lane != rank) {
      unsigned* fl = wsl + (WS_FLAG + (lane * 32 + slot) * 128) / 4; unsigned* tmo = wsl + WS_TMO / 4;
      const unsigned long long t0 = __builtin_amdgcn_s_memrealtime();
      bool bad = false;
      while (__hip_atomic_load(fl, __ATOMIC_RELAXED, __HIP_MEMORY_SCOPE_SYSTEM) < call) {
        if (__hip_atomic_load(tmo, __ATOMIC_RELAXED, __HIP_MEMORY_SCOPE_AGENT) == call || __builtin_amdgcn_s_memrealtime() - t0 > DEADLINE) {
          __hip_atomic_store(tmo, call, __ATOMIC_RELAXED, __HIP_MEMORY_SCOPE_AGENT); bad = true; break; }
        __builtin_amdgcn_s_sleep(1);
      }
      if (bad) *(unsigned short*)ob = 0x7fc0;
    }
  }
}
}

extern "C" void kernel_launch(int rank, int nranks, ncclComm_t comm, hipStream_t stream,
                              void* const* d_in, const int* in_sizes, int n_in, void* d_out, int out_size,
                              const int* in_dtypes, int out_dtype, void* const* d_out_all, void* const* d_ws_all, size_t ws_bytes) {
  static bool init = false;
  if (!init) { HIP_CHECK(hipFuncSetAttribute((const void*)gemm_a2a, hipFuncAttributeMaxDynamicSharedMemorySize, LDS_BYTES)); init = true; }
  if (nranks != NR) { fprintf(stderr, "expected 8 ranks\n"); return; }
  Args a; a.x = d_in[0]; a.w = d_in[1]; a.rank = rank; a.pad = 0;
  for (int p = 0; p < NR; ++p) { a.out[p] = d_out_all[p]; a.ws[p] = d_ws_all[p]; }
  hipLaunchKernelGGL(gemm_a2a, dim3(NWG), dim3(256), LDS_BYTES, stream, a);
}
```

```cpp
#include <hip/hip_runtime.h>
#include <rccl/rccl.h>
#include <cstdio>
#include <cstdlib>

#define HIP_CHECK(expr) do { hipError_t _e = (expr); if (_e != hipSuccess) { \
    fprintf(stderr, "HIP error %s at %s:%d\n", hipGetErrorString(_e), __FILE__, __LINE__); \
    return; } } while (0)

namespace {
typedef unsigned u32x2 __attribute__((ext_vector_type(2)));
typedef unsigned u32x4 __attribute__((ext_vector_type(4)));
typedef unsigned u32x8 __attribute__((ext_vector_type(8)));
typedef unsigned u32x16 __attribute__((ext_vector_type(16)));

constexpr int NR = 8;
constexpr unsigned ML = 512, KK = 4096, NN = 4096, NS = 512;
constexpr int LDS_BYTES = 160 * 1024;
constexpr int NWG = 256;
constexpr unsigned WS_FLAG = 0;
constexpr unsigned WS_CTR = 256 * 1024;
constexpr unsigned WS_TMO = 320 * 1024;
constexpr unsigned WS_XF = 640 * 1024;
constexpr unsigned WS_PART = 1024 * 1024;
constexpr unsigned PART_BYTES = 64u * 393216u;
constexpr unsigned long long DEADLINE = 50000000ull;

struct Peer { char* out; char* flag; };
struct Args { const char* x; const char* w; char* wsl; unsigned rank; unsigned pad; Peer p[NR];
};
constexpr unsigned DLY_ODD = 45, DLY_SPREAD = 180, DLY_LOCAL = 240;

__device__ __forceinline__ u32x4 mk_srd(const char* p, unsigned bytes) {
  const unsigned long long a = (unsigned long long)p;
  u32x4 r = {(unsigned)a, (unsigned)(a >> 32) & 0xffffu, bytes, 0x00020000u};
  return r;
}

__global__ __launch_bounds__(256, 1) void gemm_a2a(Args a) {
  extern __shared__ __attribute__((aligned(16))) char smem[];
  const unsigned long long t0 = __builtin_amdgcn_s_memrealtime();
  const char* const px = a.x; const char* const pw = a.w; char* const wsl = a.wsl; unsigned rk = a.rank;
  const unsigned tid = threadIdx.x, lane = tid & 63;
  const unsigned b = blockIdx.x & 255u, xc = b & 7, kq = xc & 3, ch = xc >> 2, j = b >> 3, rg = j & 3, cg = j >> 2;
  const unsigned e = cg & 1, late = e, mbit = kq >> 1, nbit = kq & 1;
  const unsigned dst = ((cg >> 1) * 2 + ch) & 7u;
  const Peer pr = a.p[dst];
  const unsigned fr = lane & 15, fg = lane >> 4, qq = (lane & 15) >> 2, pp = lane & 3;
  u32x8 v0;
#pragma unroll
  for (unsigned i = 0; i < 4; ++i) {
    const unsigned rx = 4 * i + fg, rw = 8 * i + (lane >> 3);
    v0[i] = rx * 8192 + ((fr ^ rx) * 16) - 1024 * i;
    v0[4 + i] = rw * 8192 + (((lane & 7) ^ ((rw & 2) | ((rw >> 1) & 4))) * 16) - 1024 * i;
  }
  const unsigned w = __builtin_amdgcn_readfirstlane(tid >> 6);
  const unsigned rot = (2 * rg + cg) & 7;
  const unsigned ldsb = (unsigned)(size_t)(__attribute__((address_space(3))) char*)smem;
  unsigned xo = (rg * 128 + 32 * w) * (KK * 2) + kq * 2048;
  unsigned wo = (kq * 1024 + 32 * w) * (NN * 2) + dst * 1024 + e * 128;
  const unsigned ldsw = ldsb + w * 4096;
  const u32x2 smx = {mbit * 16 * 8192, (mbit ^ 1) * 16 * 8192};
  const u32x2 s2 = {ldsw, w};
  unsigned dly_r = late * DLY_ODD + (rg * 4 + ((cg >> 1) ^ 3)) * DLY_SPREAD / 32, dly_l = DLY_LOCAL + (4 * e + rg) * 4;
  asm volatile("" : "+s"(xo), "+s"(wo), "+s"(rk), "+s"(dly_r), "+s"(dly_l) : "{s64}"(rot << 20), "{s[72:73]}"(smx), "{s[80:81]}"(s2), "{v[8:15]}"(v0));
  const unsigned rank = rk & 7u;
  const u32x4 sx = mk_srd(px + xo, ML * KK * 2 - xo);
  const u32x4 sw = mk_srd(pw + wo, KK * NN * 2 - wo);
  {
    const unsigned dly = dst == rank ? dly_l : dly_r;
    asm volatile("" :: "{s[36:39]}"(sx), "{s[40:43]}"(sw));
    if (dly > 20) {
      const unsigned long long ts = t0 + dly;
      while (__builtin_amdgcn_s_memrealtime() < ts) __builtin_amdgcn_s_sleep(1);
    }
  }
  asm volatile(
    "s_lshr_b32 s88, s64, 12\n\t"
    "s_add_u32 m0, s80, 0\n\t"
    "s_add_u32 s88, s88, s72\n\t"
    "buffer_load_dwordx4 v8, s[36:39], s88 offen offset:0 lds\n\t"
    "buffer_load_dwordx4 v9, s[36:39], s88 offen offset:1024 lds\n\t"
    "buffer_load_dwordx4 v10, s[36:39], s88 offen offset:2048 lds\n\t"
    "buffer_load_dwordx4 v11, s[36:39], s88 offen offset:3072 lds\n\t"
    "s_lshr_b32 s88, s64, 12\n\t"
    "s_add_u32 m0, s80, 16384\n\t"
    "s_add_u32 s88, s88, s73\n\t"
    "buffer_load_dwordx4 v8, s[36:39], s88 offen offset:0 lds\n\t"
    "buffer_load_dwordx4 v9, s[36:39], s88 offen offset:1024 lds\n\t"
    "buffer_load_dwordx4 v10, s[36:39], s88 offen offset:2048 lds\n\t"
    "buffer_load_dwordx4 v11, s[36:39], s88 offen offset:3072 lds\n\t"
    "s_add_u32 m0, s80, 32768\n\t"
    "s_add_u32 s88, s64, 256\n\t"
    "buffer_load_dwordx4 v12, s[40:43], s88 offen offset:0 lds\n\t"
    "buffer_load_dwordx4 v13, s[40:43], s88 offen offset:1024 lds\n\t"
    "buffer_load_dwordx4 v14, s[40:43], s88 offen offset:2048 lds\n\t"
    "buffer_load_dwordx4 v15, s[40:43], s88 offen offset:3072 lds\n\t"
    :: "{s[36:39]}"(sx), "{s[40:43]}"(sw), "{s64}"(rot << 20), "{s[72:73]}"(smx), "{s[80:81]}"(s2), "{v[8:15]}"(v0) : "s88", "scc", "memory");

  const unsigned po = (((4 * late + rg) * 4 + (cg >> 1)) * 2 + ch) * 98304u + w * 2048;
  const u32x4 sp = mk_srd(wsl + WS_PART + po, PART_BYTES - po);
  const u32x4 sf = mk_srd(wsl + WS_XF + (ch * 32 + j) * 2048, 2048);
  const u32x4 sc = mk_srd(wsl + WS_CTR + b * 128, 8);
  const unsigned long long dl = t0 + DEADLINE;
  const unsigned slot = (kq * 4 + rg) * 2 + e;
  char* const fout = pr.flag + slot * 128;
  u32x16 s1;
#pragma unroll
  for (unsigned g = 0; g < 8; ++g) s1[g] = ((g + rot) & 7) << 20;
  s1[8] = smx[0]; s1[9] = smx[1];
#pragma unroll
  for (unsigned l = 1; l < 4; ++l) { s1[9 + l] = ((kq ^ l) * 3 + l - 1) * 8192; s1[12 + l] = (kq * 3 + l - 1) * 8192; }
  u32x8 v1;
#pragma unroll
  for (unsigned i = 0; i < 4; ++i) {
    v1[i] = ldsw + fr * 256 + (((4 * i + fg) ^ fr) * 16);
    v1[4 + i] = ldsb + (8 * fg + qq) * 128 + (((2 * (i ^ (2 * nbit)) + (pp >> 1)) ^ ((qq & 2) | ((fg & 1) << 2))) * 16) + 8 * (pp & 1);
  }
  const float k1 = -2.0f * 0.7978845608028654f * 1.4426950408889634f, k3 = k1 * 0.044715f;
  const unsigned orow = (rg * 128 + 32 * w + 16 * mbit + fr) * (NS * 2);
  const unsigned oo = orow + e * 128 + 64 * nbit + (fg & 1) * 32 + (fg >> 1) * 16;
  const unsigned one = fg == qq ? 0x3f80u << (16 * (pp & 1)) : 0u;
  u32x8 v2 = {lane * 16, oo, __float_as_uint(k1), __float_as_uint(k3), tid < 4 ? lane * 512 + kq * 4 : 0x7fffff00u, kq * 512 + (lane & 3) * 4, pp < 2 ? one : 0u, pp < 2 ? 0u : one};
  unsigned bad, call;
  asm volatile(
    "v_add_u32 v32, 65536, v16\n\t"
    "v_add_u32 v40, 65536, v20\n\t"
    "v_add_u32 v33, 65536, v17\n\t"
    "v_add_u32 v41, 65536, v21\n\t"
    "v_add_u32 v34, 65536, v18\n\t"
    "v_add_u32 v42, 65536, v22\n\t"
    "v_add_u32 v35, 65536, v19\n\t"
    "v_add_u32 v43, 65536, v23\n\t"
    "v_add_u32 v36, 131072, v16\n\t"
    "v_add_u32 v44, 131072, v20\n\t"
    "v_add_u32 v37, 131072, v17\n\t"
    "v_add_u32 v45, 131072, v21\n\t"
    "v_add_u32 v38, 131072, v18\n\t"
    "v_add_u32 v46, 131072, v22\n\t"
    "v_add_u32 v39, 131072, v19\n\t"
    "v_add_u32 v47, 131072, v23\n\t"
    "s_add_u32 s82, s44, s74\n\t"
    "s_addc_u32 s83, s45, 0\n\t"
    "s_add_u32 s84, s44, s75\n\t"
    "s_addc_u32 s85, s45, 0\n\t"
    "s_add_u32 s86, s44, s76\n\t"
    "s_addc_u32 s87, s45, 0\n\t"
    "v_mov_b32 v58, 0\n\t"
    "v_mov_b32 v59, 0\n\t"
    "v_mov_b32 v62, 0\n\t"
    "v_mov_b32 v63, 0\n\t"
    "v_mov_b32 v60, v30\n\t"
    "v_mov_b32 v61, v31\n\t"
    "v_add_u32 v52, 18874368, v24\n\t"
    "s_mov_b32 s90, 0\n\t"
    "v_mov_b32 v51, 0\n\t"
    "buffer_load_dwordx2 v[54:55], v51, s[48:51], 0 offen sc0 sc1\n\t"
    "s_add_u32 m0, s80, 49152\n\t"
    "s_add_u32 s88, s65, 256\n\t"
    "buffer_load_dwordx4 v12, s[40:43], s88 offen offset:0 lds\n\t"
    "buffer_load_dwordx4 v13, s[40:43], s88 offen offset:1024 lds\n\t"
    "buffer_load_dwordx4 v14, s[40:43], s88 offen offset:2048 lds\n\t"
    "buffer_load_dwordx4 v15, s[40:43], s88 offen offset:3072 lds\n\t"
    "s_waitcnt vmcnt(5)\n\t"
    "s_waitcnt lgkmcnt(0)\n\t"
    "s_barrier\n\t"
    "ds_read_b128 a[0:3], v16 offset:0\n\t"
    "ds_read_b128 a[4:7], v16 offset:16384\n\t"
    "ds_read_b64_tr_b16 v[188:189], v20 offset:32768\n\t"
    "ds_read_b64_tr_b16 v[190:191], v20 offset:33280\n\t"
    "ds_read_b64_tr_b16 v[192:193], v21 offset:32768\n\t"
    "ds_read_b64_tr_b16 v[194:195], v21 offset:33280\n\t"
    "ds_read_b64_tr_b16 v[196:197], v22 offset:32768\n\t"
    "ds_read_b64_tr_b16 v[198:199], v22 offset:33280\n\t"
    "ds_read_b64_tr_b16 v[200:201], v23 offset:32768\n\t"
    "ds_read_b64_tr_b16 v[202:203], v23 offset:33280\n\t"
    "ds_read_b64_tr_b16 v[204:205], v20 offset:36864\n\t"
    "ds_read_b64_tr_b16 v[206:207], v20 offset:37376\n\t"
    "ds_read_b128 a[8:11], v17 offset:0\n\t"
    "ds_read_b128 a[12:15], v17 offset:16384\n\t"
    "ds_read_b128 a[16:19], v18 offset:0\n\t"
    "ds_read_b128 a[20:23], v18 offset:16384\n\t"
    "ds_read_b128 a[24:27], v19 offset:0\n\t"
    "ds_read_b128 a[28:31], v19 offset:16384\n\t"
    "ds_read_b64_tr_b16 v[208:209], v21 offset:36864\n\t"
    "s_waitcnt lgkmcnt(13)\n\t"
    "s_lshr_b32 s88, s65, 12\n\t"
    "v_mfma_f32_16x16x32_bf16 v[64:67], v[188:191], a[0:3], 0\n\t"
    "ds_read_b64_tr_b16 v[210:211], v21 offset:37376\n\t"
    "s_add_u32 m0, s80, 65536\n\t"
    "s_add_u32 s88, s88, s72\n\t"
    "v_mfma_f32_16x16x32_bf16 v[68:71], v[188:191], a[4:7], 0\n\t"
    "ds_read_b64_tr_b16 v[212:213], v22 offset:36864\n\t"
    "buffer_load_dwordx4 v8, s[36:39], s88 offen offset:0 lds\n\t"
    "v_mfma_f32_16x16x32_bf16 v[72:75], v[192:195], a[0:3], 0\n\t"
    "ds_read_b64_tr_b16 v[214:215], v22 offset:37376\n\t"
    "buffer_load_dwordx4 v9, s[36:39], s88 offen offset:1024 lds\n\t"
    "v_mfma_f32_16x16x32_bf16 v[76:79], v[192:195], a[4:7], 0\n\t"
    "ds_read_b64_tr_b16 v[216:217], v23 offset:36864\n\t"
    "s_waitcnt lgkmcnt(13)\n\t"
    "buffer_load_dwordx4 v10, s[36:39], s88 offen offset:2048 lds\n\t"
    "v_mfma_f32_16x16x32_bf16 v[80:83], v[196:199], a[0:3], 0\n\t"
    "ds_read_b64_tr_b16 v[218:219], v23 offset:37376\n\t"
    "buffer_load_dwordx4 v11, s[36:39], s88 offen offset:3072 lds\n\t"
    "s_lshr_b32 s88, s65, 12\n\t"
    "v_mfma_f32_16x16x32_bf16 v[84:87], v[196:199], a[4:7], 0\n\t"
    "ds_read_b64_tr_b16 v[188:189], v20 offset:40960\n\t"
    "s_add_u32 m0, s80, 81920\n\t"
    "s_add_u32 s88, s88, s73\n\t"
    "v_mfma_f32_16x16x32_bf16 v[88:91], v[200:203], a[0:3], 0\n\t"
    "ds_read_b64_tr_b16 v[190:191], v20 offset:41472\n\t"
    "buffer_load_dwordx4 v8, s[36:39], s88 offen offset:0 lds\n\t"
    "v_mfma_f32_16x16x32_bf16 v[92:95], v[200:203], a[4:7], 0\n\t"
    "ds_read_b64_tr_b16 v[192:193], v21 offset:40960\n\t"
    "s_waitcnt lgkmcnt(7)\n\t"
    "buffer_load_dwordx4 v9, s[36:39], s88 offen offset:1024 lds\n\t"
    "v_mfma_f32_16x16x32_bf16 v[64:67], v[204:207], a[8:11], v[64:67]\n\t"
    "ds_read_b64_tr_b16 v[194:195], v21 offset:41472\n\t"
    "buffer_load_dwordx4 v10, s[36:39], s88 offen offset:2048 lds\n\t"
    "v_mfma_f32_16x16x32_bf16 v[68:71], v[204:207], a[12:15], v[68:71]\n\t"
    "ds_read_b64_tr_b16 v[196:197], v22 offset:40960\n\t"
    "buffer_load_dwordx4 v11, s[36:39], s88 offen offset:3072 lds\n\t"
    "s_lshr_b32 s88, s66, 12\n\t"
    "v_mfma_f32_16x16x32_bf16 v[72:75], v[208:211], a[8:11], v[72:75]\n\t"
    "ds_read_b64_tr_b16 v[198:199], v22 offset:41472\n\t"
    "s_add_u32 m0, s80, 98304\n\t"
    "s_add_u32 s88, s88, s72\n\t"
    "v_mfma_f32_16x16x32_bf16 v[76:79], v[208:211], a[12:15], v[76:79]\n\t"
    "ds_read_b64_tr_b16 v[200:201], v23 offset:40960\n\t"
    "s_waitcnt lgkmcnt(7)\n\t"
    "buffer_load_dwordx4 v8, s[36:39], s88 offen offset:0 lds\n\t"
    "v_mfma_f32_16x16x32_bf16 v[80:83], v[212:215], a[8:11], v[80:83]\n\t"
    "ds_read_b64_tr_b16 v[202:203], v23 offset:41472\n\t"
    "buffer_load_dwordx4 v9, s[36:39], s88 offen offset:1024 lds\n\t"
    "v_mfma_f32_16x16x32_bf16 v[84:87], v[212:215], a[12:15], v[84:87]\n\t"
    "ds_read_b64_tr_b16 v[204:205], v20 offset:45056\n\t"
    "buffer_load_dwordx4 v10, s[36:39], s88 offen offset:2048 lds\n\t"
    "v_mfma_f32_16x16x32_bf16 v[88:91], v[216:219], a[8:11], v[88:91]\n\t"
    "ds_read_b64_tr_b16 v[206:207], v20 offset:45568\n\t"
    "buffer_load_dwordx4 v11, s[36:39], s88 offen offset:3072 lds\n\t"
    "s_lshr_b32 s88, s66, 12\n\t"
    "v_mfma_f32_16x16x32_bf16 v[92:95], v[216:219], a[12:15], v[92:95]\n\t"
    "ds_read_b64_tr_b16 v[208:209], v21 offset:45056\n\t"
    "s_waitcnt lgkmcnt(7)\n\t"
    "s_add_u32 m0, s80, 114688\n\t"
    "v_mfma_f32_16x16x32_bf16 v[64:67], v[188:191], a[16:19], v[64:67]\n\t"
    "ds_read_b64_tr_b16 v[210:211], v21 offset:45568\n\t"
    "s_add_u32 s88, s88, s73\n\t"
    "buffer_load_dwordx4 v8, s[36:39], s88 offen offset:0 lds\n\t"
    "v_mfma_f32_16x16x32_bf16 v[68:71], v[188:191], a[20:23], v[68:71]\n\t"
    "ds_read_b64_tr_b16 v[212:213], v22 offset:45056\n\t"
    "buffer_load_dwordx4 v9, s[36:39], s88 offen offset:1024 lds\n\t"
    "v_mfma_f32_16x16x32_bf16 v[72:75], v[192:195], a[16:19], v[72:75]\n\t"
    "ds_read_b64_tr_b16 v[214:215], v22 offset:45568\n\t"
    "buffer_load_dwordx4 v10, s[36:39], s88 offen offset:2048 lds\n\t"
    "v_mfma_f32_16x16x32_bf16 v[76:79], v[192:195], a[20:23], v[76:79]\n\t"
    "s_waitcnt vmcnt(7)\n\t"
    "s_barrier\n\t"
    "ds_read_b64_tr_b16 v[216:217], v23 offset:45056\n\t"
    "s_waitcnt lgkmcnt(7)\n\t"
    "buffer_load_dwordx4 v11, s[36:39], s88 offen offset:3072 lds\n\t"
    "v_mfma_f32_16x16x32_bf16 v[80:83], v[196:199], a[16:19], v[80:83]\n\t"
    "ds_read_b64_tr_b16 v[218:219], v23 offset:45568\n\t"
    "ds_read_b128 a[32:35], v32 offset:0\n\t"
    "ds_read_b128 a[36:39], v32 offset:16384\n\t"
    "v_mfma_f32_16x16x32_bf16 v[84:87], v[196:199], a[20:23], v[84:87]\n\t"
    "ds_read_b64_tr_b16 v[188:189], v20 offset:49152\n\t"
    "ds_read_b128 a[40:43], v33 offset:0\n\t"
    "ds_read_b128 a[44:47], v33 offset:16384\n\t"
    "v_mfma_f32_16x16x32_bf16 v[88:91], v[200:203], a[16:19], v[88:91]\n\t"
    "ds_read_b64_tr_b16 v[190:191], v20 offset:49664\n\t"
    "ds_read_b128 a[48:51], v34 offset:0\n\t"
    "ds_read_b128 a[52:55], v34 offset:16384\n\t"
    "v_mfma_f32_16x16x32_bf16 v[92:95], v[200:203], a[20:23], v[92:95]\n\t"
    "ds_read_b64_tr_b16 v[192:193], v21 offset:49152\n\t"
    "s_waitcnt lgkmcnt(13)\n\t"
    "ds_read_b128 a[56:59], v35 offset:0\n\t"
    "v_mfma_f32_16x16x32_bf16 v[64:67], v[204:207], a[24:27], v[64:67]\n\t"
    "ds_read_b64_tr_b16 v[194:195], v21 offset:49664\n\t"
    "ds_read_b128 a[60:63], v35 offset:16384\n\t"
    "s_add_u32 m0, s80, 131072\n\t"
    "v_mfma_f32_16x16x32_bf16 v[68:71], v[204:207], a[28:31], v[68:71]\n\t"
    "ds_read_b64_tr_b16 v[196:197], v22 offset:49152\n\t"
    "s_add_u32 s88, s66, 256\n\t"
    "buffer_load_dwordx4 v12, s[40:43], s88 offen offset:0 lds\n\t"
    "v_mfma_f32_16x16x32_bf16 v[72:75], v[208:211], a[24:27], v[72:75]\n\t"
    "ds_read_b64_tr_b16 v[198:199], v22 offset:49664\n\t"
    "buffer_load_dwordx4 v13, s[40:43], s88 offen offset:1024 lds\n\t"
    "v_mfma_f32_16x16x32_bf16 v[76:79], v[208:211], a[28:31], v[76:79]\n\t"
    "ds_read_b64_tr_b16 v[200:201], v23 offset:49152\n\t"
    "s_waitcnt lgkmcnt(15)\n\t"
    "v_mfma_f32_16x16x32_bf16 v[80:83], v[212:215], a[24:27], v[80:83]\n\t"
    "ds_read_b64_tr_b16 v[202:203], v23 offset:49664\n\t"
    "buffer_load_dwordx4 v14, s[40:43], s88 offen offset:2048 lds\n\t"
    "v_mfma_f32_16x16x32_bf16 v[84:87], v[212:215], a[28:31], v[84:87]\n\t"
    "ds_read_b64_tr_b16 v[204:205], v20 offset:53248\n\t"
    "v_mfma_f32_16x16x32_bf16 v[88:91], v[216:219], a[24:27], v[88:91]\n\t"
    "ds_read_b64_tr_b16 v[206:207], v20 offset:53760\n\t"
    "buffer_load_dwordx4 v15, s[40:43], s88 offen offset:3072 lds\n\t"
    "s_lshr_b32 s88, s67, 12\n\t"
    "v_mfma_f32_16x16x32_bf16 v[92:95], v[216:219], a[28:31], v[92:95]\n\t"
    "ds_read_b64_tr_b16 v[208:209], v21 offset:53248\n\t"
    "s_waitcnt lgkmcnt(8)\n\t"
    "s_add_u32 m0, s80, 147456\n\t"
    "v_mfma_f32_16x16x32_bf16 v[64:67], v[188:191], a[32:35], v[64:67]\n\t"
    "ds_read_b64_tr_b16 v[210:211], v21 offset:53760\n\t"
    "s_add_u32 s88, s88, s72\n\t"
    "buffer_load_dwordx4 v8, s[36:39], s88 offen offset:0 lds\n\t"
    "v_mfma_f32_16x16x32_bf16 v[68:71], v[188:191], a[36:39], v[68:71]\n\t"
    "ds_read_b64_tr_b16 v[212:213], v22 offset:53248\n\t"
    "buffer_load_dwordx4 v9, s[36:39], s88 offen offset:1024 lds\n\t"
    "v_mfma_f32_16x16x32_bf16 v[72:75], v[192:195], a[32:35], v[72:75]\n\t"
    "ds_read_b64_tr_b16 v[214:215], v22 offset:53760\n\t"
    "v_mfma_f32_16x16x32_bf16 v[76:79], v[192:195], a[36:39], v[76:79]\n\t"
    "ds_read_b64_tr_b16 v[216:217], v23 offset:53248\n\t"
    "s_waitcnt lgkmcnt(7)\n\t"
    "buffer_load_dwordx4 v10, s[36:39], s88 offen offset:2048 lds\n\t"
    "v_mfma_f32_16x16x32_bf16 v[80:83], v[196:199], a[32:35], v[80:83]\n\t"
    "ds_read_b64_tr_b16 v[218:219], v23 offset:53760\n\t"
    "v_mfma_f32_16x16x32_bf16 v[84:87], v[196:199], a[36:39], v[84:87]\n\t"
    "ds_read_b64_tr_b16 v[188:189], v20 offset:57344\n\t"
    "buffer_load_dwordx4 v11, s[36:39], s88 offen offset:3072 lds\n\t"
    "s_lshr_b32 s88, s67, 12\n\t"
    "v_mfma_f32_16x16x32_bf16 v[88:91], v[200:203], a[32:35], v[88:91]\n\t"
    "ds_read_b64_tr_b16 v[190:191], v20 offset:57856\n\t"
    "s_add_u32 m0, s80, 0\n\t"
    "s_add_u32 s88, s88, s73\n\t"
    "v_mfma_f32_16x16x32_bf16 v[92:95], v[200:203], a[36:39], v[92:95]\n\t"
    "ds_read_b64_tr_b16 v[192:193], v21 offset:57344\n\t"
    "s_waitcnt lgkmcnt(7)\n\t"
    "buffer_load_dwordx4 v8, s[36:39], s88 offen offset:0 lds\n\t"
    "v_mfma_f32_16x16x32_bf16 v[64:67], v[204:207], a[40:43], v[64:67]\n\t"
    "ds_read_b64_tr_b16 v[194:195], v21 offset:57856\n\t"
    "v_mfma_f32_16x16x32_bf16 v[68:71], v[204:207], a[44:47], v[68:71]\n\t"
    "ds_read_b64_tr_b16 v[196:197], v22 offset:57344\n\t"
    "buffer_load_dwordx4 v9, s[36:39], s88 offen offset:1024 lds\n\t"
    "v_mfma_f32_16x16x32_bf16 v[72:75], v[208:211], a[40:43], v[72:75]\n\t"
    "ds_read_b64_tr_b16 v[198:199], v22 offset:57856\n\t"
    "buffer_load_dwordx4 v10, s[36:39], s88 offen offset:2048 lds\n\t"
    "v_mfma_f32_16x16x32_bf16 v[76:79], v[208:211], a[44:47], v[76:79]\n\t"
    "ds_read_b64_tr_b16 v[200:201], v23 offset:57344\n\t"
    "s_waitcnt lgkmcnt(7)\n\t"
    "v_mfma_f32_16x16x32_bf16 v[80:83], v[212:215], a[40:43], v[80:83]\n\t"
    "ds_read_b64_tr_b16 v[202:203], v23 offset:57856\n\t"
    "buffer_load_dwordx4 v11, s[36:39], s88 offen offset:3072 lds\n\t"
    "s_add_u32 m0, s80, 16384\n\t"
    "v_mfma_f32_16x16x32_bf16 v[84:87], v[212:215], a[44:47], v[84:87]\n\t"
    "ds_read_b64_tr_b16 v[204:205], v20 offset:61440\n\t"
    "s_add_u32 s88, s67, 256\n\t"
    "v_mfma_f32_16x16x32_bf16 v[88:91], v[216:219], a[40:43], v[88:91]\n\t"
    "ds_read_b64_tr_b16 v[206:207], v20 offset:61952\n\t"
    "buffer_load_dwordx4 v12, s[40:43], s88 offen offset:0 lds\n\t"
    "v_mfma_f32_16x16x32_bf16 v[92:95], v[216:219], a[44:47], v[92:95]\n\t"
    "ds_read_b64_tr_b16 v[208:209], v21 offset:61440\n\t"
    "s_waitcnt lgkmcnt(7)\n\t"
    "v_mfma_f32_16x16x32_bf16 v[64:67], v[188:191], a[48:51], v[64:67]\n\t"
    "ds_read_b64_tr_b16 v[210:211], v21 offset:61952\n\t"
    "buffer_load_dwordx4 v13, s[40:43], s88 offen offset:1024 lds\n\t"
    "v_mfma_f32_16x16x32_bf16 v[68:71], v[188:191], a[52:55], v[68:71]\n\t"
    "ds_read_b64_tr_b16 v[212:213], v22 offset:61440\n\t"
    "v_mfma_f32_16x16x32_bf16 v[72:75], v[192:195], a[48:51], v[72:75]\n\t"
    "ds_read_b64_tr_b16 v[214:215], v22 offset:61952\n\t"
    "buffer_load_dwordx4 v14, s[40:43], s88 offen offset:2048 lds\n\t"
    "v_mfma_f32_16x16x32_bf16 v[76:79], v[192:195], a[52:55], v[76:79]\n\t"
    "s_waitcnt vmcnt(11)\n\t"
    "s_barrier\n\t"
    "ds_read_b64_tr_b16 v[216:217], v23 offset:61440\n\t"
    "s_waitcnt lgkmcnt(7)\n\t"
    "buffer_load_dwordx4 v15, s[40:43], s88 offen offset:3072 lds\n\t"
    "v_mfma_f32_16x16x32_bf16 v[80:83], v[196:199], a[48:51], v[80:83]\n\t"
    "ds_read_b64_tr_b16 v[218:219], v23 offset:61952\n\t"
    "ds_read_b128 a[64:67], v32 offset:32768\n\t"
    "ds_read_b128 a[68:71], v32 offset:49152\n\t"
    "v_mfma_f32_16x16x32_bf16 v[84:87], v[196:199], a[52:55], v[84:87]\n\t"
    "ds_read_b64_tr_b16 v[188:189], v44 offset:0\n\t"
    "ds_read_b128 a[72:75], v33 offset:32768\n\t"
    "ds_read_b128 a[76:79], v33 offset:49152\n\t"
    "v_mfma_f32_16x16x32_bf16 v[88:91], v[200:203], a[48:51], v[88:91]\n\t"
    "ds_read_b64_tr_b16 v[190:191], v44 offset:512\n\t"
    "ds_read_b128 a[80:83], v34 offset:32768\n\t"
    "ds_read_b128 a[84:87], v34 offset:49152\n\t"
    "v_mfma_f32_16x16x32_bf16 v[92:95], v[200:203], a[52:55], v[92:95]\n\t"
    "ds_read_b64_tr_b16 v[192:193], v45 offset:0\n\t"
    "s_waitcnt lgkmcnt(13)\n\t"
    "ds_read_b128 a[88:91], v35 offset:32768\n\t"
    "v_mfma_f32_16x16x32_bf16 v[64:67], v[204:207], a[56:59], v[64:67]\n\t"
    "ds_read_b64_tr_b16 v[194:195], v45 offset:512\n\t"
    "ds_read_b128 a[92:95], v35 offset:49152\n\t"
    "s_lshr_b32 s88, s68, 12\n\t"
    "v_mfma_f32_16x16x32_bf16 v[68:71], v[204:207], a[60:63], v[68:71]\n\t"
    "ds_read_b64_tr_b16 v[196:197], v46 offset:0\n\t"
    "s_add_u32 m0, s80, 32768\n\t"
    "s_add_u32 s88, s88, s72\n\t"
    "v_mfma_f32_16x16x32_bf16 v[72:75], v[208:211], a[56:59], v[72:75]\n\t"
    "ds_read_b64_tr_b16 v[198:199], v46 offset:512\n\t"
    "buffer_load_dwordx4 v8, s[36:39], s88 offen offset:0 lds\n\t"
    "v_mfma_f32_16x16x32_bf16 v[76:79], v[208:211], a[60:63], v[76:79]\n\t"
    "ds_read_b64_tr_b16 v[200:201], v47 offset:0\n\t"
    "s_waitcnt lgkmcnt(15)\n\t"
    "buffer_load_dwordx4 v9, s[36:39], s88 offen offset:1024 lds\n\t"
    "v_mfma_f32_16x16x32_bf16 v[80:83], v[212:215], a[56:59], v[80:83]\n\t"
    "ds_read_b64_tr_b16 v[202:203], v47 offset:512\n\t"
    "v_mfma_f32_16x16x32_bf16 v[84:87], v[212:215], a[60:63], v[84:87]\n\t"
    "ds_read_b64_tr_b16 v[204:205], v44 offset:4096\n\t"
    "buffer_load_dwordx4 v10, s[36:39], s88 offen offset:2048 lds\n\t"
    "v_mfma_f32_16x16x32_bf16 v[88:91], v[216:219], a[56:59], v[88:91]\n\t"
    "ds_read_b64_tr_b16 v[206:207], v44 offset:4608\n\t"
    "v_mfma_f32_16x16x32_bf16 v[92:95], v[216:219], a[60:63], v[92:95]\n\t"
    "ds_read_b64_tr_b16 v[208:209], v45 offset:4096\n\t"
    "s_waitcnt lgkmcnt(8)\n\t"
    "v_mfma_f32_16x16x32_bf16 v[64:67], v[188:191], a[64:67], v[64:67]\n\t"
    "ds_read_b64_tr_b16 v[210:211], v45 offset:4608\n\t"
    "buffer_load_dwordx4 v11, s[36:39], s88 offen offset:3072 lds\n\t"
    "s_lshr_b32 s88, s68, 12\n\t"
    "v_mfma_f32_16x16x32_bf16 v[68:71], v[188:191], a[68:71], v[68:71]\n\t"
    "ds_read_b64_tr_b16 v[212:213], v46 offset:4096\n\t"
    "s_add_u32 m0, s80, 65536\n\t"
    "s_add_u32 s88, s88, s73\n\t"
    "v_mfma_f32_16x16x32_bf16 v[72:75], v[192:195], a[64:67], v[72:75]\n\t"
    "ds_read_b64_tr_b16 v[214:215], v46 offset:4608\n\t"
    "buffer_load_dwordx4 v8, s[36:39], s88 offen offset:0 lds\n\t"
    "v_mfma_f32_16x16x32_bf16 v[76:79], v[192:195], a[68:71], v[76:79]\n\t"
    "ds_read_b64_tr_b16 v[216:217], v47 offset:4096\n\t"
    "s_waitcnt lgkmcnt(7)\n\t"
    "v_mfma_f32_16x16x32_bf16 v[80:83], v[196:199], a[64:67], v[80:83]\n\t"
    "ds_read_b64_tr_b16 v[218:219], v47 offset:4608\n\t"
    "v_mfma_f32_16x16x32_bf16 v[84:87], v[196:199], a[68:71], v[84:87]\n\t"
    "ds_read_b64_tr_b16 v[188:189], v44 offset:8192\n\t"
    "buffer_load_dwordx4 v9, s[36:39], s88 offen offset:1024 lds\n\t"
    "v_mfma_f32_16x16x32_bf16 v[88:91], v[200:203], a[64:67], v[88:91]\n\t"
    "ds_read_b64_tr_b16 v[190:191], v44 offset:8704\n\t"
    "v_mfma_f32_16x16x32_bf16 v[92:95], v[200:203], a[68:71], v[92:95]\n\t"
    "ds_read_b64_tr_b16 v[192:193], v45 offset:8192\n\t"
    "s_waitcnt lgkmcnt(7)\n\t"
    "buffer_load_dwordx4 v10, s[36:39], s88 offen offset:2048 lds\n\t"
    "v_mfma_f32_16x16x32_bf16 v[64:67], v[204:207], a[72:75], v[64:67]\n\t"
    "ds_read_b64_tr_b16 v[194:195], v45 offset:8704\n\t"
    "v_mfma_f32_16x16x32_bf16 v[68:71], v[204:207], a[76:79], v[68:71]\n\t"
    "ds_read_b64_tr_b16 v[196:197], v46 offset:8192\n\t"
    "buffer_load_dwordx4 v11, s[36:39], s88 offen offset:3072 lds\n\t"
    "s_add_u32 m0, s80, 81920\n\t"
    "v_mfma_f32_16x16x32_bf16 v[72:75], v[208:211], a[72:75], v[72:75]\n\t"
    "ds_read_b64_tr_b16 v[198:199], v46 offset:8704\n\t"
    "s_add_u32 s88, s68, 256\n\t"
    "v_mfma_f32_16x16x32_bf16 v[76:79], v[208:211], a[76:79], v[76:79]\n\t"
    "ds_read_b64_tr_b16 v[200:201], v47 offset:8192\n\t"
    "s_waitcnt lgkmcnt(7)\n\t"
    "v_mfma_f32_16x16x32_bf16 v[80:83], v[212:215], a[72:75], v[80:83]\n\t"
    "ds_read_b64_tr_b16 v[202:203], v47 offset:8704\n\t"
    "buffer_load_dwordx4 v12, s[40:43], s88 offen offset:0 lds\n\t"
    "v_mfma_f32_16x16x32_bf16 v[84:87], v[212:215], a[76:79], v[84:87]\n\t"
    "ds_read_b64_tr_b16 v[204:205], v44 offset:12288\n\t"
    "v_mfma_f32_16x16x32_bf16 v[88:91], v[216:219], a[72:75], v[88:91]\n\t"
    "ds_read_b64_tr_b16 v[206:207], v44 offset:12800\n\t"
    "buffer_load_dwordx4 v13, s[40:43], s88 offen offset:1024 lds\n\t"
    "v_mfma_f32_16x16x32_bf16 v[92:95], v[216:219], a[76:79], v[92:95]\n\t"
    "ds_read_b64_tr_b16 v[208:209], v45 offset:12288\n\t"
    "s_waitcnt lgkmcnt(7)\n\t"
    "v_mfma_f32_16x16x32_bf16 v[64:67], v[188:191], a[80:83], v[64:67]\n\t"
    "ds_read_b64_tr_b16 v[210:211], v45 offset:12800\n\t"
    "buffer_load_dwordx4 v14, s[40:43], s88 offen offset:2048 lds\n\t"
    "v_mfma_f32_16x16x32_bf16 v[68:71], v[188:191], a[84:87], v[68:71]\n\t"
    "ds_read_b64_tr_b16 v[212:213], v46 offset:12288\n\t"
    "v_mfma_f32_16x16x32_bf16 v[72:75], v[192:195], a[80:83], v[72:75]\n\t"
    "ds_read_b64_tr_b16 v[214:215], v46 offset:12800\n\t"
    "v_mfma_f32_16x16x32_bf16 v[76:79], v[192:195], a[84:87], v[76:79]\n\t"
    "s_waitcnt vmcnt(11)\n\t"
    "s_barrier\n\t"
    "ds_read_b64_tr_b16 v[216:217], v47 offset:12288\n\t"
    "s_waitcnt lgkmcnt(7)\n\t"
    "buffer_load_dwordx4 v15, s[40:43], s88 offen offset:3072 lds\n\t"
    "v_mfma_f32_16x16x32_bf16 v[80:83], v[196:199], a[80:83], v[80:83]\n\t"
    "ds_read_b64_tr_b16 v[218:219], v47 offset:12800\n\t"
    "ds_read_b128 a[96:99], v36 offset:16384\n\t"
    "ds_read_b128 a[100:103], v16 offset:0\n\t"
    "v_mfma_f32_16x16x32_bf16 v[84:87], v[196:199], a[84:87], v[84:87]\n\t"
    "ds_read_b64_tr_b16 v[188:189], v20 offset:16384\n\t"
    "ds_read_b128 a[104:107], v37 offset:16384\n\t"
    "ds_read_b128 a[108:111], v17 offset:0\n\t"
    "v_mfma_f32_16x16x32_bf16 v[88:91], v[200:203], a[80:83], v[88:91]\n\t"
    "ds_read_b64_tr_b16 v[190:191], v20 offset:16896\n\t"
    "ds_read_b128 a[112:115], v38 offset:16384\n\t"
    "ds_read_b128 a[116:119], v18 offset:0\n\t"
    "v_mfma_f32_16x16x32_bf16 v[92:95], v[200:203], a[84:87], v[92:95]\n\t"
    "ds_read_b64_tr_b16 v[192:193], v21 offset:16384\n\t"
    "s_waitcnt lgkmcnt(13)\n\t"
    "ds_read_b128 a[120:123], v39 offset:16384\n\t"
    "v_mfma_f32_16x16x32_bf16 v[64:67], v[204:207], a[88:91], v[64:67]\n\t"
    "ds_read_b64_tr_b16 v[194:195], v21 offset:16896\n\t"
    "ds_read_b128 a[124:127], v19 offset:0\n\t"
    "s_lshr_b32 s88, s69, 12\n\t"
    "v_mfma_f32_16x16x32_bf16 v[68:71], v[204:207], a[92:95], v[68:71]\n\t"
    "ds_read_b64_tr_b16 v[196:197], v22 offset:16384\n\t"
    "s_add_u32 m0, s80, 49152\n\t"
    "s_add_u32 s88, s88, s72\n\t"
    "v_mfma_f32_16x16x32_bf16 v[72:75], v[208:211], a[88:91], v[72:75]\n\t"
    "ds_read_b64_tr_b16 v[198:199], v22 offset:16896\n\t"
    "buffer_load_dwordx4 v8, s[36:39], s88 offen offset:0 lds\n\t"
    "v_mfma_f32_16x16x32_bf16 v[76:79], v[208:211], a[92:95], v[76:79]\n\t"
    "ds_read_b64_tr_b16 v[200:201], v23 offset:16384\n\t"
    "s_waitcnt lgkmcnt(15)\n\t"
    "buffer_load_dwordx4 v9, s[36:39], s88 offen offset:1024 lds\n\t"
    "v_mfma_f32_16x16x32_bf16 v[80:83], v[212:215], a[88:91], v[80:83]\n\t"
    "ds_read_b64_tr_b16 v[202:203], v23 offset:16896\n\t"
    "v_mfma_f32_16x16x32_bf16 v[84:87], v[212:215], a[92:95], v[84:87]\n\t"
    "ds_read_b64_tr_b16 v[204:205], v20 offset:20480\n\t"
    "buffer_load_dwordx4 v10, s[36:39], s88 offen offset:2048 lds\n\t"
    "v_mfma_f32_16x16x32_bf16 v[88:91], v[216:219], a[88:91], v[88:91]\n\t"
    "ds_read_b64_tr_b16 v[206:207], v20 offset:20992\n\t"
    "v_mfma_f32_16x16x32_bf16 v[92:95], v[216:219], a[92:95], v[92:95]\n\t"
    "ds_read_b64_tr_b16 v[208:209], v21 offset:20480\n\t"
    "s_waitcnt lgkmcnt(8)\n\t"
    "v_mfma_f32_16x16x32_bf16 v[64:67], v[188:191], a[96:99], v[64:67]\n\t"
    "ds_read_b64_tr_b16 v[210:211], v21 offset:20992\n\t"
    "buffer_load_dwordx4 v11, s[36:39], s88 offen offset:3072 lds\n\t"
    "s_lshr_b32 s88, s69, 12\n\t"
    "v_mfma_f32_16x16x32_bf16 v[68:71], v[188:191], a[100:103], v[68:71]\n\t"
    "ds_read_b64_tr_b16 v[212:213], v22 offset:20480\n\t"
    "s_add_u32 m0, s80, 98304\n\t"
    "s_add_u32 s88, s88, s73\n\t"
    "v_mfma_f32_16x16x32_bf16 v[72:75], v[192:195], a[96:99], v[72:75]\n\t"
    "ds_read_b64_tr_b16 v[214:215], v22 offset:20992\n\t"
    "buffer_load_dwordx4 v8, s[36:39], s88 offen offset:0 lds\n\t"
    "v_mfma_f32_16x16x32_bf16 v[76:79], v[192:195], a[100:103], v[76:79]\n\t"
    "ds_read_b64_tr_b16 v[216:217], v23 offset:20480\n\t"
    "s_waitcnt lgkmcnt(7)\n\t"
    "v_mfma_f32_16x16x32_bf16 v[80:83], v[196:199], a[96:99], v[80:83]\n\t"
    "ds_read_b64_tr_b16 v[218:219], v23 offset:20992\n\t"
    "v_mfma_f32_16x16x32_bf16 v[84:87], v[196:199], a[100:103], v[84:87]\n\t"
    "ds_read_b64_tr_b16 v[188:189], v20 offset:24576\n\t"
    "buffer_load_dwordx4 v9, s[36:39], s88 offen offset:1024 lds\n\t"
    "v_mfma_f32_16x16x32_bf16 v[88:91], v[200:203], a[96:99], v[88:91]\n\t"
    "ds_read_b64_tr_b16 v[190:191], v20 offset:25088\n\t"
    "v_mfma_f32_16x16x32_bf16 v[92:95], v[200:203], a[100:103], v[92:95]\n\t"
    "ds_read_b64_tr_b16 v[192:193], v21 offset:24576\n\t"
    "s_waitcnt lgkmcnt(7)\n\t"
    "buffer_load_dwordx4 v10, s[36:39], s88 offen offset:2048 lds\n\t"
    "v_mfma_f32_16x16x32_bf16 v[64:67], v[204:207], a[104:107], v[64:67]\n\t"
    "ds_read_b64_tr_b16 v[194:195], v21 offset:25088\n\t"
    "v_mfma_f32_16x16x32_bf16 v[68:71], v[204:207], a[108:111], v[68:71]\n\t"
    "ds_read_b64_tr_b16 v[196:197], v22 offset:24576\n\t"
    "buffer_load_dwordx4 v11, s[36:39], s88 offen offset:3072 lds\n\t"
    "s_add_u32 m0, s80, 114688\n\t"
    "v_mfma_f32_16x16x32_bf16 v[72:75], v[208:211], a[104:107], v[72:75]\n\t"
    "ds_read_b64_tr_b16 v[198:199], v22 offset:25088\n\t"
    "s_add_u32 s88, s69, 256\n\t"
    "v_mfma_f32_16x16x32_bf16 v[76:79], v[208:211], a[108:111], v[76:79]\n\t"
    "ds_read_b64_tr_b16 v[200:201], v23 offset:24576\n\t"
    "s_waitcnt lgkmcnt(7)\n\t"
    "v_mfma_f32_16x16x32_bf16 v[80:83], v[212:215], a[104:107], v[80:83]\n\t"
    "ds_read_b64_tr_b16 v[202:203], v23 offset:25088\n\t"
    "buffer_load_dwordx4 v12, s[40:43], s88 offen offset:0 lds\n\t"
    "v_mfma_f32_16x16x32_bf16 v[84:87], v[212:215], a[108:111], v[84:87]\n\t"
    "ds_read_b64_tr_b16 v[204:205], v20 offset:28672\n\t"
    "v_mfma_f32_16x16x32_bf16 v[88:91], v[216:219], a[104:107], v[88:91]\n\t"
    "ds_read_b64_tr_b16 v[206:207], v20 offset:29184\n\t"
    "buffer_load_dwordx4 v13, s[40:43], s88 offen offset:1024 lds\n\t"
    "v_mfma_f32_16x16x32_bf16 v[92:95], v[216:219], a[108:111], v[92:95]\n\t"
    "ds_read_b64_tr_b16 v[208:209], v21 offset:28672\n\t"
    "s_waitcnt lgkmcnt(7)\n\t"
    "v_mfma_f32_16x16x32_bf16 v[64:67], v[188:191], a[112:115], v[64:67]\n\t"
    "ds_read_b64_tr_b16 v[210:211], v21 offset:29184\n\t"
    "buffer_load_dwordx4 v14, s[40:43], s88 offen offset:2048 lds\n\t"
    "v_mfma_f32_16x16x32_bf16 v[68:71], v[188:191], a[116:119], v[68:71]\n\t"
    "ds_read_b64_tr_b16 v[212:213], v22 offset:28672\n\t"
    "v_mfma_f32_16x16x32_bf16 v[72:75], v[192:195], a[112:115], v[72:75]\n\t"
    "ds_read_b64_tr_b16 v[214:215], v22 offset:29184\n\t"
    "v_mfma_f32_16x16x32_bf16 v[76:79], v[192:195], a[116:119], v[76:79]\n\t"
    "s_waitcnt vmcnt(11)\n\t"
    "s_barrier\n\t"
    "ds_read_b64_tr_b16 v[216:217], v23 offset:28672\n\t"
    "s_waitcnt lgkmcnt(7)\n\t"
    "buffer_load_dwordx4 v15, s[40:43], s88 offen offset:3072 lds\n\t"
    "v_mfma_f32_16x16x32_bf16 v[80:83], v[196:199], a[112:115], v[80:83]\n\t"
    "ds_read_b64_tr_b16 v[218:219], v23 offset:29184\n\t"
    "ds_read_b128 a[128:131], v16 offset:32768\n\t"
    "ds_read_b128 a[132:135], v32 offset:0\n\t"
    "v_mfma_f32_16x16x32_bf16 v[84:87], v[196:199], a[116:119], v[84:87]\n\t"
    "ds_read_b64_tr_b16 v[188:189], v40 offset:16384\n\t"
    "ds_read_b128 a[136:139], v17 offset:32768\n\t"
    "ds_read_b128 a[140:143], v33 offset:0\n\t"
    "v_mfma_f32_16x16x32_bf16 v[88:91], v[200:203], a[112:115], v[88:91]\n\t"
    "ds_read_b64_tr_b16 v[190:191], v40 offset:16896\n\t"
    "ds_read_b128 a[144:147], v18 offset:32768\n\t"
    "ds_read_b128 a[148:151], v34 offset:0\n\t"
    "v_mfma_f32_16x16x32_bf16 v[92:95], v[200:203], a[116:119], v[92:95]\n\t"
    "ds_read_b64_tr_b16 v[192:193], v41 offset:16384\n\t"
    "s_waitcnt lgkmcnt(13)\n\t"
    "ds_read_b128 a[152:155], v19 offset:32768\n\t"
    "v_mfma_f32_16x16x32_bf16 v[64:67], v[204:207], a[120:123], v[64:67]\n\t"
    "ds_read_b64_tr_b16 v[194:195], v41 offset:16896\n\t"
    "ds_read_b128 a[156:159], v35 offset:0\n\t"
    "s_lshr_b32 s88, s70, 12\n\t"
    "v_mfma_f32_16x16x32_bf16 v[68:71], v[204:207], a[124:127], v[68:71]\n\t"
    "ds_read_b64_tr_b16 v[196:197], v42 offset:16384\n\t"
    "s_add_u32 m0, s80, 131072\n\t"
    "s_add_u32 s88, s88, s72\n\t"
    "v_mfma_f32_16x16x32_bf16 v[72:75], v[208:211], a[120:123], v[72:75]\n\t"
    "ds_read_b64_tr_b16 v[198:199], v42 offset:16896\n\t"
    "buffer_load_dwordx4 v8, s[36:39], s88 offen offset:0 lds\n\t"
    "v_mfma_f32_16x16x32_bf16 v[76:79], v[208:211], a[124:127], v[76:79]\n\t"
    "ds_read_b64_tr_b16 v[200:201], v43 offset:16384\n\t"
    "s_waitcnt lgkmcnt(15)\n\t"
    "buffer_load_dwordx4 v9, s[36:39], s88 offen offset:1024 lds\n\t"
    "v_mfma_f32_16x16x32_bf16 v[80:83], v[212:215], a[120:123], v[80:83]\n\t"
    "ds_read_b64_tr_b16 v[202:203], v43 offset:16896\n\t"
    "v_mfma_f32_16x16x32_bf16 v[84:87], v[212:215], a[124:127], v[84:87]\n\t"
    "ds_read_b64_tr_b16 v[204:205], v40 offset:20480\n\t"
    "buffer_load_dwordx4 v10, s[36:39], s88 offen offset:2048 lds\n\t"
    "v_mfma_f32_16x16x32_bf16 v[88:91], v[216:219], a[120:123], v[88:91]\n\t"
    "ds_read_b64_tr_b16 v[206:207], v40 offset:20992\n\t"
    "v_mfma_f32_16x16x32_bf16 v[92:95], v[216:219], a[124:127], v[92:95]\n\t"
    "ds_read_b64_tr_b16 v[208:209], v41 offset:20480\n\t"
    "s_waitcnt lgkmcnt(8)\n\t"
    "v_mfma_f32_16x16x32_bf16 v[64:67], v[188:191], a[128:131], v[64:67]\n\t"
    "ds_read_b64_tr_b16 v[210:211], v41 offset:20992\n\t"
    "buffer_load_dwordx4 v11, s[36:39], s88 offen offset:3072 lds\n\t"
    "s_lshr_b32 s88, s70, 12\n\t"
    "v_mfma_f32_16x16x32_bf16 v[68:71], v[188:191], a[132:135], v[68:71]\n\t"
    "ds_read_b64_tr_b16 v[212:213], v42 offset:20480\n\t"
    "s_add_u32 m0, s80, 147456\n\t"
    "s_add_u32 s88, s88, s73\n\t"
    "v_mfma_f32_16x16x32_bf16 v[72:75], v[192:195], a[128:131], v[72:75]\n\t"
    "ds_read_b64_tr_b16 v[214:215], v42 offset:20992\n\t"
    "buffer_load_dwordx4 v8, s[36:39], s88 offen offset:0 lds\n\t"
    "v_mfma_f32_16x16x32_bf16 v[76:79], v[192:195], a[132:135], v[76:79]\n\t"
    "ds_read_b64_tr_b16 v[216:217], v43 offset:20480\n\t"
    "s_waitcnt lgkmcnt(7)\n\t"
    "v_mfma_f32_16x16x32_bf16 v[80:83], v[196:199], a[128:131], v[80:83]\n\t"
    "ds_read_b64_tr_b16 v[218:219], v43 offset:20992\n\t"
    "v_mfma_f32_16x16x32_bf16 v[84:87], v[196:199], a[132:135], v[84:87]\n\t"
    "ds_read_b64_tr_b16 v[188:189], v40 offset:24576\n\t"
    "buffer_load_dwordx4 v9, s[36:39], s88 offen offset:1024 lds\n\t"
    "v_mfma_f32_16x16x32_bf16 v[88:91], v[200:203], a[128:131], v[88:91]\n\t"
    "ds_read_b64_tr_b16 v[190:191], v40 offset:25088\n\t"
    "v_mfma_f32_16x16x32_bf16 v[92:95], v[200:203], a[132:135], v[92:95]\n\t"
    "ds_read_b64_tr_b16 v[192:193], v41 offset:24576\n\t"
    "s_waitcnt lgkmcnt(7)\n\t"
    "buffer_load_dwordx4 v10, s[36:39], s88 offen offset:2048 lds\n\t"
    "v_mfma_f32_16x16x32_bf16 v[64:67], v[204:207], a[136:139], v[64:67]\n\t"
    "ds_read_b64_tr_b16 v[194:195], v41 offset:25088\n\t"
    "v_mfma_f32_16x16x32_bf16 v[68:71], v[204:207], a[140:143], v[68:71]\n\t"
    "ds_read_b64_tr_b16 v[196:197], v42 offset:24576\n\t"
    "buffer_load_dwordx4 v11, s[36:39], s88 offen offset:3072 lds\n\t"
    "s_add_u32 m0, s80, 0\n\t"
    "v_mfma_f32_16x16x32_bf16 v[72:75], v[208:211], a[136:139], v[72:75]\n\t"
    "ds_read_b64_tr_b16 v[198:199], v42 offset:25088\n\t"
    "s_add_u32 s88, s70, 256\n\t"
    "v_mfma_f32_16x16x32_bf16 v[76:79], v[208:211], a[140:143], v[76:79]\n\t"
    "ds_read_b64_tr_b16 v[200:201], v43 offset:24576\n\t"
    "s_waitcnt lgkmcnt(7)\n\t"
    "v_mfma_f32_16x16x32_bf16 v[80:83], v[212:215], a[136:139], v[80:83]\n\t"
    "ds_read_b64_tr_b16 v[202:203], v43 offset:25088\n\t"
    "buffer_load_dwordx4 v12, s[40:43], s88 offen offset:0 lds\n\t"
    "v_mfma_f32_16x16x32_bf16 v[84:87], v[212:215], a[140:143], v[84:87]\n\t"
    "ds_read_b64_tr_b16 v[204:205], v40 offset:28672\n\t"
    "v_mfma_f32_16x16x32_bf16 v[88:91], v[216:219], a[136:139], v[88:91]\n\t"
    "ds_read_b64_tr_b16 v[206:207], v40 offset:29184\n\t"
    "buffer_load_dwordx4 v13, s[40:43], s88 offen offset:1024 lds\n\t"
    "v_mfma_f32_16x16x32_bf16 v[92:95], v[216:219], a[140:143], v[92:95]\n\t"
    "ds_read_b64_tr_b16 v[208:209], v41 offset:28672\n\t"
    "s_waitcnt lgkmcnt(7)\n\t"
    "v_mfma_f32_16x16x32_bf16 v[64:67], v[188:191], a[144:147], v[64:67]\n\t"
    "ds_read_b64_tr_b16 v[210:211], v41 offset:29184\n\t"
    "buffer_load_dwordx4 v14, s[40:43], s88 offen offset:2048 lds\n\t"
    "v_mfma_f32_16x16x32_bf16 v[68:71], v[188:191], a[148:151], v[68:71]\n\t"
    "ds_read_b64_tr_b16 v[212:213], v42 offset:28672\n\t"
    "v_mfma_f32_16x16x32_bf16 v[72:75], v[192:195], a[144:147], v[72:75]\n\t"
    "ds_read_b64_tr_b16 v[214:215], v42 offset:29184\n\t"
    "v_mfma_f32_16x16x32_bf16 v[76:79], v[192:195], a[148:151], v[76:79]\n\t"
    "s_waitcnt vmcnt(11)\n\t"
    "s_barrier\n\t"
    "ds_read_b64_tr_b16 v[216:217], v43 offset:28672\n\t"
    "s_waitcnt lgkmcnt(7)\n\t"
    "buffer_load_dwordx4 v15, s[40:43], s88 offen offset:3072 lds\n\t"
    "v_mfma_f32_16x16x32_bf16 v[80:83], v[196:199], a[144:147], v[80:83]\n\t"
    "ds_read_b64_tr_b16 v[218:219], v43 offset:29184\n\t"
    "ds_read_b128 a[160:163], v16 offset:49152\n\t"
    "ds_read_b128 a[164:167], v32 offset:32768\n\t"
    "v_mfma_f32_16x16x32_bf16 v[84:87], v[196:199], a[148:151], v[84:87]\n\t"
    "ds_read_b64_tr_b16 v[188:189], v40 offset:49152\n\t"
    "ds_read_b128 a[168:171], v17 offset:49152\n\t"
    "ds_read_b128 a[172:175], v33 offset:32768\n\t"
    "v_mfma_f32_16x16x32_bf16 v[88:91], v[200:203], a[144:147], v[88:91]\n\t"
    "ds_read_b64_tr_b16 v[190:191], v40 offset:49664\n\t"
    "ds_read_b128 a[176:179], v18 offset:49152\n\t"
    "ds_read_b128 a[180:183], v34 offset:32768\n\t"
    "v_mfma_f32_16x16x32_bf16 v[92:95], v[200:203], a[148:151], v[92:95]\n\t"
    "ds_read_b64_tr_b16 v[192:193], v41 offset:49152\n\t"
    "s_waitcnt lgkmcnt(13)\n\t"
    "ds_read_b128 a[184:187], v19 offset:49152\n\t"
    "v_mfma_f32_16x16x32_bf16 v[64:67], v[204:207], a[152:155], v[64:67]\n\t"
    "ds_read_b64_tr_b16 v[194:195], v41 offset:49664\n\t"
    "ds_read_b128 a[188:191], v35 offset:32768\n\t"
    "s_lshr_b32 s88, s71, 12\n\t"
    "v_mfma_f32_16x16x32_bf16 v[68:71], v[204:207], a[156:159], v[68:71]\n\t"
    "ds_read_b64_tr_b16 v[196:197], v42 offset:49152\n\t"
    "s_add_u32 m0, s80, 16384\n\t"
    "s_add_u32 s88, s88, s72\n\t"
    "v_mfma_f32_16x16x32_bf16 v[72:75], v[208:211], a[152:155], v[72:75]\n\t"
    "ds_read_b64_tr_b16 v[198:199], v42 offset:49664\n\t"
    "buffer_load_dwordx4 v8, s[36:39], s88 offen offset:0 lds\n\t"
    "v_mfma_f32_16x16x32_bf16 v[76:79], v[208:211], a[156:159], v[76:79]\n\t"
    "ds_read_b64_tr_b16 v[200:201], v43 offset:49152\n\t"
    "s_waitcnt lgkmcnt(15)\n\t"
    "buffer_load_dwordx4 v9, s[36:39], s88 offen offset:1024 lds\n\t"
    "v_mfma_f32_16x16x32_bf16 v[80:83], v[212:215], a[152:155], v[80:83]\n\t"
    "ds_read_b64_tr_b16 v[202:203], v43 offset:49664\n\t"
    "v_mfma_f32_16x16x32_bf16 v[84:87], v[212:215], a[156:159], v[84:87]\n\t"
    "ds_read_b64_tr_b16 v[204:205], v40 offset:53248\n\t"
    "buffer_load_dwordx4 v10, s[36:39], s88 offen offset:2048 lds\n\t"
    "v_mfma_f32_16x16x32_bf16 v[88:91], v[216:219], a[152:155], v[88:91]\n\t"
    "ds_read_b64_tr_b16 v[206:207], v40 offset:53760\n\t"
    "v_mfma_f32_16x16x32_bf16 v[92:95], v[216:219], a[156:159], v[92:95]\n\t"
    "ds_read_b64_tr_b16 v[208:209], v41 offset:53248\n\t"
    "s_waitcnt lgkmcnt(8)\n\t"
    "v_mfma_f32_16x16x32_bf16 v[64:67], v[188:191], a[160:163], v[64:67]\n\t"
    "ds_read_b64_tr_b16 v[210:211], v41 offset:53760\n\t"
    "buffer_load_dwordx4 v11, s[36:39], s88 offen offset:3072 lds\n\t"
    "s_lshr_b32 s88, s71, 12\n\t"
    "v_mfma_f32_16x16x32_bf16 v[68:71], v[188:191], a[164:167], v[68:71]\n\t"
    "ds_read_b64_tr_b16 v[212:213], v42 offset:53248\n\t"
    "s_add_u32 m0, s80, 32768\n\t"
    "s_add_u32 s88, s88, s73\n\t"
    "v_mfma_f32_16x16x32_bf16 v[72:75], v[192:195], a[160:163], v[72:75]\n\t"
    "ds_read_b64_tr_b16 v[214:215], v42 offset:53760\n\t"
    "buffer_load_dwordx4 v8, s[36:39], s88 offen offset:0 lds\n\t"
    "v_mfma_f32_16x16x32_bf16 v[76:79], v[192:195], a[164:167], v[76:79]\n\t"
    "ds_read_b64_tr_b16 v[216:217], v43 offset:53248\n\t"
    "s_waitcnt lgkmcnt(7)\n\t"
    "v_mfma_f32_16x16x32_bf16 v[80:83], v[196:199], a[160:163], v[80:83]\n\t"
    "ds_read_b64_tr_b16 v[218:219], v43 offset:53760\n\t"
    "v_mfma_f32_16x16x32_bf16 v[84:87], v[196:199], a[164:167], v[84:87]\n\t"
    "ds_read_b64_tr_b16 v[188:189], v40 offset:57344\n\t"
    "buffer_load_dwordx4 v9, s[36:39], s88 offen offset:1024 lds\n\t"
    "v_mfma_f32_16x16x32_bf16 v[88:91], v[200:203], a[160:163], v[88:91]\n\t"
    "ds_read_b64_tr_b16 v[190:191], v40 offset:57856\n\t"
    "v_mfma_f32_16x16x32_bf16 v[92:95], v[200:203], a[164:167], v[92:95]\n\t"
    "ds_read_b64_tr_b16 v[192:193], v41 offset:57344\n\t"
    "s_waitcnt lgkmcnt(7)\n\t"
    "buffer_load_dwordx4 v10, s[36:39], s88 offen offset:2048 lds\n\t"
    "v_mfma_f32_16x16x32_bf16 v[64:67], v[204:207], a[168:171], v[64:67]\n\t"
    "ds_read_b64_tr_b16 v[194:195], v41 offset:57856\n\t"
    "v_mfma_f32_16x16x32_bf16 v[68:71], v[204:207], a[172:175], v[68:71]\n\t"
    "ds_read_b64_tr_b16 v[196:197], v42 offset:57344\n\t"
    "buffer_load_dwordx4 v11, s[36:39], s88 offen offset:3072 lds\n\t"
    "s_add_u32 m0, s80, 65536\n\t"
    "v_mfma_f32_16x16x32_bf16 v[72:75], v[208:211], a[168:171], v[72:75]\n\t"
    "ds_read_b64_tr_b16 v[198:199], v42 offset:57856\n\t"
    "s_add_u32 s88, s71, 256\n\t"
    "v_mfma_f32_16x16x32_bf16 v[76:79], v[208:211], a[172:175], v[76:79]\n\t"
    "ds_read_b64_tr_b16 v[200:201], v43 offset:57344\n\t"
    "s_waitcnt lgkmcnt(7)\n\t"
    "v_mfma_f32_16x16x32_bf16 v[80:83], v[212:215], a[168:171], v[80:83]\n\t"
    "ds_read_b64_tr_b16 v[202:203], v43 offset:57856\n\t"
    "buffer_load_dwordx4 v12, s[40:43], s88 offen offset:0 lds\n\t"
    "v_mfma_f32_16x16x32_bf16 v[84:87], v[212:215], a[172:175], v[84:87]\n\t"
    "ds_read_b64_tr_b16 v[204:205], v40 offset:61440\n\t"
    "v_mfma_f32_16x16x32_bf16 v[88:91], v[216:219], a[168:171], v[88:91]\n\t"
    "ds_read_b64_tr_b16 v[206:207], v40 offset:61952\n\t"
    "buffer_load_dwordx4 v13, s[40:43], s88 offen offset:1024 lds\n\t"
    "v_mfma_f32_16x16x32_bf16 v[92:95], v[216:219], a[172:175], v[92:95]\n\t"
    "ds_read_b64_tr_b16 v[208:209], v41 offset:61440\n\t"
    "s_waitcnt lgkmcnt(7)\n\t"
    "v_mfma_f32_16x16x32_bf16 v[64:67], v[188:191], a[176:179], v[64:67]\n\t"
    "ds_read_b64_tr_b16 v[210:211], v41 offset:61952\n\t"
    "buffer_load_dwordx4 v14, s[40:43], s88 offen offset:2048 lds\n\t"
    "v_mfma_f32_16x16x32_bf16 v[68:71], v[188:191], a[180:183], v[68:71]\n\t"
    "ds_read_b64_tr_b16 v[212:213], v42 offset:61440\n\t"
    "v_mfma_f32_16x16x32_bf16 v[72:75], v[192:195], a[176:179], v[72:75]\n\t"
    "ds_read_b64_tr_b16 v[214:215], v42 offset:61952\n\t"
    "v_mfma_f32_16x16x32_bf16 v[76:79], v[192:195], a[180:183], v[76:79]\n\t"
    "s_waitcnt vmcnt(11)\n\t"
    "s_barrier\n\t"
    "ds_read_b64_tr_b16 v[216:217], v43 offset:61440\n\t"
    "s_waitcnt lgkmcnt(7)\n\t"
    "buffer_load_dwordx4 v15, s[40:43], s88 offen offset:3072 lds\n\t"
    "v_mfma_f32_16x16x32_bf16 v[80:83], v[196:199], a[176:179], v[80:83]\n\t"
    "ds_read_b64_tr_b16 v[218:219], v43 offset:61952\n\t"
    "ds_read_b128 a[192:195], v36 offset:0\n\t"
    "ds_read_b128 a[196:199], v36 offset:16384\n\t"
    "v_mfma_f32_16x16x32_bf16 v[84:87], v[196:199], a[180:183], v[84:87]\n\t"
    "ds_read_b64_tr_b16 v[188:189], v20 offset:0\n\t"
    "ds_read_b128 a[200:203], v37 offset:0\n\t"
    "ds_read_b128 a[204:207], v37 offset:16384\n\t"
    "v_mfma_f32_16x16x32_bf16 v[88:91], v[200:203], a[176:179], v[88:91]\n\t"
    "ds_read_b64_tr_b16 v[190:191], v20 offset:512\n\t"
    "ds_read_b128 a[208:211], v38 offset:0\n\t"
    "ds_read_b128 a[212:215], v38 offset:16384\n\t"
    "v_mfma_f32_16x16x32_bf16 v[92:95], v[200:203], a[180:183], v[92:95]\n\t"
    "ds_read_b64_tr_b16 v[192:193], v21 offset:0\n\t"
    "s_waitcnt lgkmcnt(13)\n\t"
    "ds_read_b128 a[216:219], v39 offset:0\n\t"
    "v_mfma_f32_16x16x32_bf16 v[64:67], v[204:207], a[184:187], v[64:67]\n\t"
    "ds_read_b64_tr_b16 v[194:195], v21 offset:512\n\t"
    "ds_read_b128 a[220:223], v39 offset:16384\n\t"
    "v_readfirstlane_b32 s91, v54\n\t"
    "v_mfma_f32_16x16x32_bf16 v[68:71], v[204:207], a[188:191], v[68:71]\n\t"
    "ds_read_b64_tr_b16 v[196:197], v22 offset:0\n\t"
    "v_readfirstlane_b32 s60, v55\n\t"
    "s_nop 3\n\t"
    "v_mfma_f32_16x16x32_bf16 v[72:75], v[208:211], a[184:187], v[72:75]\n\t"
    "ds_read_b64_tr_b16 v[198:199], v22 offset:512\n\t"
    "s_sub_u32 s60, s62, s60\n\t"
    "s_add_u32 s91, s91, 1\n\t"
    "v_mfma_f32_16x16x32_bf16 v[76:79], v[208:211], a[188:191], v[76:79]\n\t"
    "ds_read_b64_tr_b16 v[200:201], v23 offset:0\n\t"
    "s_waitcnt lgkmcnt(15)\n\t"
    "s_and_b32 s89, s91, 31\n\t"
    "v_mfma_f32_16x16x32_bf16 v[80:83], v[212:215], a[184:187], v[80:83]\n\t"
    "ds_read_b64_tr_b16 v[202:203], v23 offset:512\n\t"
    "s_or_b32 s89, s89, 32\n\t"
    "v_mov_b32 v48, s89\n\t"
    "v_mfma_f32_16x16x32_bf16 v[84:87], v[212:215], a[188:191], v[84:87]\n\t"
    "ds_read_b64_tr_b16 v[204:205], v20 offset:4096\n\t"
    "v_mov_b32 v50, s91\n\t"
    "s_add_u32 m0, s80, 81920\n\t"
    "v_mfma_f32_16x16x32_bf16 v[88:91], v[216:219], a[184:187], v[88:91]\n\t"
    "ds_read_b64_tr_b16 v[206:207], v20 offset:4608\n\t"
    "s_add_u32 s88, s64, 0\n\t"
    "buffer_load_dwordx4 v12, s[40:43], s88 offen offset:0 lds\n\t"
    "v_mfma_f32_16x16x32_bf16 v[92:95], v[216:219], a[188:191], v[92:95]\n\t"
    "ds_read_b64_tr_b16 v[208:209], v21 offset:4096\n\t"
    "s_waitcnt lgkmcnt(8)\n\t"
    "buffer_load_dwordx4 v13, s[40:43], s88 offen offset:1024 lds\n\t"
    "v_mfma_f32_16x16x32_bf16 v[64:67], v[188:191], a[192:195], v[64:67]\n\t"
    "ds_read_b64_tr_b16 v[210:211], v21 offset:4608\n\t"
    "v_mfma_f32_16x16x32_bf16 v[68:71], v[188:191], a[196:199], v[68:71]\n\t"
    "ds_read_b64_tr_b16 v[212:213], v22 offset:4096\n\t"
    "buffer_load_dwordx4 v14, s[40:43], s88 offen offset:2048 lds\n\t"
    "v_mfma_f32_16x16x32_bf16 v[72:75], v[192:195], a[192:195], v[72:75]\n\t"
    "ds_read_b64_tr_b16 v[214:215], v22 offset:4608\n\t"
    "v_mfma_f32_16x16x32_bf16 v[76:79], v[192:195], a[196:199], v[76:79]\n\t"
    "ds_read_b64_tr_b16 v[216:217], v23 offset:4096\n\t"
    "s_waitcnt lgkmcnt(7)\n\t"
    "v_mfma_f32_16x16x32_bf16 v[80:83], v[196:199], a[192:195], v[80:83]\n\t"
    "ds_read_b64_tr_b16 v[218:219], v23 offset:4608\n\t"
    "buffer_load_dwordx4 v15, s[40:43], s88 offen offset:3072 lds\n\t"
    "s_add_u32 m0, s80, 49152\n\t"
    "v_mfma_f32_16x16x32_bf16 v[84:87], v[196:199], a[196:199], v[84:87]\n\t"
    "ds_read_b64_tr_b16 v[188:189], v20 offset:8192\n\t"
    "s_add_u32 s88, s65, 0\n\t"
    "v_mfma_f32_16x16x32_bf16 v[88:91], v[200:203], a[192:195], v[88:91]\n\t"
    "ds_read_b64_tr_b16 v[190:191], v20 offset:8704\n\t"
    "buffer_load_dwordx4 v12, s[40:43], s88 offen offset:0 lds\n\t"
    "v_mfma_f32_16x16x32_bf16 v[92:95], v[200:203], a[196:199], v[92:95]\n\t"
    "ds_read_b64_tr_b16 v[192:193], v21 offset:8192\n\t"
    "s_waitcnt lgkmcnt(7)\n\t"
    "v_mfma_f32_16x16x32_bf16 v[64:67], v[204:207], a[200:203], v[64:67]\n\t"
    "ds_read_b64_tr_b16 v[194:195], v21 offset:8704\n\t"
    "v_mfma_f32_16x16x32_bf16 v[68:71], v[204:207], a[204:207], v[68:71]\n\t"
    "ds_read_b64_tr_b16 v[196:197], v22 offset:8192\n\t"
    "buffer_load_dwordx4 v13, s[40:43], s88 offen offset:1024 lds\n\t"
    "v_mfma_f32_16x16x32_bf16 v[72:75], v[208:211], a[200:203], v[72:75]\n\t"
    "ds_read_b64_tr_b16 v[198:199], v22 offset:8704\n\t"
    "v_mfma_f32_16x16x32_bf16 v[76:79], v[208:211], a[204:207], v[76:79]\n\t"
    "ds_read_b64_tr_b16 v[200:201], v23 offset:8192\n\t"
    "s_waitcnt lgkmcnt(7)\n\t"
    "buffer_load_dwordx4 v14, s[40:43], s88 offen offset:2048 lds\n\t"
    "v_mfma_f32_16x16x32_bf16 v[80:83], v[212:215], a[200:203], v[80:83]\n\t"
    "ds_read_b64_tr_b16 v[202:203], v23 offset:8704\n\t"
    "v_mfma_f32_16x16x32_bf16 v[84:87], v[212:215], a[204:207], v[84:87]\n\t"
    "ds_read_b64_tr_b16 v[204:205], v20 offset:12288\n\t"
    "buffer_load_dwordx4 v15, s[40:43], s88 offen offset:3072 lds\n\t"
    "s_add_u32 m0, s80, 98304\n\t"
    "v_mfma_f32_16x16x32_bf16 v[88:91], v[216:219], a[200:203], v[88:91]\n\t"
    "ds_read_b64_tr_b16 v[206:207], v20 offset:12800\n\t"
    "s_add_u32 s88, s66, 0\n\t"
    "v_mfma_f32_16x16x32_bf16 v[92:95], v[216:219], a[204:207], v[92:95]\n\t"
    "ds_read_b64_tr_b16 v[208:209], v21 offset:12288\n\t"
    "s_waitcnt lgkmcnt(7)\n\t"
    "v_mfma_f32_16x16x32_bf16 v[64:67], v[188:191], a[208:211], v[64:67]\n\t"
    "ds_read_b64_tr_b16 v[210:211], v21 offset:12800\n\t"
    "buffer_load_dwordx4 v12, s[40:43], s88 offen offset:0 lds\n\t"
    "v_mfma_f32_16x16x32_bf16 v[68:71], v[188:191], a[212:215], v[68:71]\n\t"
    "ds_read_b64_tr_b16 v[212:213], v22 offset:12288\n\t"
    "v_mfma_f32_16x16x32_bf16 v[72:75], v[192:195], a[208:211], v[72:75]\n\t"
    "ds_read_b64_tr_b16 v[214:215], v22 offset:12800\n\t"
    "buffer_load_dwordx4 v13, s[40:43], s88 offen offset:1024 lds\n\t"
    "v_mfma_f32_16x16x32_bf16 v[76:79], v[192:195], a[212:215], v[76:79]\n\t"
    "s_waitcnt vmcnt(10)\n\t"
    "s_barrier\n\t"
    "ds_read_b64_tr_b16 v[216:217], v23 offset:12288\n\t"
    "s_waitcnt lgkmcnt(7)\n\t"
    "buffer_load_dwordx4 v14, s[40:43], s88 offen offset:2048 lds\n\t"
    "v_mfma_f32_16x16x32_bf16 v[80:83], v[196:199], a[208:211], v[80:83]\n\t"
    "ds_read_b64_tr_b16 v[218:219], v23 offset:12800\n\t"
    "ds_read_b128 a[224:227], v16 offset:16384\n\t"
    "ds_read_b128 a[228:231], v16 offset:32768\n\t"
    "v_mfma_f32_16x16x32_bf16 v[84:87], v[196:199], a[212:215], v[84:87]\n\t"
    "ds_read_b64_tr_b16 v[188:189], v40 offset:0\n\t"
    "ds_read_b128 a[232:235], v17 offset:16384\n\t"
    "ds_read_b128 a[236:239], v17 offset:32768\n\t"
    "v_mfma_f32_16x16x32_bf16 v[88:91], v[200:203], a[208:211], v[88:91]\n\t"
    "ds_read_b64_tr_b16 v[190:191], v40 offset:512\n\t"
    "ds_read_b128 a[240:243], v18 offset:16384\n\t"
    "ds_read_b128 a[244:247], v18 offset:32768\n\t"
    "v_mfma_f32_16x16x32_bf16 v[92:95], v[200:203], a[212:215], v[92:95]\n\t"
    "ds_read_b64_tr_b16 v[192:193], v41 offset:0\n\t"
    "s_waitcnt lgkmcnt(13)\n\t"
    "buffer_load_dwordx4 v15, s[40:43], s88 offen offset:3072 lds\n\t"
    "v_mfma_f32_16x16x32_bf16 v[64:67], v[204:207], a[216:219], v[64:67]\n\t"
    "ds_read_b64_tr_b16 v[194:195], v41 offset:512\n\t"
    "ds_read_b128 a[248:251], v19 offset:16384\n\t"
    "ds_read_b128 a[252:255], v19 offset:32768\n\t"
    "v_mfma_f32_16x16x32_bf16 v[68:71], v[204:207], a[220:223], v[68:71]\n\t"
    "ds_read_b64_tr_b16 v[196:197], v42 offset:0\n\t"
    "s_add_u32 m0, s80, 114688\n\t"
    "s_add_u32 s88, s67, 0\n\t"
    "v_mfma_f32_16x16x32_bf16 v[72:75], v[208:211], a[216:219], v[72:75]\n\t"
    "ds_read_b64_tr_b16 v[198:199], v42 offset:512\n\t"
    "buffer_load_dwordx4 v12, s[40:43], s88 offen offset:0 lds\n\t"
    "v_mfma_f32_16x16x32_bf16 v[76:79], v[208:211], a[220:223], v[76:79]\n\t"
    "ds_read_b64_tr_b16 v[200:201], v43 offset:0\n\t"
    "s_waitcnt lgkmcnt(15)\n\t"
    "v_mfma_f32_16x16x32_bf16 v[80:83], v[212:215], a[216:219], v[80:83]\n\t"
    "ds_read_b64_tr_b16 v[202:203], v43 offset:512\n\t"
    "v_mfma_f32_16x16x32_bf16 v[84:87], v[212:215], a[220:223], v[84:87]\n\t"
    "ds_read_b64_tr_b16 v[204:205], v40 offset:4096\n\t"
    "buffer_load_dwordx4 v13, s[40:43], s88 offen offset:1024 lds\n\t"
    "v_mfma_f32_16x16x32_bf16 v[88:91], v[216:219], a[216:219], v[88:91]\n\t"
    "ds_read_b64_tr_b16 v[206:207], v40 offset:4608\n\t"
    "v_mfma_f32_16x16x32_bf16 v[92:95], v[216:219], a[220:223], v[92:95]\n\t"
    "ds_read_b64_tr_b16 v[208:209], v41 offset:4096\n\t"
    "s_waitcnt lgkmcnt(9)\n\t"
    "v_mfma_f32_16x16x32_bf16 v[64:67], v[188:191], a[224:227], v[64:67]\n\t"
    "ds_read_b64_tr_b16 v[210:211], v41 offset:4608\n\t"
    "buffer_load_dwordx4 v14, s[40:43], s88 offen offset:2048 lds\n\t"
    "v_mfma_f32_16x16x32_bf16 v[68:71], v[188:191], a[228:231], v[68:71]\n\t"
    "ds_read_b64_tr_b16 v[212:213], v42 offset:4096\n\t"
    "v_mfma_f32_16x16x32_bf16 v[72:75], v[192:195], a[224:227], v[72:75]\n\t"
    "ds_read_b64_tr_b16 v[214:215], v42 offset:4608\n\t"
    "v_mfma_f32_16x16x32_bf16 v[76:79], v[192:195], a[228:231], v[76:79]\n\t"
    "ds_read_b64_tr_b16 v[216:217], v43 offset:4096\n\t"
    "s_waitcnt lgkmcnt(7)\n\t"
    "buffer_load_dwordx4 v15, s[40:43], s88 offen offset:3072 lds\n\t"
    "v_mfma_f32_16x16x32_bf16 v[80:83], v[196:199], a[224:227], v[80:83]\n\t"
    "ds_read_b64_tr_b16 v[218:219], v43 offset:4608\n\t"
    "s_add_u32 m0, s80, 131072\n\t"
    "s_add_u32 s88, s68, 0\n\t"
    "v_mfma_f32_16x16x32_bf16 v[84:87], v[196:199], a[228:231], v[84:87]\n\t"
    "ds_read_b64_tr_b16 v[188:189], v40 offset:8192\n\t"
    "v_mfma_f32_16x16x32_bf16 v[88:91], v[200:203], a[224:227], v[88:91]\n\t"
    "ds_read_b64_tr_b16 v[190:191], v40 offset:8704\n\t"
    "buffer_load_dwordx4 v12, s[40:43], s88 offen offset:0 lds\n\t"
    "v_mfma_f32_16x16x32_bf16 v[92:95], v[200:203], a[228:231], v[92:95]\n\t"
    "ds_read_b64_tr_b16 v[192:193], v41 offset:8192\n\t"
    "s_waitcnt lgkmcnt(7)\n\t"
    "v_mfma_f32_16x16x32_bf16 v[64:67], v[204:207], a[232:235], v[64:67]\n\t"
    "ds_read_b64_tr_b16 v[194:195], v41 offset:8704\n\t"
    "v_mfma_f32_16x16x32_bf16 v[68:71], v[204:207], a[236:239], v[68:71]\n\t"
    "ds_read_b64_tr_b16 v[196:197], v42 offset:8192\n\t"
    "buffer_load_dwordx4 v13, s[40:43], s88 offen offset:1024 lds\n\t"
    "v_mfma_f32_16x16x32_bf16 v[72:75], v[208:211], a[232:235], v[72:75]\n\t"
    "ds_read_b64_tr_b16 v[198:199], v42 offset:8704\n\t"
    "v_mfma_f32_16x16x32_bf16 v[76:79], v[208:211], a[236:239], v[76:79]\n\t"
    "ds_read_b64_tr_b16 v[200:201], v43 offset:8192\n\t"
    "s_waitcnt lgkmcnt(7)\n\t"
    "v_mfma_f32_16x16x32_bf16 v[80:83], v[212:215], a[232:235], v[80:83]\n\t"
    "ds_read_b64_tr_b16 v[202:203], v43 offset:8704\n\t"
    "buffer_load_dwordx4 v14, s[40:43], s88 offen offset:2048 lds\n\t"
    "v_mfma_f32_16x16x32_bf16 v[84:87], v[212:215], a[236:239], v[84:87]\n\t"
    "ds_read_b64_tr_b16 v[204:205], v40 offset:12288\n\t"
    "v_mfma_f32_16x16x32_bf16 v[88:91], v[216:219], a[232:235], v[88:91]\n\t"
    "ds_read_b64_tr_b16 v[206:207], v40 offset:12800\n\t"
    "v_mfma_f32_16x16x32_bf16 v[92:95], v[216:219], a[236:239], v[92:95]\n\t"
    "ds_read_b64_tr_b16 v[208:209], v41 offset:12288\n\t"
    "s_waitcnt lgkmcnt(7)\n\t"
    "buffer_load_dwordx4 v15, s[40:43], s88 offen offset:3072 lds\n\t"
    "v_mfma_f32_16x16x32_bf16 v[64:67], v[188:191], a[240:243], v[64:67]\n\t"
    "ds_read_b64_tr_b16 v[210:211], v41 offset:12800\n\t"
    "v_mfma_f32_16x16x32_bf16 v[68:71], v[188:191], a[244:247], v[68:71]\n\t"
    "ds_read_b64_tr_b16 v[212:213], v42 offset:12288\n\t"
    "v_mfma_f32_16x16x32_bf16 v[72:75], v[192:195], a[240:243], v[72:75]\n\t"
    "ds_read_b64_tr_b16 v[214:215], v42 offset:12800\n\t"
    "v_mfma_f32_16x16x32_bf16 v[76:79], v[192:195], a[244:247], v[76:79]\n\t"
    "s_waitcnt vmcnt(16)\n\t"
    "s_barrier\n\t"
    "ds_read_b64_tr_b16 v[216:217], v43 offset:12288\n\t"
    "s_waitcnt lgkmcnt(7)\n\t"
    "s_add_u32 m0, s80, 147456\n\t"
    "v_mfma_f32_16x16x32_bf16 v[80:83], v[196:199], a[240:243], v[80:83]\n\t"
    "ds_read_b64_tr_b16 v[218:219], v43 offset:12800\n\t"
    "s_add_u32 s88, s64, 512\n\t"
    "v_mfma_f32_16x16x32_bf16 v[84:87], v[196:199], a[244:247], v[84:87]\n\t"
    "ds_read_b64_tr_b16 v[188:189], v40 offset:16384\n\t"
    "v_mfma_f32_16x16x32_bf16 v[88:91], v[200:203], a[240:243], v[88:91]\n\t"
    "ds_read_b64_tr_b16 v[190:191], v40 offset:16896\n\t"
    "v_mfma_f32_16x16x32_bf16 v[92:95], v[200:203], a[244:247], v[92:95]\n\t"
    "ds_read_b64_tr_b16 v[192:193], v41 offset:16384\n\t"
    "s_waitcnt lgkmcnt(7)\n\t"
    "v_mfma_f32_16x16x32_bf16 v[64:67], v[204:207], a[248:251], v[64:67]\n\t"
    "ds_read_b64_tr_b16 v[194:195], v41 offset:16896\n\t"
    "v_mfma_f32_16x16x32_bf16 v[68:71], v[204:207], a[252:255], v[68:71]\n\t"
    "ds_read_b64_tr_b16 v[196:197], v42 offset:16384\n\t"
    "v_mfma_f32_16x16x32_bf16 v[72:75], v[208:211], a[248:251], v[72:75]\n\t"
    "ds_read_b64_tr_b16 v[198:199], v42 offset:16896\n\t"
    "v_mfma_f32_16x16x32_bf16 v[76:79], v[208:211], a[252:255], v[76:79]\n\t"
    "ds_read_b64_tr_b16 v[200:201], v43 offset:16384\n\t"
    "s_waitcnt lgkmcnt(7)\n\t"
    "v_mfma_f32_16x16x32_bf16 v[80:83], v[212:215], a[248:251], v[80:83]\n\t"
    "ds_read_b64_tr_b16 v[202:203], v43 offset:16896\n\t"
    "v_mfma_f32_16x16x32_bf16 v[84:87], v[212:215], a[252:255], v[84:87]\n\t"
    "ds_read_b64_tr_b16 v[204:205], v40 offset:20480\n\t"
    "v_mfma_f32_16x16x32_bf16 v[88:91], v[216:219], a[248:251], v[88:91]\n\t"
    "ds_read_b64_tr_b16 v[206:207], v40 offset:20992\n\t"
    "v_mfma_f32_16x16x32_bf16 v[92:95], v[216:219], a[252:255], v[92:95]\n\t"
    "s_cmp_lt_u32 s60, 10000\n\t"
    "s_cbranch_scc0 LP256%=\n\t"
    "s_sleep 13\n\t"
    "LP256%=:\n\t"
    "ds_read_b64_tr_b16 v[208:209], v41 offset:20480\n\t"
    "s_waitcnt lgkmcnt(7)\n\t"
    "v_mov_b32 v96, v64\n\t"
    "v_mov_b32 v97, v65\n\t"
    "v_mov_b32 v98, v66\n\t"
    "v_mov_b32 v99, v67\n\t"
    "v_mfma_f32_16x16x32_bf16 v[64:67], v[188:191], a[0:3], 0\n\t"
    "ds_read_b64_tr_b16 v[210:211], v41 offset:20992\n\t"
    "v_cvt_pk_bf16_f32 v112, v68, v69\n\t"
    "v_cvt_pk_bf16_f32 v113, v70, v71\n\t"
    "v_mfma_f32_16x16x32_bf16 v[68:71], v[188:191], a[4:7], 0\n\t"
    "ds_read_b64_tr_b16 v[212:213], v42 offset:20480\n\t"
    "v_mov_b32 v100, v72\n\t"
    "v_mov_b32 v101, v73\n\t"
    "v_mov_b32 v102, v74\n\t"
    "v_mov_b32 v103, v75\n\t"
    "v_mfma_f32_16x16x32_bf16 v[72:75], v[192:195], a[0:3], 0\n\t"
    "ds_read_b64_tr_b16 v[214:215], v42 offset:20992\n\t"
    "v_cvt_pk_bf16_f32 v114, v76, v77\n\t"
    "v_cvt_pk_bf16_f32 v115, v78, v79\n\t"
    "v_mfma_f32_16x16x32_bf16 v[76:79], v[192:195], a[4:7], 0\n\t"
    "global_store_dwordx4 v24, v[112:115], s[84:85] sc0 sc1\n\t"
    "ds_read_b64_tr_b16 v[216:217], v43 offset:20480\n\t"
    "s_waitcnt lgkmcnt(7)\n\t"
    "v_cvt_pk_bf16_f32 v104, v80, v81\n\t"
    "v_cvt_pk_bf16_f32 v105, v82, v83\n\t"
    "v_mfma_f32_16x16x32_bf16 v[80:83], v[196:199], a[0:3], 0\n\t"
    "ds_read_b64_tr_b16 v[218:219], v43 offset:20992\n\t"
    "v_cvt_pk_bf16_f32 v120, v84, v85\n\t"
    "v_cvt_pk_bf16_f32 v121, v86, v87\n\t"
    "v_mfma_f32_16x16x32_bf16 v[84:87], v[196:199], a[4:7], 0\n\t"
    "ds_read_b64_tr_b16 v[188:189], v40 offset:24576\n\t"
    "v_cvt_pk_bf16_f32 v106, v88, v89\n\t"
    "v_cvt_pk_bf16_f32 v107, v90, v91\n\t"
    "v_mfma_f32_16x16x32_bf16 v[88:91], v[200:203], a[0:3], 0\n\t"
    "global_store_dwordx4 v24, v[104:107], s[82:83] sc0 sc1\n\t"
    "ds_read_b64_tr_b16 v[190:191], v40 offset:25088\n\t"
    "v_cvt_pk_bf16_f32 v122, v92, v93\n\t"
    "v_cvt_pk_bf16_f32 v123, v94, v95\n\t"
    "v_mfma_f32_16x16x32_bf16 v[92:95], v[200:203], a[4:7], 0\n\t"
    "ds_read_b64_tr_b16 v[192:193], v41 offset:24576\n\t"
    "s_waitcnt lgkmcnt(7)\n\t"
    "v_mfma_f32_16x16x32_bf16 v[64:67], v[204:207], a[8:11], v[64:67]\n\t"
    "ds_read_b64_tr_b16 v[194:195], v41 offset:25088\n\t"
    "v_mfma_f32_16x16x32_bf16 v[68:71], v[204:207], a[12:15], v[68:71]\n\t"
    "global_store_dwordx4 v24, v[120:123], s[86:87] sc0 sc1\n\t"
    "ds_read_b64_tr_b16 v[196:197], v42 offset:24576\n\t"
    "v_mfma_f32_16x16x32_bf16 v[72:75], v[208:211], a[8:11], v[72:75]\n\t"
    "ds_read_b64_tr_b16 v[198:199], v42 offset:25088\n\t"
    "v_mfma_f32_16x16x32_bf16 v[76:79], v[208:211], a[12:15], v[76:79]\n\t"
    "ds_read_b64_tr_b16 v[200:201], v43 offset:24576\n\t"
    "s_waitcnt lgkmcnt(7)\n\t"
    "v_mfma_f32_16x16x32_bf16 v[80:83], v[212:215], a[8:11], v[80:83]\n\t"
    "ds_read_b64_tr_b16 v[202:203], v43 offset:25088\n\t"
    "v_mfma_f32_16x16x32_bf16 v[84:87], v[212:215], a[12:15], v[84:87]\n\t"
    "ds_read_b64_tr_b16 v[204:205], v40 offset:28672\n\t"
    "buffer_load_dwordx4 v12, s[40:43], s88 offen offset:0 lds\n\t"
    "v_mfma_f32_16x16x32_bf16 v[88:91], v[216:219], a[8:11], v[88:91]\n\t"
    "ds_read_b64_tr_b16 v[206:207], v40 offset:29184\n\t"
    "v_mfma_f32_16x16x32_bf16 v[92:95], v[216:219], a[12:15], v[92:95]\n\t"
    "ds_read_b64_tr_b16 v[208:209], v41 offset:28672\n\t"
    "s_waitcnt lgkmcnt(7)\n\t"
    "v_mfma_f32_16x16x32_bf16 v[64:67], v[188:191], a[16:19], v[64:67]\n\t"
    "ds_read_b64_tr_b16 v[210:211], v41 offset:29184\n\t"
    "buffer_load_dwordx4 v13, s[40:43], s88 offen offset:1024 lds\n\t"
    "v_mfma_f32_16x16x32_bf16 v[68:71], v[188:191], a[20:23], v[68:71]\n\t"
    "ds_read_b64_tr_b16 v[212:213], v42 offset:28672\n\t"
    "buffer_load_dwordx4 v14, s[40:43], s88 offen offset:2048 lds\n\t"
    "v_mfma_f32_16x16x32_bf16 v[72:75], v[192:195], a[16:19], v[72:75]\n\t"
    "ds_read_b64_tr_b16 v[214:215], v42 offset:29184\n\t"
    "buffer_load_dwordx4 v15, s[40:43], s88 offen offset:3072 lds\n\t"
    "v_mfma_f32_16x16x32_bf16 v[76:79], v[192:195], a[20:23], v[76:79]\n\t"
    "s_waitcnt vmcnt(19)\n\t"
    "s_barrier\n\t"
    "ds_read_b64_tr_b16 v[216:217], v43 offset:28672\n\t"
    "s_waitcnt lgkmcnt(7)\n\t"
    "v_mfma_f32_16x16x32_bf16 v[80:83], v[196:199], a[16:19], v[80:83]\n\t"
    "ds_read_b64_tr_b16 v[218:219], v43 offset:29184\n\t"
    "v_mfma_f32_16x16x32_bf16 v[84:87], v[196:199], a[20:23], v[84:87]\n\t"
    "ds_read_b64_tr_b16 v[188:189], v20 offset:49152\n\t"
    "v_mfma_f32_16x16x32_bf16 v[88:91], v[200:203], a[16:19], v[88:91]\n\t"
    "ds_read_b64_tr_b16 v[190:191], v20 offset:49664\n\t"
    "v_mfma_f32_16x16x32_bf16 v[92:95], v[200:203], a[20:23], v[92:95]\n\t"
    "ds_read_b64_tr_b16 v[192:193], v21 offset:49152\n\t"
    "s_waitcnt lgkmcnt(7)\n\t"
    "v_mfma_f32_16x16x32_bf16 v[64:67], v[204:207], a[24:27], v[64:67]\n\t"
    "ds_read_b64_tr_b16 v[194:195], v21 offset:49664\n\t"
    "v_mfma_f32_16x16x32_bf16 v[68:71], v[204:207], a[28:31], v[68:71]\n\t"
    "ds_read_b64_tr_b16 v[196:197], v22 offset:49152\n\t"
    "v_mfma_f32_16x16x32_bf16 v[72:75], v[208:211], a[24:27], v[72:75]\n\t"
    "ds_read_b64_tr_b16 v[198:199], v22 offset:49664\n\t"
    "v_mfma_f32_16x16x32_bf16 v[76:79], v[208:211], a[28:31], v[76:79]\n\t"
    "ds_read_b64_tr_b16 v[200:201], v23 offset:49152\n\t"
    "s_waitcnt lgkmcnt(7)\n\t"
    "v_mfma_f32_16x16x32_bf16 v[80:83], v[212:215], a[24:27], v[80:83]\n\t"
    "ds_read_b64_tr_b16 v[202:203], v23 offset:49664\n\t"
    "v_mfma_f32_16x16x32_bf16 v[84:87], v[212:215], a[28:31], v[84:87]\n\t"
    "ds_read_b64_tr_b16 v[204:205], v20 offset:53248\n\t"
    "v_mfma_f32_16x16x32_bf16 v[88:91], v[216:219], a[24:27], v[88:91]\n\t"
    "ds_read_b64_tr_b16 v[206:207], v20 offset:53760\n\t"
    "v_mfma_f32_16x16x32_bf16 v[92:95], v[216:219], a[28:31], v[92:95]\n\t"
    "s_cmp_lt_u32 s60, 10000\n\t"
    "s_cbranch_scc0 LP288%=\n\t"
    "s_sleep 13\n\t"
    "LP288%=:\n\t"
    "ds_read_b64_tr_b16 v[208:209], v21 offset:53248\n\t"
    "s_waitcnt lgkmcnt(7)\n\t"
    "v_mfma_f32_16x16x32_bf16 v[64:67], v[188:191], a[32:35], v[64:67]\n\t"
    "ds_read_b64_tr_b16 v[210:211], v21 offset:53760\n\t"
    "v_mfma_f32_16x16x32_bf16 v[68:71], v[188:191], a[36:39], v[68:71]\n\t"
    "ds_read_b64_tr_b16 v[212:213], v22 offset:53248\n\t"
    "v_mfma_f32_16x16x32_bf16 v[72:75], v[192:195], a[32:35], v[72:75]\n\t"
    "ds_read_b64_tr_b16 v[214:215], v22 offset:53760\n\t"
    "v_mfma_f32_16x16x32_bf16 v[76:79], v[192:195], a[36:39], v[76:79]\n\t"
    "ds_read_b64_tr_b16 v[216:217], v23 offset:53248\n\t"
    "s_waitcnt lgkmcnt(7)\n\t"
    "v_mfma_f32_16x16x32_bf16 v[80:83], v[196:199], a[32:35], v[80:83]\n\t"
    "ds_read_b64_tr_b16 v[218:219], v23 offset:53760\n\t"
    "v_mfma_f32_16x16x32_bf16 v[84:87], v[196:199], a[36:39], v[84:87]\n\t"
    "ds_read_b64_tr_b16 v[188:189], v20 offset:57344\n\t"
    "v_mfma_f32_16x16x32_bf16 v[88:91], v[200:203], a[32:35], v[88:91]\n\t"
    "ds_read_b64_tr_b16 v[190:191], v20 offset:57856\n\t"
    "v_mfma_f32_16x16x32_bf16 v[92:95], v[200:203], a[36:39], v[92:95]\n\t"
    "ds_read_b64_tr_b16 v[192:193], v21 offset:57344\n\t"
    "s_waitcnt lgkmcnt(7)\n\t"
    "v_mfma_f32_16x16x32_bf16 v[64:67], v[204:207], a[40:43], v[64:67]\n\t"
    "ds_read_b64_tr_b16 v[194:195], v21 offset:57856\n\t"
    "v_mfma_f32_16x16x32_bf16 v[68:71], v[204:207], a[44:47], v[68:71]\n\t"
    "ds_read_b64_tr_b16 v[196:197], v22 offset:57344\n\t"
    "v_mfma_f32_16x16x32_bf16 v[72:75], v[208:211], a[40:43], v[72:75]\n\t"
    "ds_read_b64_tr_b16 v[198:199], v22 offset:57856\n\t"
    "v_mfma_f32_16x16x32_bf16 v[76:79], v[208:211], a[44:47], v[76:79]\n\t"
    "ds_read_b64_tr_b16 v[200:201], v23 offset:57344\n\t"
    "s_waitcnt lgkmcnt(7)\n\t"
    "v_mfma_f32_16x16x32_bf16 v[80:83], v[212:215], a[40:43], v[80:83]\n\t"
    "ds_read_b64_tr_b16 v[202:203], v23 offset:57856\n\t"
    "v_mfma_f32_16x16x32_bf16 v[84:87], v[212:215], a[44:47], v[84:87]\n\t"
    "ds_read_b64_tr_b16 v[204:205], v20 offset:61440\n\t"
    "v_mfma_f32_16x16x32_bf16 v[88:91], v[216:219], a[40:43], v[88:91]\n\t"
    "ds_read_b64_tr_b16 v[206:207], v20 offset:61952\n\t"
    "v_mfma_f32_16x16x32_bf16 v[92:95], v[216:219], a[44:47], v[92:95]\n\t"
    "ds_read_b64_tr_b16 v[208:209], v21 offset:61440\n\t"
    "s_waitcnt lgkmcnt(7)\n\t"
    "v_mfma_f32_16x16x32_bf16 v[64:67], v[188:191], a[48:51], v[64:67]\n\t"
    "ds_read_b64_tr_b16 v[210:211], v21 offset:61952\n\t"
    "v_mfma_f32_16x16x32_bf16 v[68:71], v[188:191], a[52:55], v[68:71]\n\t"
    "ds_read_b64_tr_b16 v[212:213], v22 offset:61440\n\t"
    "v_mfma_f32_16x16x32_bf16 v[72:75], v[192:195], a[48:51], v[72:75]\n\t"
    "ds_read_b64_tr_b16 v[214:215], v22 offset:61952\n\t"
    "v_mfma_f32_16x16x32_bf16 v[76:79], v[192:195], a[52:55], v[76:79]\n\t"
    "s_waitcnt vmcnt(15)\n\t"
    "s_barrier\n\t"
    "ds_read_b64_tr_b16 v[216:217], v23 offset:61440\n\t"
    "s_waitcnt lgkmcnt(7)\n\t"
    "s_add_u32 m0, s80, 0\n\t"
    "s_add_u32 s88, s65, 512\n\t"
    "v_mfma_f32_16x16x32_bf16 v[80:83], v[196:199], a[48:51], v[80:83]\n\t"
    "ds_read_b64_tr_b16 v[218:219], v23 offset:61952\n\t"
    "buffer_load_dwordx4 v12, s[40:43], s88 offen offset:0 lds\n\t"
    "s_waitcnt vmcnt(5)\n\t"
    "s_barrier\n\t"
    "buffer_store_dword v50, v28, s[56:59], 0 offen offset:0 sc0 sc1\n\t"
    "v_mfma_f32_16x16x32_bf16 v[84:87], v[196:199], a[52:55], v[84:87]\n\t"
    "ds_read_b64_tr_b16 v[188:189], v40 offset:32768\n\t"
    "v_mfma_f32_16x16x32_bf16 v[88:91], v[200:203], a[48:51], v[88:91]\n\t"
    "ds_read_b64_tr_b16 v[190:191], v40 offset:33280\n\t"
    "buffer_load_dwordx4 v13, s[40:43], s88 offen offset:1024 lds\n\t"
    "v_mfma_f32_16x16x32_bf16 v[92:95], v[200:203], a[52:55], v[92:95]\n\t"
    "ds_read_b64_tr_b16 v[192:193], v41 offset:32768\n\t"
    "s_waitcnt lgkmcnt(7)\n\t"
    "v_mfma_f32_16x16x32_bf16 v[64:67], v[204:207], a[56:59], v[64:67]\n\t"
    "ds_read_b64_tr_b16 v[194:195], v41 offset:33280\n\t"
    "v_mfma_f32_16x16x32_bf16 v[68:71], v[204:207], a[60:63], v[68:71]\n\t"
    "ds_read_b64_tr_b16 v[196:197], v42 offset:32768\n\t"
    "buffer_load_dwordx4 v14, s[40:43], s88 offen offset:2048 lds\n\t"
    "v_mfma_f32_16x16x32_bf16 v[72:75], v[208:211], a[56:59], v[72:75]\n\t"
    "ds_read_b64_tr_b16 v[198:199], v42 offset:33280\n\t"
    "v_mfma_f32_16x16x32_bf16 v[76:79], v[208:211], a[60:63], v[76:79]\n\t"
    "ds_read_b64_tr_b16 v[200:201], v43 offset:32768\n\t"
    "s_waitcnt lgkmcnt(7)\n\t"
    "v_mfma_f32_16x16x32_bf16 v[80:83], v[212:215], a[56:59], v[80:83]\n\t"
    "ds_read_b64_tr_b16 v[202:203], v43 offset:33280\n\t"
    "buffer_load_dwordx4 v15, s[40:43], s88 offen offset:3072 lds\n\t"
    "v_mfma_f32_16x16x32_bf16 v[84:87], v[212:215], a[60:63], v[84:87]\n\t"
    "ds_read_b64_tr_b16 v[204:205], v40 offset:36864\n\t"
    "v_mfma_f32_16x16x32_bf16 v[88:91], v[216:219], a[56:59], v[88:91]\n\t"
    "ds_read_b64_tr_b16 v[206:207], v40 offset:37376\n\t"
    "v_mfma_f32_16x16x32_bf16 v[92:95], v[216:219], a[60:63], v[92:95]\n\t"
    "s_cmp_lt_u32 s60, 10000\n\t"
    "s_cbranch_scc0 LP320%=\n\t"
    "s_sleep 13\n\t"
    "LP320%=:\n\t"
    "ds_read_b64_tr_b16 v[208:209], v41 offset:36864\n\t"
    "s_waitcnt lgkmcnt(7)\n\t"
    "v_mfma_f32_16x16x32_bf16 v[64:67], v[188:191], a[64:67], v[64:67]\n\t"
    "ds_read_b64_tr_b16 v[210:211], v41 offset:37376\n\t"
    "v_mfma_f32_16x16x32_bf16 v[68:71], v[188:191], a[68:71], v[68:71]\n\t"
    "ds_read_b64_tr_b16 v[212:213], v42 offset:36864\n\t"
    "v_mfma_f32_16x16x32_bf16 v[72:75], v[192:195], a[64:67], v[72:75]\n\t"
    "ds_read_b64_tr_b16 v[214:215], v42 offset:37376\n\t"
    "v_mfma_f32_16x16x32_bf16 v[76:79], v[192:195], a[68:71], v[76:79]\n\t"
    "ds_read_b64_tr_b16 v[216:217], v43 offset:36864\n\t"
    "s_waitcnt lgkmcnt(7)\n\t"
    "v_mfma_f32_16x16x32_bf16 v[80:83], v[196:199], a[64:67], v[80:83]\n\t"
    "ds_read_b64_tr_b16 v[218:219], v43 offset:37376\n\t"
    "v_mfma_f32_16x16x32_bf16 v[84:87], v[196:199], a[68:71], v[84:87]\n\t"
    "ds_read_b64_tr_b16 v[188:189], v40 offset:40960\n\t"
    "v_mfma_f32_16x16x32_bf16 v[88:91], v[200:203], a[64:67], v[88:91]\n\t"
    "ds_read_b64_tr_b16 v[190:191], v40 offset:41472\n\t"
    "v_mfma_f32_16x16x32_bf16 v[92:95], v[200:203], a[68:71], v[92:95]\n\t"
    "ds_read_b64_tr_b16 v[192:193], v41 offset:40960\n\t"
    "s_waitcnt lgkmcnt(7)\n\t"
    "v_mfma_f32_16x16x32_bf16 v[64:67], v[204:207], a[72:75], v[64:67]\n\t"
    "ds_read_b64_tr_b16 v[194:195], v41 offset:41472\n\t"
    "v_mfma_f32_16x16x32_bf16 v[68:71], v[204:207], a[76:79], v[68:71]\n\t"
    "ds_read_b64_tr_b16 v[196:197], v42 offset:40960\n\t"
    "v_mfma_f32_16x16x32_bf16 v[72:75], v[208:211], a[72:75], v[72:75]\n\t"
    "ds_read_b64_tr_b16 v[198:199], v42 offset:41472\n\t"
    "v_mfma_f32_16x16x32_bf16 v[76:79], v[208:211], a[76:79], v[76:79]\n\t"
    "ds_read_b64_tr_b16 v[200:201], v43 offset:40960\n\t"
    "s_waitcnt lgkmcnt(7)\n\t"
    "v_mfma_f32_16x16x32_bf16 v[80:83], v[212:215], a[72:75], v[80:83]\n\t"
    "ds_read_b64_tr_b16 v[202:203], v43 offset:41472\n\t"
    "v_mfma_f32_16x16x32_bf16 v[84:87], v[212:215], a[76:79], v[84:87]\n\t"
    "ds_read_b64_tr_b16 v[204:205], v40 offset:45056\n\t"
    "v_mfma_f32_16x16x32_bf16 v[88:91], v[216:219], a[72:75], v[88:91]\n\t"
    "ds_read_b64_tr_b16 v[206:207], v40 offset:45568\n\t"
    "v_mfma_f32_16x16x32_bf16 v[92:95], v[216:219], a[76:79], v[92:95]\n\t"
    "ds_read_b64_tr_b16 v[208:209], v41 offset:45056\n\t"
    "s_waitcnt lgkmcnt(7)\n\t"
    "v_mfma_f32_16x16x32_bf16 v[64:67], v[188:191], a[80:83], v[64:67]\n\t"
    "ds_read_b64_tr_b16 v[210:211], v41 offset:45568\n\t"
    "v_mfma_f32_16x16x32_bf16 v[68:71], v[188:191], a[84:87], v[68:71]\n\t"
    "ds_read_b64_tr_b16 v[212:213], v42 offset:45056\n\t"
    "v_mfma_f32_16x16x32_bf16 v[72:75], v[192:195], a[80:83], v[72:75]\n\t"
    "ds_read_b64_tr_b16 v[214:215], v42 offset:45568\n\t"
    "v_mfma_f32_16x16x32_bf16 v[76:79], v[192:195], a[84:87], v[76:79]\n\t"
    "s_waitcnt vmcnt(16)\n\t"
    "s_barrier\n\t"
    "ds_read_b64_tr_b16 v[216:217], v43 offset:45056\n\t"
    "s_waitcnt lgkmcnt(7)\n\t"
    "v_mfma_f32_16x16x32_bf16 v[80:83], v[196:199], a[80:83], v[80:83]\n\t"
    "ds_read_b64_tr_b16 v[218:219], v43 offset:45568\n\t"
    "s_add_u32 m0, s80, 16384\n\t"
    "v_mfma_f32_16x16x32_bf16 v[84:87], v[196:199], a[84:87], v[84:87]\n\t"
    "ds_read_b64_tr_b16 v[188:189], v40 offset:49152\n\t"
    "s_add_u32 s88, s69, 0\n\t"
    "v_mfma_f32_16x16x32_bf16 v[88:91], v[200:203], a[80:83], v[88:91]\n\t"
    "ds_read_b64_tr_b16 v[190:191], v40 offset:49664\n\t"
    "v_mfma_f32_16x16x32_bf16 v[92:95], v[200:203], a[84:87], v[92:95]\n\t"
    "ds_read_b64_tr_b16 v[192:193], v41 offset:49152\n\t"
    "s_waitcnt lgkmcnt(7)\n\t"
    "v_mfma_f32_16x16x32_bf16 v[64:67], v[204:207], a[88:91], v[64:67]\n\t"
    "ds_read_b64_tr_b16 v[194:195], v41 offset:49664\n\t"
    "v_mfma_f32_16x16x32_bf16 v[68:71], v[204:207], a[92:95], v[68:71]\n\t"
    "ds_read_b64_tr_b16 v[196:197], v42 offset:49152\n\t"
    "v_mfma_f32_16x16x32_bf16 v[72:75], v[208:211], a[88:91], v[72:75]\n\t"
    "ds_read_b64_tr_b16 v[198:199], v42 offset:49664\n\t"
    "v_mfma_f32_16x16x32_bf16 v[76:79], v[208:211], a[92:95], v[76:79]\n\t"
    "ds_read_b64_tr_b16 v[200:201], v43 offset:49152\n\t"
    "s_waitcnt lgkmcnt(7)\n\t"
    "v_mfma_f32_16x16x32_bf16 v[80:83], v[212:215], a[88:91], v[80:83]\n\t"
    "ds_read_b64_tr_b16 v[202:203], v43 offset:49664\n\t"
    "v_mfma_f32_16x16x32_bf16 v[84:87], v[212:215], a[92:95], v[84:87]\n\t"
    "ds_read_b64_tr_b16 v[204:205], v40 offset:53248\n\t"
    "v_mfma_f32_16x16x32_bf16 v[88:91], v[216:219], a[88:91], v[88:91]\n\t"
    "ds_read_b64_tr_b16 v[206:207], v40 offset:53760\n\t"
    "v_mfma_f32_16x16x32_bf16 v[92:95], v[216:219], a[92:95], v[92:95]\n\t"
    "s_cmp_lt_u32 s60, 10000\n\t"
    "s_cbranch_scc0 LP352%=\n\t"
    "s_sleep 13\n\t"
    "LP352%=:\n\t"
    "ds_read_b64_tr_b16 v[208:209], v41 offset:53248\n\t"
    "s_waitcnt lgkmcnt(7)\n\t"
    "buffer_load_dword v56, v29, s[56:59], 0 offen offset:0 sc0 sc1\n\t"
    "v_mfma_f32_16x16x32_bf16 v[64:67], v[188:191], a[96:99], v[64:67]\n\t"
    "ds_read_b64_tr_b16 v[210:211], v41 offset:53760\n\t"
    "v_mfma_f32_16x16x32_bf16 v[68:71], v[188:191], a[100:103], v[68:71]\n\t"
    "ds_read_b64_tr_b16 v[212:213], v42 offset:53248\n\t"
    "v_mfma_f32_16x16x32_bf16 v[72:75], v[192:195], a[96:99], v[72:75]\n\t"
    "ds_read_b64_tr_b16 v[214:215], v42 offset:53760\n\t"
    "v_mfma_f32_16x16x32_bf16 v[76:79], v[192:195], a[100:103], v[76:79]\n\t"
    "ds_read_b64_tr_b16 v[216:217], v43 offset:53248\n\t"
    "s_waitcnt lgkmcnt(7)\n\t"
    "v_mfma_f32_16x16x32_bf16 v[80:83], v[196:199], a[96:99], v[80:83]\n\t"
    "ds_read_b64_tr_b16 v[218:219], v43 offset:53760\n\t"
    "buffer_load_dwordx4 v12, s[40:43], s88 offen offset:0 lds\n\t"
    "v_mfma_f32_16x16x32_bf16 v[84:87], v[196:199], a[100:103], v[84:87]\n\t"
    "ds_read_b64_tr_b16 v[188:189], v40 offset:57344\n\t"
    "v_mfma_f32_16x16x32_bf16 v[88:91], v[200:203], a[96:99], v[88:91]\n\t"
    "ds_read_b64_tr_b16 v[190:191], v40 offset:57856\n\t"
    "buffer_load_dwordx4 v13, s[40:43], s88 offen offset:1024 lds\n\t"
    "v_mfma_f32_16x16x32_bf16 v[92:95], v[200:203], a[100:103], v[92:95]\n\t"
    "ds_read_b64_tr_b16 v[192:193], v41 offset:57344\n\t"
    "s_waitcnt lgkmcnt(7)\n\t"
    "v_mfma_f32_16x16x32_bf16 v[64:67], v[204:207], a[104:107], v[64:67]\n\t"
    "ds_read_b64_tr_b16 v[194:195], v41 offset:57856\n\t"
    "v_mfma_f32_16x16x32_bf16 v[68:71], v[204:207], a[108:111], v[68:71]\n\t"
    "ds_read_b64_tr_b16 v[196:197], v42 offset:57344\n\t"
    "buffer_load_dwordx4 v14, s[40:43], s88 offen offset:2048 lds\n\t"
    "v_mfma_f32_16x16x32_bf16 v[72:75], v[208:211], a[104:107], v[72:75]\n\t"
    "ds_read_b64_tr_b16 v[198:199], v42 offset:57856\n\t"
    "v_mfma_f32_16x16x32_bf16 v[76:79], v[208:211], a[108:111], v[76:79]\n\t"
    "ds_read_b64_tr_b16 v[200:201], v43 offset:57344\n\t"
    "s_waitcnt lgkmcnt(7)\n\t"
    "v_mfma_f32_16x16x32_bf16 v[80:83], v[212:215], a[104:107], v[80:83]\n\t"
    "ds_read_b64_tr_b16 v[202:203], v43 offset:57856\n\t"
    "buffer_load_dwordx4 v15, s[40:43], s88 offen offset:3072 lds\n\t"
    "v_mfma_f32_16x16x32_bf16 v[84:87], v[212:215], a[108:111], v[84:87]\n\t"
    "ds_read_b64_tr_b16 v[204:205], v40 offset:61440\n\t"
    "v_mfma_f32_16x16x32_bf16 v[88:91], v[216:219], a[104:107], v[88:91]\n\t"
    "ds_read_b64_tr_b16 v[206:207], v40 offset:61952\n\t"
    "v_mfma_f32_16x16x32_bf16 v[92:95], v[216:219], a[108:111], v[92:95]\n\t"
    "ds_read_b64_tr_b16 v[208:209], v41 offset:61440\n\t"
    "s_waitcnt lgkmcnt(7)\n\t"
    "v_mfma_f32_16x16x32_bf16 v[64:67], v[188:191], a[112:115], v[64:67]\n\t"
    "ds_read_b64_tr_b16 v[210:211], v41 offset:61952\n\t"
    "v_mfma_f32_16x16x32_bf16 v[68:71], v[188:191], a[116:119], v[68:71]\n\t"
    "ds_read_b64_tr_b16 v[212:213], v42 offset:61440\n\t"
    "v_mfma_f32_16x16x32_bf16 v[72:75], v[192:195], a[112:115], v[72:75]\n\t"
    "ds_read_b64_tr_b16 v[214:215], v42 offset:61952\n\t"
    "v_mfma_f32_16x16x32_bf16 v[76:79], v[192:195], a[116:119], v[76:79]\n\t"
    "s_waitcnt vmcnt(17)\n\t"
    "s_barrier\n\t"
    "ds_read_b64_tr_b16 v[216:217], v43 offset:61440\n\t"
    "s_waitcnt lgkmcnt(7)\n\t"
    "v_mfma_f32_16x16x32_bf16 v[80:83], v[196:199], a[112:115], v[80:83]\n\t"
    "ds_read_b64_tr_b16 v[218:219], v43 offset:61952\n\t"
    "s_add_u32 m0, s80, 32768\n\t"
    "v_mfma_f32_16x16x32_bf16 v[84:87], v[196:199], a[116:119], v[84:87]\n\t"
    "ds_read_b64_tr_b16 v[188:189], v44 offset:0\n\t"
    "s_add_u32 s88, s70, 0\n\t"
    "v_mfma_f32_16x16x32_bf16 v[88:91], v[200:203], a[112:115], v[88:91]\n\t"
    "ds_read_b64_tr_b16 v[190:191], v44 offset:512\n\t"
    "v_mfma_f32_16x16x32_bf16 v[92:95], v[200:203], a[116:119], v[92:95]\n\t"
    "ds_read_b64_tr_b16 v[192:193], v45 offset:0\n\t"
    "s_waitcnt lgkmcnt(7)\n\t"
    "v_mfma_f32_16x16x32_bf16 v[64:67], v[204:207], a[120:123], v[64:67]\n\t"
    "ds_read_b64_tr_b16 v[194:195], v45 offset:512\n\t"
    "v_mfma_f32_16x16x32_bf16 v[68:71], v[204:207], a[124:127], v[68:71]\n\t"
    "ds_read_b64_tr_b16 v[196:197], v46 offset:0\n\t"
    "v_mfma_f32_16x16x32_bf16 v[72:75], v[208:211], a[120:123], v[72:75]\n\t"
    "ds_read_b64_tr_b16 v[198:199], v46 offset:512\n\t"
    "v_mfma_f32_16x16x32_bf16 v[76:79], v[208:211], a[124:127], v[76:79]\n\t"
    "ds_read_b64_tr_b16 v[200:201], v47 offset:0\n\t"
    "s_waitcnt lgkmcnt(7)\n\t"
    "v_mfma_f32_16x16x32_bf16 v[80:83], v[212:215], a[120:123], v[80:83]\n\t"
    "ds_read_b64_tr_b16 v[202:203], v47 offset:512\n\t"
    "v_mfma_f32_16x16x32_bf16 v[84:87], v[212:215], a[124:127], v[84:87]\n\t"
    "ds_read_b64_tr_b16 v[204:205], v44 offset:4096\n\t"
    "v_mfma_f32_16x16x32_bf16 v[88:91], v[216:219], a[120:123], v[88:91]\n\t"
    "ds_read_b64_tr_b16 v[206:207], v44 offset:4608\n\t"
    "v_mfma_f32_16x16x32_bf16 v[92:95], v[216:219], a[124:127], v[92:95]\n\t"
    "s_cmp_lt_u32 s60, 10000\n\t"
    "s_cbranch_scc0 LP384%=\n\t"
    "s_sleep 13\n\t"
    "LP384%=:\n\t"
    "ds_read_b64_tr_b16 v[208:209], v45 offset:4096\n\t"
    "s_waitcnt lgkmcnt(7)\n\t"
    "v_mfma_f32_16x16x32_bf16 v[64:67], v[188:191], a[128:131], v[64:67]\n\t"
    "ds_read_b64_tr_b16 v[210:211], v45 offset:4608\n\t"
    "v_mfma_f32_16x16x32_bf16 v[68:71], v[188:191], a[132:135], v[68:71]\n\t"
    "ds_read_b64_tr_b16 v[212:213], v46 offset:4096\n\t"
    "v_mfma_f32_16x16x32_bf16 v[72:75], v[192:195], a[128:131], v[72:75]\n\t"
    "ds_read_b64_tr_b16 v[214:215], v46 offset:4608\n\t"
    "v_mfma_f32_16x16x32_bf16 v[76:79], v[192:195], a[132:135], v[76:79]\n\t"
    "ds_read_b64_tr_b16 v[216:217], v47 offset:4096\n\t"
    "s_waitcnt lgkmcnt(7)\n\t"
    "v_mfma_f32_16x16x32_bf16 v[80:83], v[196:199], a[128:131], v[80:83]\n\t"
    "ds_read_b64_tr_b16 v[218:219], v47 offset:4608\n\t"
    "v_mfma_f32_16x16x32_bf16 v[84:87], v[196:199], a[132:135], v[84:87]\n\t"
    "ds_read_b64_tr_b16 v[188:189], v44 offset:8192\n\t"
    "s_waitcnt vmcnt(4)\n\t"
    "v_cmp_gt_u32 vcc, v50, v56\n\t"
    "s_nop 0\n\t"
    "s_cbranch_vccnz LS0%=\n\t"
    "LB0%=:\n\t"
    "buffer_load_dwordx4 v[128:131], v24, s[44:47], s77 offen offset:0 sc0 sc1\n\t"
    "v_mfma_f32_16x16x32_bf16 v[88:91], v[200:203], a[128:131], v[88:91]\n\t"
    "ds_read_b64_tr_b16 v[190:191], v44 offset:8704\n\t"
    "v_mfma_f32_16x16x32_bf16 v[92:95], v[200:203], a[132:135], v[92:95]\n\t"
    "ds_read_b64_tr_b16 v[192:193], v45 offset:8192\n\t"
    "s_waitcnt lgkmcnt(7)\n\t"
    "v_mfma_f32_16x16x32_bf16 v[64:67], v[204:207], a[136:139], v[64:67]\n\t"
    "ds_read_b64_tr_b16 v[194:195], v45 offset:8704\n\t"
    "buffer_load_dwordx4 v12, s[40:43], s88 offen offset:0 lds\n\t"
    "v_mfma_f32_16x16x32_bf16 v[68:71], v[204:207], a[140:143], v[68:71]\n\t"
    "ds_read_b64_tr_b16 v[196:197], v46 offset:8192\n\t"
    "buffer_load_dwordx4 v[136:139], v24, s[44:47], s78 offen offset:0 sc0 sc1\n\t"
    "v_mfma_f32_16x16x32_bf16 v[72:75], v[208:211], a[136:139], v[72:75]\n\t"
    "ds_read_b64_tr_b16 v[198:199], v46 offset:8704\n\t"
    "buffer_load_dwordx4 v13, s[40:43], s88 offen offset:1024 lds\n\t"
    "v_mfma_f32_16x16x32_bf16 v[76:79], v[208:211], a[140:143], v[76:79]\n\t"
    "ds_read_b64_tr_b16 v[200:201], v47 offset:8192\n\t"
    "s_waitcnt lgkmcnt(7)\n\t"
    "buffer_load_dwordx4 v[144:147], v24, s[44:47], s79 offen offset:0 sc0 sc1\n\t"
    "v_mfma_f32_16x16x32_bf16 v[80:83], v[212:215], a[136:139], v[80:83]\n\t"
    "ds_read_b64_tr_b16 v[202:203], v47 offset:8704\n\t"
    "buffer_load_dwordx4 v14, s[40:43], s88 offen offset:2048 lds\n\t"
    "v_mfma_f32_16x16x32_bf16 v[84:87], v[212:215], a[140:143], v[84:87]\n\t"
    "ds_read_b64_tr_b16 v[204:205], v44 offset:12288\n\t"
    "v_mfma_f32_16x16x32_bf16 v[88:91], v[216:219], a[136:139], v[88:91]\n\t"
    "ds_read_b64_tr_b16 v[206:207], v44 offset:12800\n\t"
    "v_mfma_f32_16x16x32_bf16 v[92:95], v[216:219], a[140:143], v[92:95]\n\t"
    "ds_read_b64_tr_b16 v[208:209], v45 offset:12288\n\t"
    "s_waitcnt lgkmcnt(7)\n\t"
    "v_mfma_f32_16x16x32_bf16 v[64:67], v[188:191], a[144:147], v[64:67]\n\t"
    "ds_read_b64_tr_b16 v[210:211], v45 offset:12800\n\t"
    "buffer_load_dwordx4 v15, s[40:43], s88 offen offset:3072 lds\n\t"
    "v_mfma_f32_16x16x32_bf16 v[68:71], v[188:191], a[148:151], v[68:71]\n\t"
    "ds_read_b64_tr_b16 v[212:213], v46 offset:12288\n\t"
    "v_mfma_f32_16x16x32_bf16 v[72:75], v[192:195], a[144:147], v[72:75]\n\t"
    "ds_read_b64_tr_b16 v[214:215], v46 offset:12800\n\t"
    "v_mfma_f32_16x16x32_bf16 v[76:79], v[192:195], a[148:151], v[76:79]\n\t"
    "s_waitcnt vmcnt(7)\n\t"
    "s_barrier\n\t"
    "ds_read_b64_tr_b16 v[216:217], v47 offset:12288\n\t"
    "s_waitcnt lgkmcnt(7)\n\t"
    "v_mfma_f32_16x16x32_bf16 v[80:83], v[196:199], a[144:147], v[80:83]\n\t"
    "ds_read_b64_tr_b16 v[218:219], v47 offset:12800\n\t"
    "s_add_u32 m0, s80, 65536\n\t"
    "v_mfma_f32_16x16x32_bf16 v[84:87], v[196:199], a[148:151], v[84:87]\n\t"
    "ds_read_b64_tr_b16 v[188:189], v20 offset:16384\n\t"
    "s_add_u32 s88, s71, 0\n\t"
    "v_mfma_f32_16x16x32_bf16 v[88:91], v[200:203], a[144:147], v[88:91]\n\t"
    "ds_read_b64_tr_b16 v[190:191], v20 offset:16896\n\t"
    "buffer_load_dwordx4 v12, s[40:43], s88 offen offset:0 lds\n\t"
    "v_mfma_f32_16x16x32_bf16 v[92:95], v[200:203], a[148:151], v[92:95]\n\t"
    "ds_read_b64_tr_b16 v[192:193], v21 offset:16384\n\t"
    "s_waitcnt lgkmcnt(7)\n\t"
    "v_mfma_f32_16x16x32_bf16 v[64:67], v[204:207], a[152:155], v[64:67]\n\t"
    "ds_read_b64_tr_b16 v[194:195], v21 offset:16896\n\t"
    "buffer_load_dwordx4 v13, s[40:43], s88 offen offset:1024 lds\n\t"
    "v_mfma_f32_16x16x32_bf16 v[68:71], v[204:207], a[156:159], v[68:71]\n\t"
    "ds_read_b64_tr_b16 v[196:197], v22 offset:16384\n\t"
    "v_mfma_f32_16x16x32_bf16 v[72:75], v[208:211], a[152:155], v[72:75]\n\t"
    "ds_read_b64_tr_b16 v[198:199], v22 offset:16896\n\t"
    "v_mfma_f32_16x16x32_bf16 v[76:79], v[208:211], a[156:159], v[76:79]\n\t"
    "ds_read_b64_tr_b16 v[200:201], v23 offset:16384\n\t"
    "s_waitcnt lgkmcnt(7)\n\t"
    "v_mfma_f32_16x16x32_bf16 v[80:83], v[212:215], a[152:155], v[80:83]\n\t"
    "ds_read_b64_tr_b16 v[202:203], v23 offset:16896\n\t"
    "buffer_load_dwordx4 v14, s[40:43], s88 offen offset:2048 lds\n\t"
    "v_mfma_f32_16x16x32_bf16 v[84:87], v[212:215], a[156:159], v[84:87]\n\t"
    "ds_read_b64_tr_b16 v[204:205], v20 offset:20480\n\t"
    "v_mfma_f32_16x16x32_bf16 v[88:91], v[216:219], a[152:155], v[88:91]\n\t"
    "ds_read_b64_tr_b16 v[206:207], v20 offset:20992\n\t"
    "v_mfma_f32_16x16x32_bf16 v[92:95], v[216:219], a[156:159], v[92:95]\n\t"
    "s_cmp_lt_u32 s60, 10000\n\t"
    "s_cbranch_scc0 LP416%=\n\t"
    "s_sleep 13\n\t"
    "LP416%=:\n\t"
    "ds_read_b64_tr_b16 v[208:209], v21 offset:20480\n\t"
    "s_waitcnt lgkmcnt(7)\n\t"
    "v_mfma_f32_16x16x32_bf16 v[64:67], v[188:191], a[160:163], v[64:67]\n\t"
    "ds_read_b64_tr_b16 v[210:211], v21 offset:20992\n\t"
    "buffer_load_dwordx4 v15, s[40:43], s88 offen offset:3072 lds\n\t"
    "v_mfma_f32_16x16x32_bf16 v[68:71], v[188:191], a[164:167], v[68:71]\n\t"
    "ds_read_b64_tr_b16 v[212:213], v22 offset:20480\n\t"
    "s_add_u32 m0, s80, 81920\n\t"
    "v_mfma_f32_16x16x32_bf16 v[72:75], v[192:195], a[160:163], v[72:75]\n\t"
    "ds_read_b64_tr_b16 v[214:215], v22 offset:20992\n\t"
    "s_add_u32 s88, s66, 512\n\t"
    "v_mfma_f32_16x16x32_bf16 v[76:79], v[192:195], a[164:167], v[76:79]\n\t"
    "ds_read_b64_tr_b16 v[216:217], v23 offset:20480\n\t"
    "s_waitcnt lgkmcnt(7)\n\t"
    "v_mfma_f32_16x16x32_bf16 v[80:83], v[196:199], a[160:163], v[80:83]\n\t"
    "ds_read_b64_tr_b16 v[218:219], v23 offset:20992\n\t"
    "buffer_load_dwordx4 v12, s[40:43], s88 offen offset:0 lds\n\t"
    "v_mfma_f32_16x16x32_bf16 v[84:87], v[196:199], a[164:167], v[84:87]\n\t"
    "ds_read_b64_tr_b16 v[188:189], v20 offset:24576\n\t"
    "v_mfma_f32_16x16x32_bf16 v[88:91], v[200:203], a[160:163], v[88:91]\n\t"
    "ds_read_b64_tr_b16 v[190:191], v20 offset:25088\n\t"
    "v_mfma_f32_16x16x32_bf16 v[92:95], v[200:203], a[164:167], v[92:95]\n\t"
    "ds_read_b64_tr_b16 v[192:193], v21 offset:24576\n\t"
    "s_waitcnt lgkmcnt(7)\n\t"
    "v_mfma_f32_16x16x32_bf16 v[64:67], v[204:207], a[168:171], v[64:67]\n\t"
    "ds_read_b64_tr_b16 v[194:195], v21 offset:25088\n\t"
    "buffer_load_dwordx4 v13, s[40:43], s88 offen offset:1024 lds\n\t"
    "v_mfma_f32_16x16x32_bf16 v[68:71], v[204:207], a[172:175], v[68:71]\n\t"
    "ds_read_b64_tr_b16 v[196:197], v22 offset:24576\n\t"
    "v_mfma_f32_16x16x32_bf16 v[72:75], v[208:211], a[168:171], v[72:75]\n\t"
    "ds_read_b64_tr_b16 v[198:199], v22 offset:25088\n\t"
    "v_mfma_f32_16x16x32_bf16 v[76:79], v[208:211], a[172:175], v[76:79]\n\t"
    "ds_read_b64_tr_b16 v[200:201], v23 offset:24576\n\t"
    "s_waitcnt lgkmcnt(7)\n\t"
    "v_mfma_f32_16x16x32_bf16 v[80:83], v[212:215], a[168:171], v[80:83]\n\t"
    "ds_read_b64_tr_b16 v[202:203], v23 offset:25088\n\t"
    "buffer_load_dwordx4 v14, s[40:43], s88 offen offset:2048 lds\n\t"
    "v_mfma_f32_16x16x32_bf16 v[84:87], v[212:215], a[172:175], v[84:87]\n\t"
    "ds_read_b64_tr_b16 v[204:205], v20 offset:28672\n\t"
    "v_mfma_f32_16x16x32_bf16 v[88:91], v[216:219], a[168:171], v[88:91]\n\t"
    "ds_read_b64_tr_b16 v[206:207], v20 offset:29184\n\t"
    "v_mfma_f32_16x16x32_bf16 v[92:95], v[216:219], a[172:175], v[92:95]\n\t"
    "ds_read_b64_tr_b16 v[208:209], v21 offset:28672\n\t"
    "s_waitcnt lgkmcnt(7)\n\t"
    "buffer_load_dwordx4 v15, s[40:43], s88 offen offset:3072 lds\n\t"
    "s_waitcnt vmcnt(10)\n\t"
    "v_mfma_f32_16x16x32_bf16 v[64:67], v[188:191], a[176:179], v[64:67]\n\t"
    "ds_read_b64_tr_b16 v[210:211], v21 offset:29184\n\t"
    "v_mfma_f32_16x16x32_bf16 v[96:99], v[60:63], v[128:131], v[96:99]\n\t"
    "v_mfma_f32_16x16x32_bf16 v[100:103], v[58:61], v[128:131], v[100:103]\n\t"
    "v_mfma_f32_16x16x32_bf16 v[96:99], v[60:63], v[136:139], v[96:99]\n\t"
    "v_mfma_f32_16x16x32_bf16 v[68:71], v[188:191], a[180:183], v[68:71]\n\t"
    "ds_read_b64_tr_b16 v[212:213], v22 offset:28672\n\t"
    "v_mfma_f32_16x16x32_bf16 v[100:103], v[58:61], v[136:139], v[100:103]\n\t"
    "v_mfma_f32_16x16x32_bf16 v[96:99], v[60:63], v[144:147], v[96:99]\n\t"
    "v_mfma_f32_16x16x32_bf16 v[100:103], v[58:61], v[144:147], v[100:103]\n\t"
    "v_mfma_f32_16x16x32_bf16 v[72:75], v[192:195], a[176:179], v[72:75]\n\t"
    "ds_read_b64_tr_b16 v[214:215], v22 offset:29184\n\t"
    "v_mfma_f32_16x16x32_bf16 v[76:79], v[192:195], a[180:183], v[76:79]\n\t"
    "s_waitcnt vmcnt(8)\n\t"
    "s_barrier\n\t"
    "ds_read_b64_tr_b16 v[216:217], v23 offset:28672\n\t"
    "s_waitcnt lgkmcnt(7)\n\t"
    "s_add_u32 m0, s80, 49152\n\t"
    "s_add_u32 s88, s67, 512\n\t"
    "v_mfma_f32_16x16x32_bf16 v[80:83], v[196:199], a[176:179], v[80:83]\n\t"
    "ds_read_b64_tr_b16 v[218:219], v23 offset:29184\n\t"
    "buffer_load_dwordx4 v12, s[40:43], s88 offen offset:0 lds\n\t"
    "v_mfma_f32_16x16x32_bf16 v[84:87], v[196:199], a[180:183], v[84:87]\n\t"
    "ds_read_b64_tr_b16 v[188:189], v20 offset:32768\n\t"
    "v_mfma_f32_16x16x32_bf16 v[88:91], v[200:203], a[176:179], v[88:91]\n\t"
    "ds_read_b64_tr_b16 v[190:191], v20 offset:33280\n\t"
    "v_mul_f32 v152, v96, v96\n\t"
    "v_mul_f32 v153, v97, v97\n\t"
    "v_mul_f32 v154, v98, v98\n\t"
    "v_mfma_f32_16x16x32_bf16 v[92:95], v[200:203], a[180:183], v[92:95]\n\t"
    "ds_read_b64_tr_b16 v[192:193], v21 offset:32768\n\t"
    "s_waitcnt lgkmcnt(7)\n\t"
    "buffer_load_dwordx4 v13, s[40:43], s88 offen offset:1024 lds\n\t"
    "v_mul_f32 v155, v99, v99\n\t"
    "v_mfma_f32_16x16x32_bf16 v[64:67], v[204:207], a[184:187], v[64:67]\n\t"
    "ds_read_b64_tr_b16 v[194:195], v21 offset:33280\n\t"
    "v_mul_f32 v156, v100, v100\n\t"
    "v_mul_f32 v157, v101, v101\n\t"
    "v_mul_f32 v158, v102, v102\n\t"
    "v_mfma_f32_16x16x32_bf16 v[68:71], v[204:207], a[188:191], v[68:71]\n\t"
    "ds_read_b64_tr_b16 v[196:197], v22 offset:32768\n\t"
    "v_mul_f32 v159, v103, v103\n\t"
    "v_fma_f32 v152, v152, v27, v26\n\t"
    "v_fma_f32 v153, v153, v27, v26\n\t"
    "v_mfma_f32_16x16x32_bf16 v[72:75], v[208:211], a[184:187], v[72:75]\n\t"
    "ds_read_b64_tr_b16 v[198:199], v22 offset:33280\n\t"
    "v_fma_f32 v154, v154, v27, v26\n\t"
    "v_fma_f32 v155, v155, v27, v26\n\t"
    "v_fma_f32 v156, v156, v27, v26\n\t"
    "v_mfma_f32_16x16x32_bf16 v[76:79], v[208:211], a[188:191], v[76:79]\n\t"
    "ds_read_b64_tr_b16 v[200:201], v23 offset:32768\n\t"
    "s_waitcnt lgkmcnt(7)\n\t"
    "buffer_load_dwordx4 v14, s[40:43], s88 offen offset:2048 lds\n\t"
    "v_fma_f32 v157, v157, v27, v26\n\t"
    "v_mfma_f32_16x16x32_bf16 v[80:83], v[212:215], a[184:187], v[80:83]\n\t"
    "ds_read_b64_tr_b16 v[202:203], v23 offset:33280\n\t"
    "v_fma_f32 v158, v158, v27, v26\n\t"
    "v_fma_f32 v159, v159, v27, v26\n\t"
    "v_mul_f32 v152, v152, v96\n\t"
    "v_mfma_f32_16x16x32_bf16 v[84:87], v[212:215], a[188:191], v[84:87]\n\t"
    "ds_read_b64_tr_b16 v[204:205], v20 offset:36864\n\t"
    "v_mul_f32 v153, v153, v97\n\t"
    "v_mul_f32 v154, v154, v98\n\t"
    "v_mul_f32 v155, v155, v99\n\t"
    "v_mfma_f32_16x16x32_bf16 v[88:91], v[216:219], a[184:187], v[88:91]\n\t"
    "ds_read_b64_tr_b16 v[206:207], v20 offset:37376\n\t"
    "v_mul_f32 v156, v156, v100\n\t"
    "v_mul_f32 v157, v157, v101\n\t"
    "v_mul_f32 v158, v158, v102\n\t"
    "v_mfma_f32_16x16x32_bf16 v[92:95], v[216:219], a[188:191], v[92:95]\n\t"
    "s_cmp_lt_u32 s60, 10000\n\t"
    "s_cbranch_scc0 LP448%=\n\t"
    "s_sleep 13\n\t"
    "LP448%=:\n\t"
    "ds_read_b64_tr_b16 v[208:209], v21 offset:36864\n\t"
    "s_waitcnt lgkmcnt(7)\n\t"
    "buffer_load_dwordx4 v15, s[40:43], s88 offen offset:3072 lds\n\t"
    "s_add_u32 m0, s80, 98304\n\t"
    "v_mfma_f32_16x16x32_bf16 v[64:67], v[188:191], a[192:195], v[64:67]\n\t"
    "ds_read_b64_tr_b16 v[210:211], v21 offset:37376\n\t"
    "s_add_u32 s88, s68, 512\n\t"
    "v_mul_f32 v159, v159, v103\n\t"
    "v_exp_f32 v152, v152\n\t"
    "v_mfma_f32_16x16x32_bf16 v[68:71], v[188:191], a[196:199], v[68:71]\n\t"
    "ds_read_b64_tr_b16 v[212:213], v22 offset:36864\n\t"
    "v_exp_f32 v153, v153\n\t"
    "v_exp_f32 v154, v154\n\t"
    "v_exp_f32 v155, v155\n\t"
    "v_mfma_f32_16x16x32_bf16 v[72:75], v[192:195], a[192:195], v[72:75]\n\t"
    "ds_read_b64_tr_b16 v[214:215], v22 offset:37376\n\t"
    "v_exp_f32 v156, v156\n\t"
    "v_exp_f32 v157, v157\n\t"
    "v_exp_f32 v158, v158\n\t"
    "v_mfma_f32_16x16x32_bf16 v[76:79], v[192:195], a[196:199], v[76:79]\n\t"
    "ds_read_b64_tr_b16 v[216:217], v23 offset:36864\n\t"
    "s_waitcnt lgkmcnt(7)\n\t"
    "buffer_load_dwordx4 v12, s[40:43], s88 offen offset:0 lds\n\t"
    "v_exp_f32 v159, v159\n\t"
    "v_mfma_f32_16x16x32_bf16 v[80:83], v[196:199], a[192:195], v[80:83]\n\t"
    "ds_read_b64_tr_b16 v[218:219], v23 offset:37376\n\t"
    "v_add_f32 v152, 1.0, v152\n\t"
    "v_add_f32 v153, 1.0, v153\n\t"
    "v_add_f32 v154, 1.0, v154\n\t"
    "v_mfma_f32_16x16x32_bf16 v[84:87], v[196:199], a[196:199], v[84:87]\n\t"
    "ds_read_b64_tr_b16 v[188:189], v20 offset:40960\n\t"
    "v_add_f32 v155, 1.0, v155\n\t"
    "v_add_f32 v156, 1.0, v156\n\t"
    "v_add_f32 v157, 1.0, v157\n\t"
    "v_mfma_f32_16x16x32_bf16 v[88:91], v[200:203], a[192:195], v[88:91]\n\t"
    "ds_read_b64_tr_b16 v[190:191], v20 offset:41472\n\t"
    "v_add_f32 v158, 1.0, v158\n\t"
    "v_add_f32 v159, 1.0, v159\n\t"
    "v_rcp_f32 v152, v152\n\t"
    "v_mfma_f32_16x16x32_bf16 v[92:95], v[200:203], a[196:199], v[92:95]\n\t"
    "ds_read_b64_tr_b16 v[192:193], v21 offset:40960\n\t"
    "s_waitcnt lgkmcnt(7)\n\t"
    "buffer_load_dwordx4 v13, s[40:43], s88 offen offset:1024 lds\n\t"
    "v_rcp_f32 v153, v153\n\t"
    "v_mfma_f32_16x16x32_bf16 v[64:67], v[204:207], a[200:203], v[64:67]\n\t"
    "ds_read_b64_tr_b16 v[194:195], v21 offset:41472\n\t"
    "v_rcp_f32 v154, v154\n\t"
    "v_rcp_f32 v155, v155\n\t"
    "v_rcp_f32 v156, v156\n\t"
    "v_mfma_f32_16x16x32_bf16 v[68:71], v[204:207], a[204:207], v[68:71]\n\t"
    "ds_read_b64_tr_b16 v[196:197], v22 offset:40960\n\t"
    "v_rcp_f32 v157, v157\n\t"
    "v_rcp_f32 v158, v158\n\t"
    "v_rcp_f32 v159, v159\n\t"
    "v_mfma_f32_16x16x32_bf16 v[72:75], v[208:211], a[200:203], v[72:75]\n\t"
    "ds_read_b64_tr_b16 v[198:199], v22 offset:41472\n\t"
    "v_mul_f32 v152, v152, v96\n\t"
    "v_mul_f32 v153, v153, v97\n\t"
    "v_mul_f32 v154, v154, v98\n\t"
    "v_mfma_f32_16x16x32_bf16 v[76:79], v[208:211], a[204:207], v[76:79]\n\t"
    "ds_read_b64_tr_b16 v[200:201], v23 offset:40960\n\t"
    "s_waitcnt lgkmcnt(7)\n\t"
    "buffer_load_dwordx4 v14, s[40:43], s88 offen offset:2048 lds\n\t"
    "v_mul_f32 v155, v155, v99\n\t"
    "v_mfma_f32_16x16x32_bf16 v[80:83], v[212:215], a[200:203], v[80:83]\n\t"
    "ds_read_b64_tr_b16 v[202:203], v23 offset:41472\n\t"
    "v_mul_f32 v156, v156, v100\n\t"
    "v_mul_f32 v157, v157, v101\n\t"
    "v_mul_f32 v158, v158, v102\n\t"
    "v_mfma_f32_16x16x32_bf16 v[84:87], v[212:215], a[204:207], v[84:87]\n\t"
    "ds_read_b64_tr_b16 v[204:205], v20 offset:45056\n\t"
    "v_mul_f32 v159, v159, v103\n\t"
    "v_cvt_pk_bf16_f32 v160, v152, v153\n\t"
    "v_cvt_pk_bf16_f32 v161, v154, v155\n\t"
    "v_mfma_f32_16x16x32_bf16 v[88:91], v[216:219], a[200:203], v[88:91]\n\t"
    "ds_read_b64_tr_b16 v[206:207], v20 offset:45568\n\t"
    "v_cvt_pk_bf16_f32 v162, v156, v157\n\t"
    "v_cvt_pk_bf16_f32 v163, v158, v159\n\t"
    "s_nop 1\n\t"
    "v_permlane16_swap_b32 v160, v162\n\t"
    "v_permlane16_swap_b32 v161, v163\n\t"
    "v_mfma_f32_16x16x32_bf16 v[92:95], v[216:219], a[204:207], v[92:95]\n\t"
    "ds_read_b64_tr_b16 v[208:209], v21 offset:45056\n\t"
    "s_waitcnt lgkmcnt(7)\n\t"
    "buffer_load_dwordx4 v15, s[40:43], s88 offen offset:3072 lds\n\t"
    "global_store_dwordx4 v25, v[160:163], s[52:53] offset:256 sc0 sc1\n\t"
    "v_mfma_f32_16x16x32_bf16 v[64:67], v[188:191], a[208:211], v[64:67]\n\t"
    "ds_read_b64_tr_b16 v[210:211], v21 offset:45568\n\t"
    "v_add_u32 v24, 6291456, v24\n\t"
    "v_mfma_f32_16x16x32_bf16 v[68:71], v[188:191], a[212:215], v[68:71]\n\t"
    "ds_read_b64_tr_b16 v[212:213], v22 offset:45056\n\t"
    "v_mfma_f32_16x16x32_bf16 v[72:75], v[192:195], a[208:211], v[72:75]\n\t"
    "ds_read_b64_tr_b16 v[214:215], v22 offset:45568\n\t"
    "v_mfma_f32_16x16x32_bf16 v[76:79], v[192:195], a[212:215], v[76:79]\n\t"
    "s_waitcnt vmcnt(13)\n\t"
    "s_barrier\n\t"
    "ds_read_b64_tr_b16 v[216:217], v23 offset:45056\n\t"
    "s_waitcnt lgkmcnt(7)\n\t"
    "v_mfma_f32_16x16x32_bf16 v[80:83], v[196:199], a[208:211], v[80:83]\n\t"
    "ds_read_b64_tr_b16 v[218:219], v23 offset:45568\n\t"
    "v_mfma_f32_16x16x32_bf16 v[84:87], v[196:199], a[212:215], v[84:87]\n\t"
    "ds_read_b64_tr_b16 v[188:189], v40 offset:0\n\t"
    "v_mfma_f32_16x16x32_bf16 v[88:91], v[200:203], a[208:211], v[88:91]\n\t"
    "ds_read_b64_tr_b16 v[190:191], v40 offset:512\n\t"
    "v_mfma_f32_16x16x32_bf16 v[92:95], v[200:203], a[212:215], v[92:95]\n\t"
    "ds_read_b64_tr_b16 v[192:193], v41 offset:0\n\t"
    "s_waitcnt lgkmcnt(7)\n\t"
    "v_mfma_f32_16x16x32_bf16 v[64:67], v[204:207], a[216:219], v[64:67]\n\t"
    "ds_read_b64_tr_b16 v[194:195], v41 offset:512\n\t"
    "v_mfma_f32_16x16x32_bf16 v[68:71], v[204:207], a[220:223], v[68:71]\n\t"
    "ds_read_b64_tr_b16 v[196:197], v42 offset:0\n\t"
    "v_mfma_f32_16x16x32_bf16 v[72:75], v[208:211], a[216:219], v[72:75]\n\t"
    "ds_read_b64_tr_b16 v[198:199], v42 offset:512\n\t"
    "v_mfma_f32_16x16x32_bf16 v[76:79], v[208:211], a[220:223], v[76:79]\n\t"
    "ds_read_b64_tr_b16 v[200:201], v43 offset:0\n\t"
    "s_waitcnt lgkmcnt(7)\n\t"
    "v_mfma_f32_16x16x32_bf16 v[80:83], v[212:215], a[216:219], v[80:83]\n\t"
    "ds_read_b64_tr_b16 v[202:203], v43 offset:512\n\t"
    "v_mfma_f32_16x16x32_bf16 v[84:87], v[212:215], a[220:223], v[84:87]\n\t"
    "ds_read_b64_tr_b16 v[204:205], v40 offset:4096\n\t"
    "v_mfma_f32_16x16x32_bf16 v[88:91], v[216:219], a[216:219], v[88:91]\n\t"
    "ds_read_b64_tr_b16 v[206:207], v40 offset:4608\n\t"
    "v_mfma_f32_16x16x32_bf16 v[92:95], v[216:219], a[220:223], v[92:95]\n\t"
    "s_cmp_lt_u32 s60, 10000\n\t"
    "s_cbranch_scc0 LP480%=\n\t"
    "s_sleep 13\n\t"
    "LP480%=:\n\t"
    "ds_read_b64_tr_b16 v[208:209], v41 offset:4096\n\t"
    "s_waitcnt lgkmcnt(7)\n\t"
    "v_mfma_f32_16x16x32_bf16 v[64:67], v[188:191], a[224:227], v[64:67]\n\t"
    "ds_read_b64_tr_b16 v[210:211], v41 offset:4608\n\t"
    "v_mfma_f32_16x16x32_bf16 v[68:71], v[188:191], a[228:231], v[68:71]\n\t"
    "ds_read_b64_tr_b16 v[212:213], v42 offset:4096\n\t"
    "v_mfma_f32_16x16x32_bf16 v[72:75], v[192:195], a[224:227], v[72:75]\n\t"
    "ds_read_b64_tr_b16 v[214:215], v42 offset:4608\n\t"
    "v_mfma_f32_16x16x32_bf16 v[76:79], v[192:195], a[228:231], v[76:79]\n\t"
    "ds_read_b64_tr_b16 v[216:217], v43 offset:4096\n\t"
    "s_waitcnt lgkmcnt(7)\n\t"
    "v_mfma_f32_16x16x32_bf16 v[80:83], v[196:199], a[224:227], v[80:83]\n\t"
    "ds_read_b64_tr_b16 v[218:219], v43 offset:4608\n\t"
    "v_mfma_f32_16x16x32_bf16 v[84:87], v[196:199], a[228:231], v[84:87]\n\t"
    "ds_read_b64_tr_b16 v[188:189], v40 offset:8192\n\t"
    "v_mfma_f32_16x16x32_bf16 v[88:91], v[200:203], a[224:227], v[88:91]\n\t"
    "ds_read_b64_tr_b16 v[190:191], v40 offset:8704\n\t"
    "v_mfma_f32_16x16x32_bf16 v[92:95], v[200:203], a[228:231], v[92:95]\n\t"
    "ds_read_b64_tr_b16 v[192:193], v41 offset:8192\n\t"
    "s_waitcnt lgkmcnt(7)\n\t"
    "v_mfma_f32_16x16x32_bf16 v[64:67], v[204:207], a[232:235], v[64:67]\n\t"
    "ds_read_b64_tr_b16 v[194:195], v41 offset:8704\n\t"
    "v_mfma_f32_16x16x32_bf16 v[68:71], v[204:207], a[236:239], v[68:71]\n\t"
    "ds_read_b64_tr_b16 v[196:197], v42 offset:8192\n\t"
    "v_mfma_f32_16x16x32_bf16 v[72:75], v[208:211], a[232:235], v[72:75]\n\t"
    "ds_read_b64_tr_b16 v[198:199], v42 offset:8704\n\t"
    "v_mfma_f32_16x16x32_bf16 v[76:79], v[208:211], a[236:239], v[76:79]\n\t"
    "ds_read_b64_tr_b16 v[200:201], v43 offset:8192\n\t"
    "s_waitcnt lgkmcnt(7)\n\t"
    "v_mfma_f32_16x16x32_bf16 v[80:83], v[212:215], a[232:235], v[80:83]\n\t"
    "ds_read_b64_tr_b16 v[202:203], v43 offset:8704\n\t"
    "v_mfma_f32_16x16x32_bf16 v[84:87], v[212:215], a[236:239], v[84:87]\n\t"
    "ds_read_b64_tr_b16 v[204:205], v40 offset:12288\n\t"
    "v_mfma_f32_16x16x32_bf16 v[88:91], v[216:219], a[232:235], v[88:91]\n\t"
    "ds_read_b64_tr_b16 v[206:207], v40 offset:12800\n\t"
    "v_mfma_f32_16x16x32_bf16 v[92:95], v[216:219], a[236:239], v[92:95]\n\t"
    "ds_read_b64_tr_b16 v[208:209], v41 offset:12288\n\t"
    "s_waitcnt lgkmcnt(7)\n\t"
    "v_mfma_f32_16x16x32_bf16 v[64:67], v[188:191], a[240:243], v[64:67]\n\t"
    "ds_read_b64_tr_b16 v[210:211], v41 offset:12800\n\t"
    "v_mfma_f32_16x16x32_bf16 v[68:71], v[188:191], a[244:247], v[68:71]\n\t"
    "ds_read_b64_tr_b16 v[212:213], v42 offset:12288\n\t"
    "v_mfma_f32_16x16x32_bf16 v[72:75], v[192:195], a[240:243], v[72:75]\n\t"
    "ds_read_b64_tr_b16 v[214:215], v42 offset:12800\n\t"
    "v_mfma_f32_16x16x32_bf16 v[76:79], v[192:195], a[244:247], v[76:79]\n\t"
    "s_waitcnt vmcnt(34)\n\t"
    "s_barrier\n\t"
    "ds_read_b64_tr_b16 v[216:217], v43 offset:12288\n\t"
    "s_waitcnt lgkmcnt(7)\n\t"
    "v_mfma_f32_16x16x32_bf16 v[80:83], v[196:199], a[240:243], v[80:83]\n\t"
    "ds_read_b64_tr_b16 v[218:219], v43 offset:12800\n\t"
    "s_add_u32 m0, s80, 114688\n\t"
    "v_mfma_f32_16x16x32_bf16 v[84:87], v[196:199], a[244:247], v[84:87]\n\t"
    "ds_read_b64_tr_b16 v[188:189], v44 offset:16384\n\t"
    "s_add_u32 s88, s64, 768\n\t"
    "v_mfma_f32_16x16x32_bf16 v[88:91], v[200:203], a[240:243], v[88:91]\n\t"
    "ds_read_b64_tr_b16 v[190:191], v44 offset:16896\n\t"
    "v_mfma_f32_16x16x32_bf16 v[92:95], v[200:203], a[244:247], v[92:95]\n\t"
    "ds_read_b64_tr_b16 v[192:193], v45 offset:16384\n\t"
    "s_waitcnt lgkmcnt(7)\n\t"
    "v_mfma_f32_16x16x32_bf16 v[64:67], v[204:207], a[248:251], v[64:67]\n\t"
    "ds_read_b64_tr_b16 v[194:195], v45 offset:16896\n\t"
    "v_mfma_f32_16x16x32_bf16 v[68:71], v[204:207], a[252:255], v[68:71]\n\t"
    "ds_read_b64_tr_b16 v[196:197], v46 offset:16384\n\t"
    "v_mfma_f32_16x16x32_bf16 v[72:75], v[208:211], a[248:251], v[72:75]\n\t"
    "ds_read_b64_tr_b16 v[198:199], v46 offset:16896\n\t"
    "v_mfma_f32_16x16x32_bf16 v[76:79], v[208:211], a[252:255], v[76:79]\n\t"
    "ds_read_b64_tr_b16 v[200:201], v47 offset:16384\n\t"
    "s_waitcnt lgkmcnt(7)\n\t"
    "v_mfma_f32_16x16x32_bf16 v[80:83], v[212:215], a[248:251], v[80:83]\n\t"
    "ds_read_b64_tr_b16 v[202:203], v47 offset:16896\n\t"
    "v_mfma_f32_16x16x32_bf16 v[84:87], v[212:215], a[252:255], v[84:87]\n\t"
    "ds_read_b64_tr_b16 v[204:205], v44 offset:20480\n\t"
    "v_mfma_f32_16x16x32_bf16 v[88:91], v[216:219], a[248:251], v[88:91]\n\t"
    "ds_read_b64_tr_b16 v[206:207], v44 offset:20992\n\t"
    "v_mfma_f32_16x16x32_bf16 v[92:95], v[216:219], a[252:255], v[92:95]\n\t"
    "s_cmp_lt_u32 s60, 10000\n\t"
    "s_cbranch_scc0 LP512%=\n\t"
    "s_sleep 13\n\t"
    "LP512%=:\n\t"
    "ds_read_b64_tr_b16 v[208:209], v45 offset:20480\n\t"
    "s_waitcnt lgkmcnt(7)\n\t"
    "v_mov_b32 v96, v64\n\t"
    "v_mov_b32 v97, v65\n\t"
    "v_mov_b32 v98, v66\n\t"
    "v_mov_b32 v99, v67\n\t"
    "v_mfma_f32_16x16x32_bf16 v[64:67], v[188:191], a[0:3], 0\n\t"
    "ds_read_b64_tr_b16 v[210:211], v45 offset:20992\n\t"
    "v_cvt_pk_bf16_f32 v112, v68, v69\n\t"
    "v_cvt_pk_bf16_f32 v113, v70, v71\n\t"
    "v_mfma_f32_16x16x32_bf16 v[68:71], v[188:191], a[4:7], 0\n\t"
    "ds_read_b64_tr_b16 v[212:213], v46 offset:20480\n\t"
    "v_mov_b32 v100, v72\n\t"
    "v_mov_b32 v101, v73\n\t"
    "v_mov_b32 v102, v74\n\t"
    "v_mov_b32 v103, v75\n\t"
    "v_mfma_f32_16x16x32_bf16 v[72:75], v[192:195], a[0:3], 0\n\t"
    "ds_read_b64_tr_b16 v[214:215], v46 offset:20992\n\t"
    "v_cvt_pk_bf16_f32 v114, v76, v77\n\t"
    "v_cvt_pk_bf16_f32 v115, v78, v79\n\t"
    "v_mfma_f32_16x16x32_bf16 v[76:79], v[192:195], a[4:7], 0\n\t"
    "global_store_dwordx4 v24, v[112:115], s[84:85] sc0 sc1\n\t"
    "ds_read_b64_tr_b16 v[216:217], v47 offset:20480\n\t"
    "s_waitcnt lgkmcnt(7)\n\t"
    "v_cvt_pk_bf16_f32 v104, v80, v81\n\t"
    "v_cvt_pk_bf16_f32 v105, v82, v83\n\t"
    "v_mfma_f32_16x16x32_bf16 v[80:83], v[196:199], a[0:3], 0\n\t"
    "ds_read_b64_tr_b16 v[218:219], v47 offset:20992\n\t"
    "v_cvt_pk_bf16_f32 v120, v84, v85\n\t"
    "v_cvt_pk_bf16_f32 v121, v86, v87\n\t"
    "v_mfma_f32_16x16x32_bf16 v[84:87], v[196:199], a[4:7], 0\n\t"
    "ds_read_b64_tr_b16 v[188:189], v44 offset:24576\n\t"
    "v_cvt_pk_bf16_f32 v106, v88, v89\n\t"
    "v_cvt_pk_bf16_f32 v107, v90, v91\n\t"
    "v_mfma_f32_16x16x32_bf16 v[88:91], v[200:203], a[0:3], 0\n\t"
    "global_store_dwordx4 v24, v[104:107], s[82:83] sc0 sc1\n\t"
    "ds_read_b64_tr_b16 v[190:191], v44 offset:25088\n\t"
    "v_cvt_pk_bf16_f32 v122, v92, v93\n\t"
    "v_cvt_pk_bf16_f32 v123, v94, v95\n\t"
    "v_mfma_f32_16x16x32_bf16 v[92:95], v[200:203], a[4:7], 0\n\t"
    "ds_read_b64_tr_b16 v[192:193], v45 offset:24576\n\t"
    "s_waitcnt lgkmcnt(7)\n\t"
    "v_mfma_f32_16x16x32_bf16 v[64:67], v[204:207], a[8:11], v[64:67]\n\t"
    "ds_read_b64_tr_b16 v[194:195], v45 offset:25088\n\t"
    "v_mfma_f32_16x16x32_bf16 v[68:71], v[204:207], a[12:15], v[68:71]\n\t"
    "global_store_dwordx4 v24, v[120:123], s[86:87] sc0 sc1\n\t"
    "ds_read_b64_tr_b16 v[196:197], v46 offset:24576\n\t"
    "v_mfma_f32_16x16x32_bf16 v[72:75], v[208:211], a[8:11], v[72:75]\n\t"
    "ds_read_b64_tr_b16 v[198:199], v46 offset:25088\n\t"
    "v_mfma_f32_16x16x32_bf16 v[76:79], v[208:211], a[12:15], v[76:79]\n\t"
    "ds_read_b64_tr_b16 v[200:201], v47 offset:24576\n\t"
    "s_waitcnt lgkmcnt(7)\n\t"
    "v_mfma_f32_16x16x32_bf16 v[80:83], v[212:215], a[8:11], v[80:83]\n\t"
    "ds_read_b64_tr_b16 v[202:203], v47 offset:25088\n\t"
    "v_mfma_f32_16x16x32_bf16 v[84:87], v[212:215], a[12:15], v[84:87]\n\t"
    "ds_read_b64_tr_b16 v[204:205], v44 offset:28672\n\t"
    "buffer_load_dwordx4 v12, s[40:43], s88 offen offset:0 lds\n\t"
    "v_mfma_f32_16x16x32_bf16 v[88:91], v[216:219], a[8:11], v[88:91]\n\t"
    "ds_read_b64_tr_b16 v[206:207], v44 offset:29184\n\t"
    "v_mfma_f32_16x16x32_bf16 v[92:95], v[216:219], a[12:15], v[92:95]\n\t"
    "ds_read_b64_tr_b16 v[208:209], v45 offset:28672\n\t"
    "s_waitcnt lgkmcnt(7)\n\t"
    "v_mfma_f32_16x16x32_bf16 v[64:67], v[188:191], a[16:19], v[64:67]\n\t"
    "ds_read_b64_tr_b16 v[210:211], v45 offset:29184\n\t"
    "buffer_load_dwordx4 v13, s[40:43], s88 offen offset:1024 lds\n\t"
    "v_mfma_f32_16x16x32_bf16 v[68:71], v[188:191], a[20:23], v[68:71]\n\t"
    "ds_read_b64_tr_b16 v[212:213], v46 offset:28672\n\t"
    "buffer_load_dwordx4 v14, s[40:43], s88 offen offset:2048 lds\n\t"
    "v_mfma_f32_16x16x32_bf16 v[72:75], v[192:195], a[16:19], v[72:75]\n\t"
    "ds_read_b64_tr_b16 v[214:215], v46 offset:29184\n\t"
    "buffer_load_dwordx4 v15, s[40:43], s88 offen offset:3072 lds\n\t"
    "v_mfma_f32_16x16x32_bf16 v[76:79], v[192:195], a[20:23], v[76:79]\n\t"
    "s_waitcnt vmcnt(36)\n\t"
    "s_barrier\n\t"
    "ds_read_b64_tr_b16 v[216:217], v47 offset:28672\n\t"
    "s_waitcnt lgkmcnt(7)\n\t"
    "v_mfma_f32_16x16x32_bf16 v[80:83], v[196:199], a[16:19], v[80:83]\n\t"
    "ds_read_b64_tr_b16 v[218:219], v47 offset:29184\n\t"
    "v_mfma_f32_16x16x32_bf16 v[84:87], v[196:199], a[20:23], v[84:87]\n\t"
    "ds_read_b64_tr_b16 v[188:189], v20 offset:0\n\t"
    "v_mfma_f32_16x16x32_bf16 v[88:91], v[200:203], a[16:19], v[88:91]\n\t"
    "ds_read_b64_tr_b16 v[190:191], v20 offset:512\n\t"
    "v_mfma_f32_16x16x32_bf16 v[92:95], v[200:203], a[20:23], v[92:95]\n\t"
    "ds_read_b64_tr_b16 v[192:193], v21 offset:0\n\t"
    "s_waitcnt lgkmcnt(7)\n\t"
    "v_mfma_f32_16x16x32_bf16 v[64:67], v[204:207], a[24:27], v[64:67]\n\t"
    "ds_read_b64_tr_b16 v[194:195], v21 offset:512\n\t"
    "v_mfma_f32_16x16x32_bf16 v[68:71], v[204:207], a[28:31], v[68:71]\n\t"
    "ds_read_b64_tr_b16 v[196:197], v22 offset:0\n\t"
    "v_mfma_f32_16x16x32_bf16 v[72:75], v[208:211], a[24:27], v[72:75]\n\t"
    "ds_read_b64_tr_b16 v[198:199], v22 offset:512\n\t"
    "v_mfma_f32_16x16x32_bf16 v[76:79], v[208:211], a[28:31], v[76:79]\n\t"
    "ds_read_b64_tr_b16 v[200:201], v23 offset:0\n\t"
    "s_waitcnt lgkmcnt(7)\n\t"
    "v_mfma_f32_16x16x32_bf16 v[80:83], v[212:215], a[24:27], v[80:83]\n\t"
    "ds_read_b64_tr_b16 v[202:203], v23 offset:512\n\t"
    "v_mfma_f32_16x16x32_bf16 v[84:87], v[212:215], a[28:31], v[84:87]\n\t"
    "ds_read_b64_tr_b16 v[204:205], v20 offset:4096\n\t"
    "v_mfma_f32_16x16x32_bf16 v[88:91], v[216:219], a[24:27], v[88:91]\n\t"
    "ds_read_b64_tr_b16 v[206:207], v20 offset:4608\n\t"
    "v_mfma_f32_16x16x32_bf16 v[92:95], v[216:219], a[28:31], v[92:95]\n\t"
    "s_cmp_lt_u32 s60, 10000\n\t"
    "s_cbranch_scc0 LP544%=\n\t"
    "s_sleep 13\n\t"
    "LP544%=:\n\t"
    "ds_read_b64_tr_b16 v[208:209], v21 offset:4096\n\t"
    "s_waitcnt lgkmcnt(7)\n\t"
    "v_mfma_f32_16x16x32_bf16 v[64:67], v[188:191], a[32:35], v[64:67]\n\t"
    "ds_read_b64_tr_b16 v[210:211], v21 offset:4608\n\t"
    "v_mfma_f32_16x16x32_bf16 v[68:71], v[188:191], a[36:39], v[68:71]\n\t"
    "ds_read_b64_tr_b16 v[212:213], v22 offset:4096\n\t"
    "v_mfma_f32_16x16x32_bf16 v[72:75], v[192:195], a[32:35], v[72:75]\n\t"
    "ds_read_b64_tr_b16 v[214:215], v22 offset:4608\n\t"
    "v_mfma_f32_16x16x32_bf16 v[76:79], v[192:195], a[36:39], v[76:79]\n\t"
    "ds_read_b64_tr_b16 v[216:217], v23 offset:4096\n\t"
    "s_waitcnt lgkmcnt(7)\n\t"
    "v_mfma_f32_16x16x32_bf16 v[80:83], v[196:199], a[32:35], v[80:83]\n\t"
    "ds_read_b64_tr_b16 v[218:219], v23 offset:4608\n\t"
    "v_mfma_f32_16x16x32_bf16 v[84:87], v[196:199], a[36:39], v[84:87]\n\t"
    "ds_read_b64_tr_b16 v[188:189], v20 offset:8192\n\t"
    "v_mfma_f32_16x16x32_bf16 v[88:91], v[200:203], a[32:35], v[88:91]\n\t"
    "ds_read_b64_tr_b16 v[190:191], v20 offset:8704\n\t"
    "v_mfma_f32_16x16x32_bf16 v[92:95], v[200:203], a[36:39], v[92:95]\n\t"
    "ds_read_b64_tr_b16 v[192:193], v21 offset:8192\n\t"
    "s_waitcnt lgkmcnt(7)\n\t"
    "v_mfma_f32_16x16x32_bf16 v[64:67], v[204:207], a[40:43], v[64:67]\n\t"
    "ds_read_b64_tr_b16 v[194:195], v21 offset:8704\n\t"
    "v_mfma_f32_16x16x32_bf16 v[68:71], v[204:207], a[44:47], v[68:71]\n\t"
    "ds_read_b64_tr_b16 v[196:197], v22 offset:8192\n\t"
    "v_mfma_f32_16x16x32_bf16 v[72:75], v[208:211], a[40:43], v[72:75]\n\t"
    "ds_read_b64_tr_b16 v[198:199], v22 offset:8704\n\t"
    "v_mfma_f32_16x16x32_bf16 v[76:79], v[208:211], a[44:47], v[76:79]\n\t"
    "ds_read_b64_tr_b16 v[200:201], v23 offset:8192\n\t"
    "s_waitcnt lgkmcnt(7)\n\t"
    "v_mfma_f32_16x16x32_bf16 v[80:83], v[212:215], a[40:43], v[80:83]\n\t"
    "ds_read_b64_tr_b16 v[202:203], v23 offset:8704\n\t"
    "v_mfma_f32_16x16x32_bf16 v[84:87], v[212:215], a[44:47], v[84:87]\n\t"
    "ds_read_b64_tr_b16 v[204:205], v20 offset:12288\n\t"
    "v_mfma_f32_16x16x32_bf16 v[88:91], v[216:219], a[40:43], v[88:91]\n\t"
    "ds_read_b64_tr_b16 v[206:207], v20 offset:12800\n\t"
    "v_mfma_f32_16x16x32_bf16 v[92:95], v[216:219], a[44:47], v[92:95]\n\t"
    "ds_read_b64_tr_b16 v[208:209], v21 offset:12288\n\t"
    "s_waitcnt lgkmcnt(7)\n\t"
    "v_mfma_f32_16x16x32_bf16 v[64:67], v[188:191], a[48:51], v[64:67]\n\t"
    "ds_read_b64_tr_b16 v[210:211], v21 offset:12800\n\t"
    "v_mfma_f32_16x16x32_bf16 v[68:71], v[188:191], a[52:55], v[68:71]\n\t"
    "ds_read_b64_tr_b16 v[212:213], v22 offset:12288\n\t"
    "v_mfma_f32_16x16x32_bf16 v[72:75], v[192:195], a[48:51], v[72:75]\n\t"
    "ds_read_b64_tr_b16 v[214:215], v22 offset:12800\n\t"
    "v_mfma_f32_16x16x32_bf16 v[76:79], v[192:195], a[52:55], v[76:79]\n\t"
    "s_waitcnt vmcnt(16)\n\t"
    "s_barrier\n\t"
    "ds_read_b64_tr_b16 v[216:217], v23 offset:12288\n\t"
    "s_waitcnt lgkmcnt(7)\n\t"
    "s_add_u32 m0, s80, 131072\n\t"
    "s_add_u32 s88, s65, 768\n\t"
    "v_mfma_f32_16x16x32_bf16 v[80:83], v[196:199], a[48:51], v[80:83]\n\t"
    "ds_read_b64_tr_b16 v[218:219], v23 offset:12800\n\t"
    "buffer_load_dwordx4 v12, s[40:43], s88 offen offset:0 lds\n\t"
    "s_waitcnt vmcnt(5)\n\t"
    "s_barrier\n\t"
    "buffer_store_dword v50, v28, s[56:59], 0 offen offset:128 sc0 sc1\n\t"
    "v_mfma_f32_16x16x32_bf16 v[84:87], v[196:199], a[52:55], v[84:87]\n\t"
    "ds_read_b64_tr_b16 v[188:189], v40 offset:16384\n\t"
    "v_mfma_f32_16x16x32_bf16 v[88:91], v[200:203], a[48:51], v[88:91]\n\t"
    "ds_read_b64_tr_b16 v[190:191], v40 offset:16896\n\t"
    "buffer_load_dwordx4 v13, s[40:43], s88 offen offset:1024 lds\n\t"
    "v_mfma_f32_16x16x32_bf16 v[92:95], v[200:203], a[52:55], v[92:95]\n\t"
    "ds_read_b64_tr_b16 v[192:193], v41 offset:16384\n\t"
    "s_waitcnt lgkmcnt(7)\n\t"
    "v_mfma_f32_16x16x32_bf16 v[64:67], v[204:207], a[56:59], v[64:67]\n\t"
    "ds_read_b64_tr_b16 v[194:195], v41 offset:16896\n\t"
    "v_mfma_f32_16x16x32_bf16 v[68:71], v[204:207], a[60:63], v[68:71]\n\t"
    "ds_read_b64_tr_b16 v[196:197], v42 offset:16384\n\t"
    "buffer_load_dwordx4 v14, s[40:43], s88 offen offset:2048 lds\n\t"
    "v_mfma_f32_16x16x32_bf16 v[72:75], v[208:211], a[56:59], v[72:75]\n\t"
    "ds_read_b64_tr_b16 v[198:199], v42 offset:16896\n\t"
    "v_mfma_f32_16x16x32_bf16 v[76:79], v[208:211], a[60:63], v[76:79]\n\t"
    "ds_read_b64_tr_b16 v[200:201], v43 offset:16384\n\t"
    "s_waitcnt lgkmcnt(7)\n\t"
    "v_mfma_f32_16x16x32_bf16 v[80:83], v[212:215], a[56:59], v[80:83]\n\t"
    "ds_read_b64_tr_b16 v[202:203], v43 offset:16896\n\t"
    "buffer_load_dwordx4 v15, s[40:43], s88 offen offset:3072 lds\n\t"
    "v_mfma_f32_16x16x32_bf16 v[84:87], v[212:215], a[60:63], v[84:87]\n\t"
    "ds_read_b64_tr_b16 v[204:205], v40 offset:20480\n\t"
    "v_mfma_f32_16x16x32_bf16 v[88:91], v[216:219], a[56:59], v[88:91]\n\t"
    "ds_read_b64_tr_b16 v[206:207], v40 offset:20992\n\t"
    "v_mfma_f32_16x16x32_bf16 v[92:95], v[216:219], a[60:63], v[92:95]\n\t"
    "s_cmp_lt_u32 s60, 10000\n\t"
    "s_cbranch_scc0 LP576%=\n\t"
    "s_sleep 13\n\t"
    "LP576%=:\n\t"
    "ds_read_b64_tr_b16 v[208:209], v41 offset:20480\n\t"
    "s_waitcnt lgkmcnt(7)\n\t"
    "v_mfma_f32_16x16x32_bf16 v[64:67], v[188:191], a[64:67], v[64:67]\n\t"
    "ds_read_b64_tr_b16 v[210:211], v41 offset:20992\n\t"
    "v_mfma_f32_16x16x32_bf16 v[68:71], v[188:191], a[68:71], v[68:71]\n\t"
    "ds_read_b64_tr_b16 v[212:213], v42 offset:20480\n\t"
    "v_mfma_f32_16x16x32_bf16 v[72:75], v[192:195], a[64:67], v[72:75]\n\t"
    "ds_read_b64_tr_b16 v[214:215], v42 offset:20992\n\t"
    "v_mfma_f32_16x16x32_bf16 v[76:79], v[192:195], a[68:71], v[76:79]\n\t"
    "ds_read_b64_tr_b16 v[216:217], v43 offset:20480\n\t"
    "s_waitcnt lgkmcnt(7)\n\t"
    "v_mfma_f32_16x16x32_bf16 v[80:83], v[196:199], a[64:67], v[80:83]\n\t"
    "ds_read_b64_tr_b16 v[218:219], v43 offset:20992\n\t"
    "v_mfma_f32_16x16x32_bf16 v[84:87], v[196:199], a[68:71], v[84:87]\n\t"
    "ds_read_b64_tr_b16 v[188:189], v40 offset:24576\n\t"
    "v_mfma_f32_16x16x32_bf16 v[88:91], v[200:203], a[64:67], v[88:91]\n\t"
    "ds_read_b64_tr_b16 v[190:191], v40 offset:25088\n\t"
    "v_mfma_f32_16x16x32_bf16 v[92:95], v[200:203], a[68:71], v[92:95]\n\t"
    "ds_read_b64_tr_b16 v[192:193], v41 offset:24576\n\t"
    "s_waitcnt lgkmcnt(7)\n\t"
    "v_mfma_f32_16x16x32_bf16 v[64:67], v[204:207], a[72:75], v[64:67]\n\t"
    "ds_read_b64_tr_b16 v[194:195], v41 offset:25088\n\t"
    "v_mfma_f32_16x16x32_bf16 v[68:71], v[204:207], a[76:79], v[68:71]\n\t"
    "ds_read_b64_tr_b16 v[196:197], v42 offset:24576\n\t"
    "v_mfma_f32_16x16x32_bf16 v[72:75], v[208:211], a[72:75], v[72:75]\n\t"
    "ds_read_b64_tr_b16 v[198:199], v42 offset:25088\n\t"
    "v_mfma_f32_16x16x32_bf16 v[76:79], v[208:211], a[76:79], v[76:79]\n\t"
    "ds_read_b64_tr_b16 v[200:201], v43 offset:24576\n\t"
    "s_waitcnt lgkmcnt(7)\n\t"
    "v_mfma_f32_16x16x32_bf16 v[80:83], v[212:215], a[72:75], v[80:83]\n\t"
    "ds_read_b64_tr_b16 v[202:203], v43 offset:25088\n\t"
    "v_mfma_f32_16x16x32_bf16 v[84:87], v[212:215], a[76:79], v[84:87]\n\t"
    "ds_read_b64_tr_b16 v[204:205], v40 offset:28672\n\t"
    "v_mfma_f32_16x16x32_bf16 v[88:91], v[216:219], a[72:75], v[88:91]\n\t"
    "ds_read_b64_tr_b16 v[206:207], v40 offset:29184\n\t"
    "v_mfma_f32_16x16x32_bf16 v[92:95], v[216:219], a[76:79], v[92:95]\n\t"
    "ds_read_b64_tr_b16 v[208:209], v41 offset:28672\n\t"
    "s_waitcnt lgkmcnt(7)\n\t"
    "v_mfma_f32_16x16x32_bf16 v[64:67], v[188:191], a[80:83], v[64:67]\n\t"
    "ds_read_b64_tr_b16 v[210:211], v41 offset:29184\n\t"
    "v_mfma_f32_16x16x32_bf16 v[68:71], v[188:191], a[84:87], v[68:71]\n\t"
    "ds_read_b64_tr_b16 v[212:213], v42 offset:28672\n\t"
    "v_mfma_f32_16x16x32_bf16 v[72:75], v[192:195], a[80:83], v[72:75]\n\t"
    "ds_read_b64_tr_b16 v[214:215], v42 offset:29184\n\t"
    "v_mfma_f32_16x16x32_bf16 v[76:79], v[192:195], a[84:87], v[76:79]\n\t"
    "s_waitcnt vmcnt(17)\n\t"
    "s_barrier\n\t"
    "ds_read_b64_tr_b16 v[216:217], v43 offset:28672\n\t"
    "s_waitcnt lgkmcnt(7)\n\t"
    "v_mfma_f32_16x16x32_bf16 v[80:83], v[196:199], a[80:83], v[80:83]\n\t"
    "ds_read_b64_tr_b16 v[218:219], v43 offset:29184\n\t"
    "s_add_u32 m0, s80, 16384\n\t"
    "v_mfma_f32_16x16x32_bf16 v[84:87], v[196:199], a[84:87], v[84:87]\n\t"
    "ds_read_b64_tr_b16 v[188:189], v20 offset:49152\n\t"
    "s_add_u32 s88, s69, 512\n\t"
    "v_mfma_f32_16x16x32_bf16 v[88:91], v[200:203], a[80:83], v[88:91]\n\t"
    "ds_read_b64_tr_b16 v[190:191], v20 offset:49664\n\t"
    "v_mfma_f32_16x16x32_bf16 v[92:95], v[200:203], a[84:87], v[92:95]\n\t"
    "ds_read_b64_tr_b16 v[192:193], v21 offset:49152\n\t"
    "s_waitcnt lgkmcnt(7)\n\t"
    "v_mfma_f32_16x16x32_bf16 v[64:67], v[204:207], a[88:91], v[64:67]\n\t"
    "ds_read_b64_tr_b16 v[194:195], v21 offset:49664\n\t"
    "v_mfma_f32_16x16x32_bf16 v[68:71], v[204:207], a[92:95], v[68:71]\n\t"
    "ds_read_b64_tr_b16 v[196:197], v22 offset:49152\n\t"
    "v_mfma_f32_16x16x32_bf16 v[72:75], v[208:211], a[88:91], v[72:75]\n\t"
    "ds_read_b64_tr_b16 v[198:199], v22 offset:49664\n\t"
    "v_mfma_f32_16x16x32_bf16 v[76:79], v[208:211], a[92:95], v[76:79]\n\t"
    "ds_read_b64_tr_b16 v[200:201], v23 offset:49152\n\t"
    "s_waitcnt lgkmcnt(7)\n\t"
    "v_mfma_f32_16x16x32_bf16 v[80:83], v[212:215], a[88:91], v[80:83]\n\t"
    "ds_read_b64_tr_b16 v[202:203], v23 offset:49664\n\t"
    "v_mfma_f32_16x16x32_bf16 v[84:87], v[212:215], a[92:95], v[84:87]\n\t"
    "ds_read_b64_tr_b16 v[204:205], v20 offset:53248\n\t"
    "v_mfma_f32_16x16x32_bf16 v[88:91], v[216:219], a[88:91], v[88:91]\n\t"
    "ds_read_b64_tr_b16 v[206:207], v20 offset:53760\n\t"
    "v_mfma_f32_16x16x32_bf16 v[92:95], v[216:219], a[92:95], v[92:95]\n\t"
    "s_cmp_lt_u32 s60, 10000\n\t"
    "s_cbranch_scc0 LP608%=\n\t"
    "s_sleep 13\n\t"
    "LP608%=:\n\t"
    "ds_read_b64_tr_b16 v[208:209], v21 offset:53248\n\t"
    "s_waitcnt lgkmcnt(7)\n\t"
    "buffer_load_dword v56, v29, s[56:59], 0 offen offset:128 sc0 sc1\n\t"
    "v_mfma_f32_16x16x32_bf16 v[64:67], v[188:191], a[96:99], v[64:67]\n\t"
    "ds_read_b64_tr_b16 v[210:211], v21 offset:53760\n\t"
    "v_mfma_f32_16x16x32_bf16 v[68:71], v[188:191], a[100:103], v[68:71]\n\t"
    "ds_read_b64_tr_b16 v[212:213], v22 offset:53248\n\t"
    "v_mfma_f32_16x16x32_bf16 v[72:75], v[192:195], a[96:99], v[72:75]\n\t"
    "ds_read_b64_tr_b16 v[214:215], v22 offset:53760\n\t"
    "v_mfma_f32_16x16x32_bf16 v[76:79], v[192:195], a[100:103], v[76:79]\n\t"
    "ds_read_b64_tr_b16 v[216:217], v23 offset:53248\n\t"
    "s_waitcnt lgkmcnt(7)\n\t"
    "v_mfma_f32_16x16x32_bf16 v[80:83], v[196:199], a[96:99], v[80:83]\n\t"
    "ds_read_b64_tr_b16 v[218:219], v23 offset:53760\n\t"
    "buffer_load_dwordx4 v12, s[40:43], s88 offen offset:0 lds\n\t"
    "v_mfma_f32_16x16x32_bf16 v[84:87], v[196:199], a[100:103], v[84:87]\n\t"
    "ds_read_b64_tr_b16 v[188:189], v20 offset:57344\n\t"
    "v_mfma_f32_16x16x32_bf16 v[88:91], v[200:203], a[96:99], v[88:91]\n\t"
    "ds_read_b64_tr_b16 v[190:191], v20 offset:57856\n\t"
    "buffer_load_dwordx4 v13, s[40:43], s88 offen offset:1024 lds\n\t"
    "v_mfma_f32_16x16x32_bf16 v[92:95], v[200:203], a[100:103], v[92:95]\n\t"
    "ds_read_b64_tr_b16 v[192:193], v21 offset:57344\n\t"
    "s_waitcnt lgkmcnt(7)\n\t"
    "v_mfma_f32_16x16x32_bf16 v[64:67], v[204:207], a[104:107], v[64:67]\n\t"
    "ds_read_b64_tr_b16 v[194:195], v21 offset:57856\n\t"
    "v_mfma_f32_16x16x32_bf16 v[68:71], v[204:207], a[108:111], v[68:71]\n\t"
    "ds_read_b64_tr_b16 v[196:197], v22 offset:57344\n\t"
    "buffer_load_dwordx4 v14, s[40:43], s88 offen offset:2048 lds\n\t"
    "v_mfma_f32_16x16x32_bf16 v[72:75], v[208:211], a[104:107], v[72:75]\n\t"
    "ds_read_b64_tr_b16 v[198:199], v22 offset:57856\n\t"
    "v_mfma_f32_16x16x32_bf16 v[76:79], v[208:211], a[108:111], v[76:79]\n\t"
    "ds_read_b64_tr_b16 v[200:201], v23 offset:57344\n\t"
    "s_waitcnt lgkmcnt(7)\n\t"
    "v_mfma_f32_16x16x32_bf16 v[80:83], v[212:215], a[104:107], v[80:83]\n\t"
    "ds_read_b64_tr_b16 v[202:203], v23 offset:57856\n\t"
    "buffer_load_dwordx4 v15, s[40:43], s88 offen offset:3072 lds\n\t"
    "v_mfma_f32_16x16x32_bf16 v[84:87], v[212:215], a[108:111], v[84:87]\n\t"
    "ds_read_b64_tr_b16 v[204:205], v20 offset:61440\n\t"
    "v_mfma_f32_16x16x32_bf16 v[88:91], v[216:219], a[104:107], v[88:91]\n\t"
    "ds_read_b64_tr_b16 v[206:207], v20 offset:61952\n\t"
    "v_mfma_f32_16x16x32_bf16 v[92:95], v[216:219], a[108:111], v[92:95]\n\t"
    "ds_read_b64_tr_b16 v[208:209], v21 offset:61440\n\t"
    "s_waitcnt lgkmcnt(7)\n\t"
    "v_mfma_f32_16x16x32_bf16 v[64:67], v[188:191], a[112:115], v[64:67]\n\t"
    "ds_read_b64_tr_b16 v[210:211], v21 offset:61952\n\t"
    "v_mfma_f32_16x16x32_bf16 v[68:71], v[188:191], a[116:119], v[68:71]\n\t"
    "ds_read_b64_tr_b16 v[212:213], v22 offset:61440\n\t"
    "v_mfma_f32_16x16x32_bf16 v[72:75], v[192:195], a[112:115], v[72:75]\n\t"
    "ds_read_b64_tr_b16 v[214:215], v22 offset:61952\n\t"
    "v_mfma_f32_16x16x32_bf16 v[76:79], v[192:195], a[116:119], v[76:79]\n\t"
    "s_waitcnt vmcnt(18)\n\t"
    "s_barrier\n\t"
    "ds_read_b64_tr_b16 v[216:217], v23 offset:61440\n\t"
    "s_waitcnt lgkmcnt(7)\n\t"
    "v_mfma_f32_16x16x32_bf16 v[80:83], v[196:199], a[112:115], v[80:83]\n\t"
    "ds_read_b64_tr_b16 v[218:219], v23 offset:61952\n\t"
    "s_add_u32 m0, s80, 32768\n\t"
    "v_mfma_f32_16x16x32_bf16 v[84:87], v[196:199], a[116:119], v[84:87]\n\t"
    "ds_read_b64_tr_b16 v[188:189], v40 offset:32768\n\t"
    "s_add_u32 s88, s70, 512\n\t"
    "v_mfma_f32_16x16x32_bf16 v[88:91], v[200:203], a[112:115], v[88:91]\n\t"
    "ds_read_b64_tr_b16 v[190:191], v40 offset:33280\n\t"
    "v_mfma_f32_16x16x32_bf16 v[92:95], v[200:203], a[116:119], v[92:95]\n\t"
    "ds_read_b64_tr_b16 v[192:193], v41 offset:32768\n\t"
    "s_waitcnt lgkmcnt(7)\n\t"
    "v_mfma_f32_16x16x32_bf16 v[64:67], v[204:207], a[120:123], v[64:67]\n\t"
    "ds_read_b64_tr_b16 v[194:195], v41 offset:33280\n\t"
    "v_mfma_f32_16x16x32_bf16 v[68:71], v[204:207], a[124:127], v[68:71]\n\t"
    "ds_read_b64_tr_b16 v[196:197], v42 offset:32768\n\t"
    "v_mfma_f32_16x16x32_bf16 v[72:75], v[208:211], a[120:123], v[72:75]\n\t"
    "ds_read_b64_tr_b16 v[198:199], v42 offset:33280\n\t"
    "v_mfma_f32_16x16x32_bf16 v[76:79], v[208:211], a[124:127], v[76:79]\n\t"
    "ds_read_b64_tr_b16 v[200:201], v43 offset:32768\n\t"
    "s_waitcnt lgkmcnt(7)\n\t"
    "v_mfma_f32_16x16x32_bf16 v[80:83], v[212:215], a[120:123], v[80:83]\n\t"
    "ds_read_b64_tr_b16 v[202:203], v43 offset:33280\n\t"
    "v_mfma_f32_16x16x32_bf16 v[84:87], v[212:215], a[124:127], v[84:87]\n\t"
    "ds_read_b64_tr_b16 v[204:205], v40 offset:36864\n\t"
    "v_mfma_f32_16x16x32_bf16 v[88:91], v[216:219], a[120:123], v[88:91]\n\t"
    "ds_read_b64_tr_b16 v[206:207], v40 offset:37376\n\t"
    "v_mfma_f32_16x16x32_bf16 v[92:95], v[216:219], a[124:127], v[92:95]\n\t"
    "s_cmp_lt_u32 s60, 10000\n\t"
    "s_cbranch_scc0 LP640%=\n\t"
    "s_sleep 13\n\t"
    "LP640%=:\n\t"
    "ds_read_b64_tr_b16 v[208:209], v41 offset:36864\n\t"
    "s_waitcnt lgkmcnt(7)\n\t"
    "v_mfma_f32_16x16x32_bf16 v[64:67], v[188:191], a[128:131], v[64:67]\n\t"
    "ds_read_b64_tr_b16 v[210:211], v41 offset:37376\n\t"
    "v_mfma_f32_16x16x32_bf16 v[68:71], v[188:191], a[132:135], v[68:71]\n\t"
    "ds_read_b64_tr_b16 v[212:213], v42 offset:36864\n\t"
    "v_mfma_f32_16x16x32_bf16 v[72:75], v[192:195], a[128:131], v[72:75]\n\t"
    "ds_read_b64_tr_b16 v[214:215], v42 offset:37376\n\t"
    "v_mfma_f32_16x16x32_bf16 v[76:79], v[192:195], a[132:135], v[76:79]\n\t"
    "ds_read_b64_tr_b16 v[216:217], v43 offset:36864\n\t"
    "s_waitcnt lgkmcnt(7)\n\t"
    "v_mfma_f32_16x16x32_bf16 v[80:83], v[196:199], a[128:131], v[80:83]\n\t"
    "ds_read_b64_tr_b16 v[218:219], v43 offset:37376\n\t"
    "v_mfma_f32_16x16x32_bf16 v[84:87], v[196:199], a[132:135], v[84:87]\n\t"
    "ds_read_b64_tr_b16 v[188:189], v40 offset:40960\n\t"
    "s_waitcnt vmcnt(4)\n\t"
    "v_cmp_gt_u32 vcc, v50, v56\n\t"
    "s_nop 0\n\t"
    "s_cbranch_vccnz LS1%=\n\t"
    "LB1%=:\n\t"
    "buffer_load_dwordx4 v[128:131], v24, s[44:47], s77 offen offset:0 sc0 sc1\n\t"
    "v_mfma_f32_16x16x32_bf16 v[88:91], v[200:203], a[128:131], v[88:91]\n\t"
    "ds_read_b64_tr_b16 v[190:191], v40 offset:41472\n\t"
    "v_mfma_f32_16x16x32_bf16 v[92:95], v[200:203], a[132:135], v[92:95]\n\t"
    "ds_read_b64_tr_b16 v[192:193], v41 offset:40960\n\t"
    "s_waitcnt lgkmcnt(7)\n\t"
    "v_mfma_f32_16x16x32_bf16 v[64:67], v[204:207], a[136:139], v[64:67]\n\t"
    "ds_read_b64_tr_b16 v[194:195], v41 offset:41472\n\t"
    "buffer_load_dwordx4 v12, s[40:43], s88 offen offset:0 lds\n\t"
    "v_mfma_f32_16x16x32_bf16 v[68:71], v[204:207], a[140:143], v[68:71]\n\t"
    "ds_read_b64_tr_b16 v[196:197], v42 offset:40960\n\t"
    "buffer_load_dwordx4 v[136:139], v24, s[44:47], s78 offen offset:0 sc0 sc1\n\t"
    "v_mfma_f32_16x16x32_bf16 v[72:75], v[208:211], a[136:139], v[72:75]\n\t"
    "ds_read_b64_tr_b16 v[198:199], v42 offset:41472\n\t"
    "buffer_load_dwordx4 v13, s[40:43], s88 offen offset:1024 lds\n\t"
    "v_mfma_f32_16x16x32_bf16 v[76:79], v[208:211], a[140:143], v[76:79]\n\t"
    "ds_read_b64_tr_b16 v[200:201], v43 offset:40960\n\t"
    "s_waitcnt lgkmcnt(7)\n\t"
    "buffer_load_dwordx4 v[144:147], v24, s[44:47], s79 offen offset:0 sc0 sc1\n\t"
    "v_mfma_f32_16x16x32_bf16 v[80:83], v[212:215], a[136:139], v[80:83]\n\t"
    "ds_read_b64_tr_b16 v[202:203], v43 offset:41472\n\t"
    "buffer_load_dwordx4 v14, s[40:43], s88 offen offset:2048 lds\n\t"
    "v_mfma_f32_16x16x32_bf16 v[84:87], v[212:215], a[140:143], v[84:87]\n\t"
    "ds_read_b64_tr_b16 v[204:205], v40 offset:45056\n\t"
    "v_mfma_f32_16x16x32_bf16 v[88:91], v[216:219], a[136:139], v[88:91]\n\t"
    "ds_read_b64_tr_b16 v[206:207], v40 offset:45568\n\t"
    "v_mfma_f32_16x16x32_bf16 v[92:95], v[216:219], a[140:143], v[92:95]\n\t"
    "ds_read_b64_tr_b16 v[208:209], v41 offset:45056\n\t"
    "s_waitcnt lgkmcnt(7)\n\t"
    "v_mfma_f32_16x16x32_bf16 v[64:67], v[188:191], a[144:147], v[64:67]\n\t"
    "ds_read_b64_tr_b16 v[210:211], v41 offset:45568\n\t"
    "buffer_load_dwordx4 v15, s[40:43], s88 offen offset:3072 lds\n\t"
    "v_mfma_f32_16x16x32_bf16 v[68:71], v[188:191], a[148:151], v[68:71]\n\t"
    "ds_read_b64_tr_b16 v[212:213], v42 offset:45056\n\t"
    "v_mfma_f32_16x16x32_bf16 v[72:75], v[192:195], a[144:147], v[72:75]\n\t"
    "ds_read_b64_tr_b16 v[214:215], v42 offset:45568\n\t"
    "v_mfma_f32_16x16x32_bf16 v[76:79], v[192:195], a[148:151], v[76:79]\n\t"
    "s_waitcnt vmcnt(7)\n\t"
    "s_barrier\n\t"
    "ds_read_b64_tr_b16 v[216:217], v43 offset:45056\n\t"
    "s_waitcnt lgkmcnt(7)\n\t"
    "v_mfma_f32_16x16x32_bf16 v[80:83], v[196:199], a[144:147], v[80:83]\n\t"
    "ds_read_b64_tr_b16 v[218:219], v43 offset:45568\n\t"
    "s_add_u32 m0, s80, 65536\n\t"
    "v_mfma_f32_16x16x32_bf16 v[84:87], v[196:199], a[148:151], v[84:87]\n\t"
    "ds_read_b64_tr_b16 v[188:189], v20 offset:16384\n\t"
    "s_add_u32 s88, s71, 512\n\t"
    "v_mfma_f32_16x16x32_bf16 v[88:91], v[200:203], a[144:147], v[88:91]\n\t"
    "ds_read_b64_tr_b16 v[190:191], v20 offset:16896\n\t"
    "buffer_load_dwordx4 v12, s[40:43], s88 offen offset:0 lds\n\t"
    "v_mfma_f32_16x16x32_bf16 v[92:95], v[200:203], a[148:151], v[92:95]\n\t"
    "ds_read_b64_tr_b16 v[192:193], v21 offset:16384\n\t"
    "s_waitcnt lgkmcnt(7)\n\t"
    "v_mfma_f32_16x16x32_bf16 v[64:67], v[204:207], a[152:155], v[64:67]\n\t"
    "ds_read_b64_tr_b16 v[194:195], v21 offset:16896\n\t"
    "buffer_load_dwordx4 v13, s[40:43], s88 offen offset:1024 lds\n\t"
    "v_mfma_f32_16x16x32_bf16 v[68:71], v[204:207], a[156:159], v[68:71]\n\t"
    "ds_read_b64_tr_b16 v[196:197], v22 offset:16384\n\t"
    "v_mfma_f32_16x16x32_bf16 v[72:75], v[208:211], a[152:155], v[72:75]\n\t"
    "ds_read_b64_tr_b16 v[198:199], v22 offset:16896\n\t"
    "buffer_load_dwordx4 v14, s[40:43], s88 offen offset:2048 lds\n\t"
    "v_mfma_f32_16x16x32_bf16 v[76:79], v[208:211], a[156:159], v[76:79]\n\t"
    "ds_read_b64_tr_b16 v[200:201], v23 offset:16384\n\t"
    "s_waitcnt lgkmcnt(7)\n\t"
    "v_mfma_f32_16x16x32_bf16 v[80:83], v[212:215], a[152:155], v[80:83]\n\t"
    "ds_read_b64_tr_b16 v[202:203], v23 offset:16896\n\t"
    "buffer_load_dwordx4 v15, s[40:43], s88 offen offset:3072 lds\n\t"
    "v_mfma_f32_16x16x32_bf16 v[84:87], v[212:215], a[156:159], v[84:87]\n\t"
    "ds_read_b64_tr_b16 v[204:205], v20 offset:20480\n\t"
    "s_add_u32 m0, s80, 147456\n\t"
    "v_mfma_f32_16x16x32_bf16 v[88:91], v[216:219], a[152:155], v[88:91]\n\t"
    "ds_read_b64_tr_b16 v[206:207], v20 offset:20992\n\t"
    "s_add_u32 s88, s66, 768\n\t"
    "v_mfma_f32_16x16x32_bf16 v[92:95], v[216:219], a[156:159], v[92:95]\n\t"
    "s_cmp_lt_u32 s60, 10000\n\t"
    "s_cbranch_scc0 LP672%=\n\t"
    "s_sleep 13\n\t"
    "LP672%=:\n\t"
    "ds_read_b64_tr_b16 v[208:209], v21 offset:20480\n\t"
    "s_waitcnt lgkmcnt(7)\n\t"
    "v_mfma_f32_16x16x32_bf16 v[64:67], v[188:191], a[160:163], v[64:67]\n\t"
    "ds_read_b64_tr_b16 v[210:211], v21 offset:20992\n\t"
    "buffer_load_dwordx4 v12, s[40:43], s88 offen offset:0 lds\n\t"
    "v_mfma_f32_16x16x32_bf16 v[68:71], v[188:191], a[164:167], v[68:71]\n\t"
    "ds_read_b64_tr_b16 v[212:213], v22 offset:20480\n\t"
    "buffer_load_dwordx4 v13, s[40:43], s88 offen offset:1024 lds\n\t"
    "v_mfma_f32_16x16x32_bf16 v[72:75], v[192:195], a[160:163], v[72:75]\n\t"
    "ds_read_b64_tr_b16 v[214:215], v22 offset:20992\n\t"
    "v_mfma_f32_16x16x32_bf16 v[76:79], v[192:195], a[164:167], v[76:79]\n\t"
    "ds_read_b64_tr_b16 v[216:217], v23 offset:20480\n\t"
    "s_waitcnt lgkmcnt(7)\n\t"
    "v_mfma_f32_16x16x32_bf16 v[80:83], v[196:199], a[160:163], v[80:83]\n\t"
    "ds_read_b64_tr_b16 v[218:219], v23 offset:20992\n\t"
    "buffer_load_dwordx4 v14, s[40:43], s88 offen offset:2048 lds\n\t"
    "v_mfma_f32_16x16x32_bf16 v[84:87], v[196:199], a[164:167], v[84:87]\n\t"
    "ds_read_b64_tr_b16 v[188:189], v20 offset:24576\n\t"
    "v_mfma_f32_16x16x32_bf16 v[88:91], v[200:203], a[160:163], v[88:91]\n\t"
    "ds_read_b64_tr_b16 v[190:191], v20 offset:25088\n\t"
    "v_mfma_f32_16x16x32_bf16 v[92:95], v[200:203], a[164:167], v[92:95]\n\t"
    "ds_read_b64_tr_b16 v[192:193], v21 offset:24576\n\t"
    "s_waitcnt lgkmcnt(7)\n\t"
    "v_mfma_f32_16x16x32_bf16 v[64:67], v[204:207], a[168:171], v[64:67]\n\t"
    "ds_read_b64_tr_b16 v[194:195], v21 offset:25088\n\t"
    "buffer_load_dwordx4 v15, s[40:43], s88 offen offset:3072 lds\n\t"
    "v_mfma_f32_16x16x32_bf16 v[68:71], v[204:207], a[172:175], v[68:71]\n\t"
    "ds_read_b64_tr_b16 v[196:197], v22 offset:24576\n\t"
    "s_add_u32 m0, s80, 0\n\t"
    "v_mfma_f32_16x16x32_bf16 v[72:75], v[208:211], a[168:171], v[72:75]\n\t"
    "ds_read_b64_tr_b16 v[198:199], v22 offset:25088\n\t"
    "s_add_u32 s88, s67, 768\n\t"
    "v_mfma_f32_16x16x32_bf16 v[76:79], v[208:211], a[172:175], v[76:79]\n\t"
    "ds_read_b64_tr_b16 v[200:201], v23 offset:24576\n\t"
    "s_waitcnt lgkmcnt(7)\n\t"
    "v_mfma_f32_16x16x32_bf16 v[80:83], v[212:215], a[168:171], v[80:83]\n\t"
    "ds_read_b64_tr_b16 v[202:203], v23 offset:25088\n\t"
    "buffer_load_dwordx4 v12, s[40:43], s88 offen offset:0 lds\n\t"
    "v_mfma_f32_16x16x32_bf16 v[84:87], v[212:215], a[172:175], v[84:87]\n\t"
    "ds_read_b64_tr_b16 v[204:205], v20 offset:28672\n\t"
    "buffer_load_dwordx4 v13, s[40:43], s88 offen offset:1024 lds\n\t"
    "v_mfma_f32_16x16x32_bf16 v[88:91], v[216:219], a[168:171], v[88:91]\n\t"
    "ds_read_b64_tr_b16 v[206:207], v20 offset:29184\n\t"
    "v_mfma_f32_16x16x32_bf16 v[92:95], v[216:219], a[172:175], v[92:95]\n\t"
    "ds_read_b64_tr_b16 v[208:209], v21 offset:28672\n\t"
    "s_waitcnt lgkmcnt(7)\n\t"
    "s_waitcnt vmcnt(12)\n\t"
    "v_mfma_f32_16x16x32_bf16 v[96:99], v[60:63], v[128:131], v[96:99]\n\t"
    "v_mfma_f32_16x16x32_bf16 v[64:67], v[188:191], a[176:179], v[64:67]\n\t"
    "ds_read_b64_tr_b16 v[210:211], v21 offset:29184\n\t"
    "buffer_load_dwordx4 v14, s[40:43], s88 offen offset:2048 lds\n\t"
    "v_mfma_f32_16x16x32_bf16 v[100:103], v[58:61], v[128:131], v[100:103]\n\t"
    "v_mfma_f32_16x16x32_bf16 v[96:99], v[60:63], v[136:139], v[96:99]\n\t"
    "v_mfma_f32_16x16x32_bf16 v[68:71], v[188:191], a[180:183], v[68:71]\n\t"
    "ds_read_b64_tr_b16 v[212:213], v22 offset:28672\n\t"
    "v_mfma_f32_16x16x32_bf16 v[100:103], v[58:61], v[136:139], v[100:103]\n\t"
    "v_mfma_f32_16x16x32_bf16 v[96:99], v[60:63], v[144:147], v[96:99]\n\t"
    "v_mfma_f32_16x16x32_bf16 v[100:103], v[58:61], v[144:147], v[100:103]\n\t"
    "v_mfma_f32_16x16x32_bf16 v[72:75], v[192:195], a[176:179], v[72:75]\n\t"
    "ds_read_b64_tr_b16 v[214:215], v22 offset:29184\n\t"
    "buffer_load_dwordx4 v15, s[40:43], s88 offen offset:3072 lds\n\t"
    "v_mfma_f32_16x16x32_bf16 v[76:79], v[192:195], a[180:183], v[76:79]\n\t"
    "s_waitcnt vmcnt(12)\n\t"
    "s_barrier\n\t"
    "ds_read_b64_tr_b16 v[216:217], v23 offset:28672\n\t"
    "s_waitcnt lgkmcnt(7)\n\t"
    "s_add_u32 m0, s80, 81920\n\t"
    "s_add_u32 s88, s68, 768\n\t"
    "v_mfma_f32_16x16x32_bf16 v[80:83], v[196:199], a[176:179], v[80:83]\n\t"
    "ds_read_b64_tr_b16 v[218:219], v23 offset:29184\n\t"
    "buffer_load_dwordx4 v12, s[40:43], s88 offen offset:0 lds\n\t"
    "v_mfma_f32_16x16x32_bf16 v[84:87], v[196:199], a[180:183], v[84:87]\n\t"
    "ds_read_b64_tr_b16 v[188:189], v20 offset:32768\n\t"
    "v_mfma_f32_16x16x32_bf16 v[88:91], v[200:203], a[176:179], v[88:91]\n\t"
    "ds_read_b64_tr_b16 v[190:191], v20 offset:33280\n\t"
    "v_mul_f32 v152, v96, v96\n\t"
    "v_mul_f32 v153, v97, v97\n\t"
    "v_mul_f32 v154, v98, v98\n\t"
    "v_mfma_f32_16x16x32_bf16 v[92:95], v[200:203], a[180:183], v[92:95]\n\t"
    "ds_read_b64_tr_b16 v[192:193], v21 offset:32768\n\t"
    "s_waitcnt lgkmcnt(7)\n\t"
    "buffer_load_dwordx4 v13, s[40:43], s88 offen offset:1024 lds\n\t"
    "v_mul_f32 v155, v99, v99\n\t"
    "v_mfma_f32_16x16x32_bf16 v[64:67], v[204:207], a[184:187], v[64:67]\n\t"
    "ds_read_b64_tr_b16 v[194:195], v21 offset:33280\n\t"
    "v_mul_f32 v156, v100, v100\n\t"
    "v_mul_f32 v157, v101, v101\n\t"
    "v_mul_f32 v158, v102, v102\n\t"
    "v_mfma_f32_16x16x32_bf16 v[68:71], v[204:207], a[188:191], v[68:71]\n\t"
    "ds_read_b64_tr_b16 v[196:197], v22 offset:32768\n\t"
    "v_mul_f32 v159, v103, v103\n\t"
    "v_fma_f32 v152, v152, v27, v26\n\t"
    "v_fma_f32 v153, v153, v27, v26\n\t"
    "v_mfma_f32_16x16x32_bf16 v[72:75], v[208:211], a[184:187], v[72:75]\n\t"
    "ds_read_b64_tr_b16 v[198:199], v22 offset:33280\n\t"
    "v_fma_f32 v154, v154, v27, v26\n\t"
    "v_fma_f32 v155, v155, v27, v26\n\t"
    "v_fma_f32 v156, v156, v27, v26\n\t"
    "v_mfma_f32_16x16x32_bf16 v[76:79], v[208:211], a[188:191], v[76:79]\n\t"
    "ds_read_b64_tr_b16 v[200:201], v23 offset:32768\n\t"
    "s_waitcnt lgkmcnt(7)\n\t"
    "buffer_load_dwordx4 v14, s[40:43], s88 offen offset:2048 lds\n\t"
    "v_fma_f32 v157, v157, v27, v26\n\t"
    "v_mfma_f32_16x16x32_bf16 v[80:83], v[212:215], a[184:187], v[80:83]\n\t"
    "ds_read_b64_tr_b16 v[202:203], v23 offset:33280\n\t"
    "v_fma_f32 v158, v158, v27, v26\n\t"
    "v_fma_f32 v159, v159, v27, v26\n\t"
    "v_mul_f32 v152, v152, v96\n\t"
    "v_mfma_f32_16x16x32_bf16 v[84:87], v[212:215], a[188:191], v[84:87]\n\t"
    "ds_read_b64_tr_b16 v[204:205], v20 offset:36864\n\t"
    "v_mul_f32 v153, v153, v97\n\t"
    "v_mul_f32 v154, v154, v98\n\t"
    "v_mul_f32 v155, v155, v99\n\t"
    "v_mfma_f32_16x16x32_bf16 v[88:91], v[216:219], a[184:187], v[88:91]\n\t"
    "ds_read_b64_tr_b16 v[206:207], v20 offset:37376\n\t"
    "v_mul_f32 v156, v156, v100\n\t"
    "v_mul_f32 v157, v157, v101\n\t"
    "v_mul_f32 v158, v158, v102\n\t"
    "v_mfma_f32_16x16x32_bf16 v[92:95], v[216:219], a[188:191], v[92:95]\n\t"
    "s_cmp_lt_u32 s60, 10000\n\t"
    "s_cbranch_scc0 LP704%=\n\t"
    "s_sleep 13\n\t"
    "LP704%=:\n\t"
    "ds_read_b64_tr_b16 v[208:209], v21 offset:36864\n\t"
    "s_waitcnt lgkmcnt(7)\n\t"
    "buffer_load_dwordx4 v15, s[40:43], s88 offen offset:3072 lds\n\t"
    "s_add_u32 m0, s80, 49152\n\t"
    "v_mfma_f32_16x16x32_bf16 v[64:67], v[188:191], a[192:195], v[64:67]\n\t"
    "ds_read_b64_tr_b16 v[210:211], v21 offset:37376\n\t"
    "s_add_u32 s88, s69, 768\n\t"
    "v_mul_f32 v159, v159, v103\n\t"
    "v_exp_f32 v152, v152\n\t"
    "v_mfma_f32_16x16x32_bf16 v[68:71], v[188:191], a[196:199], v[68:71]\n\t"
    "ds_read_b64_tr_b16 v[212:213], v22 offset:36864\n\t"
    "v_exp_f32 v153, v153\n\t"
    "v_exp_f32 v154, v154\n\t"
    "v_exp_f32 v155, v155\n\t"
    "v_mfma_f32_16x16x32_bf16 v[72:75], v[192:195], a[192:195], v[72:75]\n\t"
    "ds_read_b64_tr_b16 v[214:215], v22 offset:37376\n\t"
    "v_exp_f32 v156, v156\n\t"
    "v_exp_f32 v157, v157\n\t"
    "v_exp_f32 v158, v158\n\t"
    "v_mfma_f32_16x16x32_bf16 v[76:79], v[192:195], a[196:199], v[76:79]\n\t"
    "ds_read_b64_tr_b16 v[216:217], v23 offset:36864\n\t"
    "s_waitcnt lgkmcnt(7)\n\t"
    "buffer_load_dwordx4 v12, s[40:43], s88 offen offset:0 lds\n\t"
    "v_exp_f32 v159, v159\n\t"
    "v_mfma_f32_16x16x32_bf16 v[80:83], v[196:199], a[192:195], v[80:83]\n\t"
    "ds_read_b64_tr_b16 v[218:219], v23 offset:37376\n\t"
    "v_add_f32 v152, 1.0, v152\n\t"
    "v_add_f32 v153, 1.0, v153\n\t"
    "v_add_f32 v154, 1.0, v154\n\t"
    "v_mfma_f32_16x16x32_bf16 v[84:87], v[196:199], a[196:199], v[84:87]\n\t"
    "ds_read_b64_tr_b16 v[188:189], v20 offset:40960\n\t"
    "v_add_f32 v155, 1.0, v155\n\t"
    "v_add_f32 v156, 1.0, v156\n\t"
    "v_add_f32 v157, 1.0, v157\n\t"
    "v_mfma_f32_16x16x32_bf16 v[88:91], v[200:203], a[192:195], v[88:91]\n\t"
    "ds_read_b64_tr_b16 v[190:191], v20 offset:41472\n\t"
    "v_add_f32 v158, 1.0, v158\n\t"
    "v_add_f32 v159, 1.0, v159\n\t"
    "v_rcp_f32 v152, v152\n\t"
    "v_mfma_f32_16x16x32_bf16 v[92:95], v[200:203], a[196:199], v[92:95]\n\t"
    "ds_read_b64_tr_b16 v[192:193], v21 offset:40960\n\t"
    "s_waitcnt lgkmcnt(7)\n\t"
    "buffer_load_dwordx4 v13, s[40:43], s88 offen offset:1024 lds\n\t"
    "v_rcp_f32 v153, v153\n\t"
    "v_mfma_f32_16x16x32_bf16 v[64:67], v[204:207], a[200:203], v[64:67]\n\t"
    "ds_read_b64_tr_b16 v[194:195], v21 offset:41472\n\t"
    "v_rcp_f32 v154, v154\n\t"
    "v_rcp_f32 v155, v155\n\t"
    "v_rcp_f32 v156, v156\n\t"
    "v_mfma_f32_16x16x32_bf16 v[68:71], v[204:207], a[204:207], v[68:71]\n\t"
    "ds_read_b64_tr_b16 v[196:197], v22 offset:40960\n\t"
    "v_rcp_f32 v157, v157\n\t"
    "v_rcp_f32 v158, v158\n\t"
    "v_rcp_f32 v159, v159\n\t"
    "v_mfma_f32_16x16x32_bf16 v[72:75], v[208:211], a[200:203], v[72:75]\n\t"
    "ds_read_b64_tr_b16 v[198:199], v22 offset:41472\n\t"
    "v_mul_f32 v152, v152, v96\n\t"
    "v_mul_f32 v153, v153, v97\n\t"
    "v_mul_f32 v154, v154, v98\n\t"
    "v_mfma_f32_16x16x32_bf16 v[76:79], v[208:211], a[204:207], v[76:79]\n\t"
    "ds_read_b64_tr_b16 v[200:201], v23 offset:40960\n\t"
    "s_waitcnt lgkmcnt(7)\n\t"
    "buffer_load_dwordx4 v14, s[40:43], s88 offen offset:2048 lds\n\t"
    "v_mul_f32 v155, v155, v99\n\t"
    "v_mfma_f32_16x16x32_bf16 v[80:83], v[212:215], a[200:203], v[80:83]\n\t"
    "ds_read_b64_tr_b16 v[202:203], v23 offset:41472\n\t"
    "v_mul_f32 v156, v156, v100\n\t"
    "v_mul_f32 v157, v157, v101\n\t"
    "v_mul_f32 v158, v158, v102\n\t"
    "v_mfma_f32_16x16x32_bf16 v[84:87], v[212:215], a[204:207], v[84:87]\n\t"
    "ds_read_b64_tr_b16 v[204:205], v20 offset:45056\n\t"
    "v_mul_f32 v159, v159, v103\n\t"
    "v_cvt_pk_bf16_f32 v160, v152, v153\n\t"
    "v_cvt_pk_bf16_f32 v161, v154, v155\n\t"
    "v_mfma_f32_16x16x32_bf16 v[88:91], v[216:219], a[200:203], v[88:91]\n\t"
    "ds_read_b64_tr_b16 v[206:207], v20 offset:45568\n\t"
    "v_cvt_pk_bf16_f32 v162, v156, v157\n\t"
    "v_cvt_pk_bf16_f32 v163, v158, v159\n\t"
    "s_nop 1\n\t"
    "v_permlane16_swap_b32 v160, v162\n\t"
    "v_permlane16_swap_b32 v161, v163\n\t"
    "v_mfma_f32_16x16x32_bf16 v[92:95], v[216:219], a[204:207], v[92:95]\n\t"
    "ds_read_b64_tr_b16 v[208:209], v21 offset:45056\n\t"
    "s_waitcnt lgkmcnt(7)\n\t"
    "buffer_load_dwordx4 v15, s[40:43], s88 offen offset:3072 lds\n\t"
    "global_store_dwordx4 v25, v[160:163], s[52:53] offset:0 sc0 sc1\n\t"
    "v_mfma_f32_16x16x32_bf16 v[64:67], v[188:191], a[208:211], v[64:67]\n\t"
    "ds_read_b64_tr_b16 v[210:211], v21 offset:45568\n\t"
    "v_add_u32 v24, 6291456, v24\n\t"
    "v_mfma_f32_16x16x32_bf16 v[68:71], v[188:191], a[212:215], v[68:71]\n\t"
    "ds_read_b64_tr_b16 v[212:213], v22 offset:45056\n\t"
    "v_mfma_f32_16x16x32_bf16 v[72:75], v[192:195], a[208:211], v[72:75]\n\t"
    "ds_read_b64_tr_b16 v[214:215], v22 offset:45568\n\t"
    "v_mfma_f32_16x16x32_bf16 v[76:79], v[192:195], a[212:215], v[76:79]\n\t"
    "s_waitcnt vmcnt(17)\n\t"
    "s_barrier\n\t"
    "ds_read_b64_tr_b16 v[216:217], v23 offset:45056\n\t"
    "s_waitcnt lgkmcnt(7)\n\t"
    "v_mfma_f32_16x16x32_bf16 v[80:83], v[196:199], a[208:211], v[80:83]\n\t"
    "ds_read_b64_tr_b16 v[218:219], v23 offset:45568\n\t"
    "v_mfma_f32_16x16x32_bf16 v[84:87], v[196:199], a[212:215], v[84:87]\n\t"
    "ds_read_b64_tr_b16 v[188:189], v40 offset:0\n\t"
    "v_mfma_f32_16x16x32_bf16 v[88:91], v[200:203], a[208:211], v[88:91]\n\t"
    "ds_read_b64_tr_b16 v[190:191], v40 offset:512\n\t"
    "v_mfma_f32_16x16x32_bf16 v[92:95], v[200:203], a[212:215], v[92:95]\n\t"
    "ds_read_b64_tr_b16 v[192:193], v41 offset:0\n\t"
    "s_waitcnt lgkmcnt(7)\n\t"
    "v_mfma_f32_16x16x32_bf16 v[64:67], v[204:207], a[216:219], v[64:67]\n\t"
    "ds_read_b64_tr_b16 v[194:195], v41 offset:512\n\t"
    "v_mfma_f32_16x16x32_bf16 v[68:71], v[204:207], a[220:223], v[68:71]\n\t"
    "ds_read_b64_tr_b16 v[196:197], v42 offset:0\n\t"
    "v_mfma_f32_16x16x32_bf16 v[72:75], v[208:211], a[216:219], v[72:75]\n\t"
    "ds_read_b64_tr_b16 v[198:199], v42 offset:512\n\t"
    "v_mfma_f32_16x16x32_bf16 v[76:79], v[208:211], a[220:223], v[76:79]\n\t"
    "ds_read_b64_tr_b16 v[200:201], v43 offset:0\n\t"
    "s_waitcnt lgkmcnt(7)\n\t"
    "v_mfma_f32_16x16x32_bf16 v[80:83], v[212:215], a[216:219], v[80:83]\n\t"
    "ds_read_b64_tr_b16 v[202:203], v43 offset:512\n\t"
    "v_mfma_f32_16x16x32_bf16 v[84:87], v[212:215], a[220:223], v[84:87]\n\t"
    "ds_read_b64_tr_b16 v[204:205], v40 offset:4096\n\t"
    "v_mfma_f32_16x16x32_bf16 v[88:91], v[216:219], a[216:219], v[88:91]\n\t"
    "ds_read_b64_tr_b16 v[206:207], v40 offset:4608\n\t"
    "v_mfma_f32_16x16x32_bf16 v[92:95], v[216:219], a[220:223], v[92:95]\n\t"
    "s_cmp_lt_u32 s60, 10000\n\t"
    "s_cbranch_scc0 LP736%=\n\t"
    "s_sleep 13\n\t"
    "LP736%=:\n\t"
    "ds_read_b64_tr_b16 v[208:209], v41 offset:4096\n\t"
    "s_waitcnt lgkmcnt(7)\n\t"
    "v_mfma_f32_16x16x32_bf16 v[64:67], v[188:191], a[224:227], v[64:67]\n\t"
    "ds_read_b64_tr_b16 v[210:211], v41 offset:4608\n\t"
    "v_mfma_f32_16x16x32_bf16 v[68:71], v[188:191], a[228:231], v[68:71]\n\t"
    "ds_read_b64_tr_b16 v[212:213], v42 offset:4096\n\t"
    "v_mfma_f32_16x16x32_bf16 v[72:75], v[192:195], a[224:227], v[72:75]\n\t"
    "ds_read_b64_tr_b16 v[214:215], v42 offset:4608\n\t"
    "v_mfma_f32_16x16x32_bf16 v[76:79], v[192:195], a[228:231], v[76:79]\n\t"
    "ds_read_b64_tr_b16 v[216:217], v43 offset:4096\n\t"
    "s_waitcnt lgkmcnt(7)\n\t"
    "v_mfma_f32_16x16x32_bf16 v[80:83], v[196:199], a[224:227], v[80:83]\n\t"
    "ds_read_b64_tr_b16 v[218:219], v43 offset:4608\n\t"
    "v_mfma_f32_16x16x32_bf16 v[84:87], v[196:199], a[228:231], v[84:87]\n\t"
    "ds_read_b64_tr_b16 v[188:189], v40 offset:8192\n\t"
    "v_mfma_f32_16x16x32_bf16 v[88:91], v[200:203], a[224:227], v[88:91]\n\t"
    "ds_read_b64_tr_b16 v[190:191], v40 offset:8704\n\t"
    "v_mfma_f32_16x16x32_bf16 v[92:95], v[200:203], a[228:231], v[92:95]\n\t"
    "ds_read_b64_tr_b16 v[192:193], v41 offset:8192\n\t"
    "s_waitcnt lgkmcnt(7)\n\t"
    "v_mfma_f32_16x16x32_bf16 v[64:67], v[204:207], a[232:235], v[64:67]\n\t"
    "ds_read_b64_tr_b16 v[194:195], v41 offset:8704\n\t"
    "v_mfma_f32_16x16x32_bf16 v[68:71], v[204:207], a[236:239], v[68:71]\n\t"
    "ds_read_b64_tr_b16 v[196:197], v42 offset:8192\n\t"
    "v_mfma_f32_16x16x32_bf16 v[72:75], v[208:211], a[232:235], v[72:75]\n\t"
    "ds_read_b64_tr_b16 v[198:199], v42 offset:8704\n\t"
    "v_mfma_f32_16x16x32_bf16 v[76:79], v[208:211], a[236:239], v[76:79]\n\t"
    "ds_read_b64_tr_b16 v[200:201], v43 offset:8192\n\t"
    "s_waitcnt lgkmcnt(7)\n\t"
    "v_mfma_f32_16x16x32_bf16 v[80:83], v[212:215], a[232:235], v[80:83]\n\t"
    "ds_read_b64_tr_b16 v[202:203], v43 offset:8704\n\t"
    "v_mfma_f32_16x16x32_bf16 v[84:87], v[212:215], a[236:239], v[84:87]\n\t"
    "ds_read_b64_tr_b16 v[204:205], v40 offset:12288\n\t"
    "v_mfma_f32_16x16x32_bf16 v[88:91], v[216:219], a[232:235], v[88:91]\n\t"
    "ds_read_b64_tr_b16 v[206:207], v40 offset:12800\n\t"
    "v_mfma_f32_16x16x32_bf16 v[92:95], v[216:219], a[236:239], v[92:95]\n\t"
    "ds_read_b64_tr_b16 v[208:209], v41 offset:12288\n\t"
    "s_waitcnt lgkmcnt(7)\n\t"
    "v_mfma_f32_16x16x32_bf16 v[64:67], v[188:191], a[240:243], v[64:67]\n\t"
    "ds_read_b64_tr_b16 v[210:211], v41 offset:12800\n\t"
    "v_mfma_f32_16x16x32_bf16 v[68:71], v[188:191], a[244:247], v[68:71]\n\t"
    "ds_read_b64_tr_b16 v[212:213], v42 offset:12288\n\t"
    "v_mfma_f32_16x16x32_bf16 v[72:75], v[192:195], a[240:243], v[72:75]\n\t"
    "ds_read_b64_tr_b16 v[214:215], v42 offset:12800\n\t"
    "v_mfma_f32_16x16x32_bf16 v[76:79], v[192:195], a[244:247], v[76:79]\n\t"
    "s_waitcnt vmcnt(38)\n\t"
    "s_barrier\n\t"
    "ds_read_b64_tr_b16 v[216:217], v43 offset:12288\n\t"
    "s_waitcnt lgkmcnt(7)\n\t"
    "v_mfma_f32_16x16x32_bf16 v[80:83], v[196:199], a[240:243], v[80:83]\n\t"
    "ds_read_b64_tr_b16 v[218:219], v43 offset:12800\n\t"
    "v_mfma_f32_16x16x32_bf16 v[84:87], v[196:199], a[244:247], v[84:87]\n\t"
    "ds_read_b64_tr_b16 v[188:189], v42 offset:49152\n\t"
    "v_mfma_f32_16x16x32_bf16 v[88:91], v[200:203], a[240:243], v[88:91]\n\t"
    "ds_read_b64_tr_b16 v[190:191], v42 offset:49664\n\t"
    "v_mfma_f32_16x16x32_bf16 v[92:95], v[200:203], a[244:247], v[92:95]\n\t"
    "ds_read_b64_tr_b16 v[192:193], v43 offset:49152\n\t"
    "s_waitcnt lgkmcnt(7)\n\t"
    "v_mfma_f32_16x16x32_bf16 v[64:67], v[204:207], a[248:251], v[64:67]\n\t"
    "ds_read_b64_tr_b16 v[194:195], v43 offset:49664\n\t"
    "v_mfma_f32_16x16x32_bf16 v[68:71], v[204:207], a[252:255], v[68:71]\n\t"
    "ds_read_b64_tr_b16 v[196:197], v42 offset:53248\n\t"
    "v_mfma_f32_16x16x32_bf16 v[72:75], v[208:211], a[248:251], v[72:75]\n\t"
    "ds_read_b64_tr_b16 v[198:199], v42 offset:53760\n\t"
    "v_mfma_f32_16x16x32_bf16 v[76:79], v[208:211], a[252:255], v[76:79]\n\t"
    "ds_read_b64_tr_b16 v[200:201], v43 offset:53248\n\t"
    "s_waitcnt lgkmcnt(7)\n\t"
    "v_mfma_f32_16x16x32_bf16 v[80:83], v[212:215], a[248:251], v[80:83]\n\t"
    "ds_read_b64_tr_b16 v[202:203], v43 offset:53760\n\t"
    "v_mfma_f32_16x16x32_bf16 v[84:87], v[212:215], a[252:255], v[84:87]\n\t"
    "ds_read_b64_tr_b16 v[204:205], v42 offset:57344\n\t"
    "v_mfma_f32_16x16x32_bf16 v[88:91], v[216:219], a[248:251], v[88:91]\n\t"
    "ds_read_b64_tr_b16 v[206:207], v42 offset:57856\n\t"
    "v_mfma_f32_16x16x32_bf16 v[92:95], v[216:219], a[252:255], v[92:95]\n\t"
    "s_nop 2\n\t"
    "s_cmp_lt_u32 s60, 10000\n\t"
    "s_cbranch_scc0 LP768%=\n\t"
    "s_sleep 13\n\t"
    "LP768%=:\n\t"
    "ds_read_b64_tr_b16 v[208:209], v43 offset:57344\n\t"
    "s_waitcnt lgkmcnt(7)\n\t"
    "v_bfi_b32 v104, 63, v48, v80\n\t"
    "v_bfi_b32 v105, 63, v48, v81\n\t"
    "v_bfi_b32 v106, 63, v48, v82\n\t"
    "v_bfi_b32 v107, 63, v48, v83\n\t"
    "v_mfma_f32_16x16x32_bf16 v[80:83], v[188:191], a[0:3], 0\n\t"
    "buffer_store_dwordx4 v[104:107], v24, s[44:47], s74 offen offset:0 sc0 sc1\n\t"
    "ds_read_b64_tr_b16 v[210:211], v43 offset:57856\n\t"
    "v_bfi_b32 v120, 63, v48, v84\n\t"
    "v_bfi_b32 v121, 63, v48, v85\n\t"
    "v_bfi_b32 v122, 63, v48, v86\n\t"
    "v_bfi_b32 v123, 63, v48, v87\n\t"
    "v_mfma_f32_16x16x32_bf16 v[84:87], v[188:191], a[4:7], 0\n\t"
    "ds_read_b64_tr_b16 v[212:213], v42 offset:61440\n\t"
    "v_bfi_b32 v108, 63, v48, v88\n\t"
    "v_bfi_b32 v109, 63, v48, v89\n\t"
    "v_bfi_b32 v110, 63, v48, v90\n\t"
    "v_bfi_b32 v111, 63, v48, v91\n\t"
    "v_mfma_f32_16x16x32_bf16 v[88:91], v[192:195], a[0:3], 0\n\t"
    "ds_read_b64_tr_b16 v[214:215], v42 offset:61952\n\t"
    "v_bfi_b32 v124, 63, v48, v92\n\t"
    "v_bfi_b32 v125, 63, v48, v93\n\t"
    "v_bfi_b32 v126, 63, v48, v94\n\t"
    "v_bfi_b32 v127, 63, v48, v95\n\t"
    "v_mfma_f32_16x16x32_bf16 v[92:95], v[192:195], a[4:7], 0\n\t"
    "buffer_store_dwordx4 v[120:123], v24, s[44:47], s76 offen offset:0 sc0 sc1\n\t"
    "s_waitcnt vmcnt(35)\n\t"
    "s_barrier\n\t"
    "ds_read_b64_tr_b16 v[216:217], v43 offset:61440\n\t"
    "s_waitcnt lgkmcnt(7)\n\t"
    "v_bfi_b32 v112, 63, v48, v68\n\t"
    "v_bfi_b32 v113, 63, v48, v69\n\t"
    "v_bfi_b32 v114, 63, v48, v70\n\t"
    "v_mfma_f32_16x16x32_bf16 v[80:83], v[196:199], a[8:11], v[80:83]\n\t"
    "ds_read_b64_tr_b16 v[218:219], v43 offset:61952\n\t"
    "v_bfi_b32 v115, 63, v48, v71\n\t"
    "v_bfi_b32 v116, 63, v48, v76\n\t"
    "v_bfi_b32 v117, 63, v48, v77\n\t"
    "v_mfma_f32_16x16x32_bf16 v[84:87], v[196:199], a[12:15], v[84:87]\n\t"
    "ds_read_b64_tr_b16 v[188:189], v46 offset:0\n\t"
    "v_bfi_b32 v118, 63, v48, v78\n\t"
    "v_bfi_b32 v119, 63, v48, v79\n\t"
    "v_mov_b32 v96, v64\n\t"
    "v_mfma_f32_16x16x32_bf16 v[88:91], v[200:203], a[8:11], v[88:91]\n\t"
    "buffer_store_dwordx4 v[108:111], v24, s[44:47], s74 offen offset:1024 sc0 sc1\n\t"
    "ds_read_b64_tr_b16 v[190:191], v46 offset:512\n\t"
    "v_mov_b32 v97, v65\n\t"
    "v_mov_b32 v98, v66\n\t"
    "v_mov_b32 v99, v67\n\t"
    "v_mov_b32 v100, v72\n\t"
    "v_mfma_f32_16x16x32_bf16 v[92:95], v[200:203], a[12:15], v[92:95]\n\t"
    "ds_read_b64_tr_b16 v[192:193], v47 offset:0\n\t"
    "s_waitcnt lgkmcnt(7)\n\t"
    "v_mov_b32 v101, v73\n\t"
    "v_mov_b32 v102, v74\n\t"
    "v_mov_b32 v103, v75\n\t"
    "v_mfma_f32_16x16x32_bf16 v[80:83], v[204:207], a[16:19], v[80:83]\n\t"
    "ds_read_b64_tr_b16 v[194:195], v47 offset:512\n\t"
    "v_mfma_f32_16x16x32_bf16 v[84:87], v[204:207], a[20:23], v[84:87]\n\t"
    "buffer_store_dwordx4 v[124:127], v24, s[44:47], s76 offen offset:1024 sc0 sc1\n\t"
    "ds_read_b64_tr_b16 v[196:197], v46 offset:4096\n\t"
    "v_mfma_f32_16x16x32_bf16 v[88:91], v[208:211], a[16:19], v[88:91]\n\t"
    "ds_read_b64_tr_b16 v[198:199], v46 offset:4608\n\t"
    "v_mfma_f32_16x16x32_bf16 v[92:95], v[208:211], a[20:23], v[92:95]\n\t"
    "ds_read_b64_tr_b16 v[200:201], v47 offset:4096\n\t"
    "s_waitcnt lgkmcnt(7)\n\t"
    "v_mfma_f32_16x16x32_bf16 v[80:83], v[212:215], a[24:27], v[80:83]\n\t"
    "buffer_store_dwordx4 v[112:115], v24, s[44:47], s75 offen offset:0 sc0 sc1\n\t"
    "ds_read_b64_tr_b16 v[202:203], v47 offset:4608\n\t"
    "v_mfma_f32_16x16x32_bf16 v[84:87], v[212:215], a[28:31], v[84:87]\n\t"
    "ds_read_b64_tr_b16 v[204:205], v46 offset:8192\n\t"
    "v_mfma_f32_16x16x32_bf16 v[88:91], v[216:219], a[24:27], v[88:91]\n\t"
    "ds_read_b64_tr_b16 v[206:207], v46 offset:8704\n\t"
    "v_mfma_f32_16x16x32_bf16 v[92:95], v[216:219], a[28:31], v[92:95]\n\t"
    "buffer_store_dwordx4 v[116:119], v24, s[44:47], s75 offen offset:1024 sc0 sc1\n\t"
    "ds_read_b64_tr_b16 v[208:209], v47 offset:8192\n\t"
    "s_waitcnt lgkmcnt(7)\n\t"
    "v_mfma_f32_16x16x32_bf16 v[80:83], v[188:191], a[32:35], v[80:83]\n\t"
    "ds_read_b64_tr_b16 v[210:211], v47 offset:8704\n\t"
    "v_mfma_f32_16x16x32_bf16 v[84:87], v[188:191], a[36:39], v[84:87]\n\t"
    "ds_read_b64_tr_b16 v[212:213], v46 offset:12288\n\t"
    "v_mfma_f32_16x16x32_bf16 v[88:91], v[192:195], a[32:35], v[88:91]\n\t"
    "ds_read_b64_tr_b16 v[214:215], v46 offset:12800\n\t"
    "v_mfma_f32_16x16x32_bf16 v[92:95], v[192:195], a[36:39], v[92:95]\n\t"
    "s_waitcnt vmcnt(19)\n\t"
    "s_barrier\n\t"
    "ds_read_b64_tr_b16 v[216:217], v47 offset:12288\n\t"
    "s_waitcnt lgkmcnt(7)\n\t"
    "v_mfma_f32_16x16x32_bf16 v[80:83], v[196:199], a[40:43], v[80:83]\n\t"
    "ds_read_b64_tr_b16 v[218:219], v47 offset:12800\n\t"
    "v_mfma_f32_16x16x32_bf16 v[84:87], v[196:199], a[44:47], v[84:87]\n\t"
    "ds_read_b64_tr_b16 v[188:189], v46 offset:16384\n\t"
    "v_mfma_f32_16x16x32_bf16 v[88:91], v[200:203], a[40:43], v[88:91]\n\t"
    "ds_read_b64_tr_b16 v[190:191], v46 offset:16896\n\t"
    "v_mfma_f32_16x16x32_bf16 v[92:95], v[200:203], a[44:47], v[92:95]\n\t"
    "ds_read_b64_tr_b16 v[192:193], v47 offset:16384\n\t"
    "s_waitcnt lgkmcnt(7)\n\t"
    "v_mfma_f32_16x16x32_bf16 v[80:83], v[204:207], a[48:51], v[80:83]\n\t"
    "ds_read_b64_tr_b16 v[194:195], v47 offset:16896\n\t"
    "v_mfma_f32_16x16x32_bf16 v[84:87], v[204:207], a[52:55], v[84:87]\n\t"
    "ds_read_b64_tr_b16 v[196:197], v46 offset:20480\n\t"
    "v_mfma_f32_16x16x32_bf16 v[88:91], v[208:211], a[48:51], v[88:91]\n\t"
    "ds_read_b64_tr_b16 v[198:199], v46 offset:20992\n\t"
    "v_mfma_f32_16x16x32_bf16 v[92:95], v[208:211], a[52:55], v[92:95]\n\t"
    "ds_read_b64_tr_b16 v[200:201], v47 offset:20480\n\t"
    "s_waitcnt lgkmcnt(7)\n\t"
    "v_mfma_f32_16x16x32_bf16 v[80:83], v[212:215], a[56:59], v[80:83]\n\t"
    "ds_read_b64_tr_b16 v[202:203], v47 offset:20992\n\t"
    "v_mfma_f32_16x16x32_bf16 v[84:87], v[212:215], a[60:63], v[84:87]\n\t"
    "ds_read_b64_tr_b16 v[204:205], v46 offset:24576\n\t"
    "v_mfma_f32_16x16x32_bf16 v[88:91], v[216:219], a[56:59], v[88:91]\n\t"
    "ds_read_b64_tr_b16 v[206:207], v46 offset:25088\n\t"
    "v_mfma_f32_16x16x32_bf16 v[92:95], v[216:219], a[60:63], v[92:95]\n\t"
    "s_cmp_lt_u32 s60, 10000\n\t"
    "s_cbranch_scc0 LP800%=\n\t"
    "s_sleep 13\n\t"
    "LP800%=:\n\t"
    "ds_read_b64_tr_b16 v[208:209], v47 offset:24576\n\t"
    "s_waitcnt lgkmcnt(7)\n\t"
    "v_mfma_f32_16x16x32_bf16 v[80:83], v[188:191], a[64:67], v[80:83]\n\t"
    "ds_read_b64_tr_b16 v[210:211], v47 offset:25088\n\t"
    "v_mfma_f32_16x16x32_bf16 v[84:87], v[188:191], a[68:71], v[84:87]\n\t"
    "ds_read_b64_tr_b16 v[212:213], v46 offset:28672\n\t"
    "v_mfma_f32_16x16x32_bf16 v[88:91], v[192:195], a[64:67], v[88:91]\n\t"
    "ds_read_b64_tr_b16 v[214:215], v46 offset:29184\n\t"
    "v_mfma_f32_16x16x32_bf16 v[92:95], v[192:195], a[68:71], v[92:95]\n\t"
    "s_waitcnt vmcnt(15)\n\t"
    "s_barrier\n\t"
    "ds_read_b64_tr_b16 v[216:217], v47 offset:28672\n\t"
    "s_waitcnt lgkmcnt(7)\n\t"
    "v_mfma_f32_16x16x32_bf16 v[80:83], v[196:199], a[72:75], v[80:83]\n\t"
    "ds_read_b64_tr_b16 v[218:219], v47 offset:29184\n\t"
    "v_mfma_f32_16x16x32_bf16 v[84:87], v[196:199], a[76:79], v[84:87]\n\t"
    "ds_read_b64_tr_b16 v[188:189], v22 offset:0\n\t"
    "v_mfma_f32_16x16x32_bf16 v[88:91], v[200:203], a[72:75], v[88:91]\n\t"
    "ds_read_b64_tr_b16 v[190:191], v22 offset:512\n\t"
    "v_mfma_f32_16x16x32_bf16 v[92:95], v[200:203], a[76:79], v[92:95]\n\t"
    "ds_read_b64_tr_b16 v[192:193], v23 offset:0\n\t"
    "s_waitcnt lgkmcnt(7)\n\t"
    "v_mfma_f32_16x16x32_bf16 v[80:83], v[204:207], a[80:83], v[80:83]\n\t"
    "ds_read_b64_tr_b16 v[194:195], v23 offset:512\n\t"
    "v_mfma_f32_16x16x32_bf16 v[84:87], v[204:207], a[84:87], v[84:87]\n\t"
    "ds_read_b64_tr_b16 v[196:197], v22 offset:4096\n\t"
    "v_mfma_f32_16x16x32_bf16 v[88:91], v[208:211], a[80:83], v[88:91]\n\t"
    "ds_read_b64_tr_b16 v[198:199], v22 offset:4608\n\t"
    "v_mfma_f32_16x16x32_bf16 v[92:95], v[208:211], a[84:87], v[92:95]\n\t"
    "ds_read_b64_tr_b16 v[200:201], v23 offset:4096\n\t"
    "s_waitcnt lgkmcnt(7)\n\t"
    "buffer_load_dwordx4 v[128:131], v24, s[44:47], s77 offen offset:0 sc0 sc1\n\t"
    "v_mfma_f32_16x16x32_bf16 v[80:83], v[212:215], a[88:91], v[80:83]\n\t"
    "ds_read_b64_tr_b16 v[202:203], v23 offset:4608\n\t"
    "v_mfma_f32_16x16x32_bf16 v[84:87], v[212:215], a[92:95], v[84:87]\n\t"
    "ds_read_b64_tr_b16 v[204:205], v22 offset:8192\n\t"
    "v_mfma_f32_16x16x32_bf16 v[88:91], v[216:219], a[88:91], v[88:91]\n\t"
    "ds_read_b64_tr_b16 v[206:207], v22 offset:8704\n\t"
    "buffer_load_dwordx4 v[132:135], v24, s[44:47], s77 offen offset:1024 sc0 sc1\n\t"
    "v_mfma_f32_16x16x32_bf16 v[92:95], v[216:219], a[92:95], v[92:95]\n\t"
    "ds_read_b64_tr_b16 v[208:209], v23 offset:8192\n\t"
    "s_waitcnt lgkmcnt(7)\n\t"
    "v_mfma_f32_16x16x32_bf16 v[80:83], v[188:191], a[96:99], v[80:83]\n\t"
    "ds_read_b64_tr_b16 v[210:211], v23 offset:8704\n\t"
    "v_mfma_f32_16x16x32_bf16 v[84:87], v[188:191], a[100:103], v[84:87]\n\t"
    "ds_read_b64_tr_b16 v[212:213], v22 offset:12288\n\t"
    "buffer_load_dwordx4 v[136:139], v24, s[44:47], s78 offen offset:0 sc0 sc1\n\t"
    "v_mfma_f32_16x16x32_bf16 v[88:91], v[192:195], a[96:99], v[88:91]\n\t"
    "ds_read_b64_tr_b16 v[214:215], v22 offset:12800\n\t"
    "v_mfma_f32_16x16x32_bf16 v[92:95], v[192:195], a[100:103], v[92:95]\n\t"
    "s_waitcnt vmcnt(14)\n\t"
    "s_barrier\n\t"
    "ds_read_b64_tr_b16 v[216:217], v23 offset:12288\n\t"
    "s_waitcnt lgkmcnt(7)\n\t"
    "v_mfma_f32_16x16x32_bf16 v[80:83], v[196:199], a[104:107], v[80:83]\n\t"
    "ds_read_b64_tr_b16 v[218:219], v23 offset:12800\n\t"
    "s_add_u32 m0, s80, 98304\n\t"
    "v_mfma_f32_16x16x32_bf16 v[84:87], v[196:199], a[108:111], v[84:87]\n\t"
    "ds_read_b64_tr_b16 v[188:189], v42 offset:16384\n\t"
    "s_add_u32 s88, s70, 768\n\t"
    "buffer_load_dwordx4 v12, s[40:43], s88 offen offset:0 lds\n\t"
    "buffer_load_dwordx4 v[140:143], v24, s[44:47], s78 offen offset:1024 sc0 sc1\n\t"
    "v_mfma_f32_16x16x32_bf16 v[88:91], v[200:203], a[104:107], v[88:91]\n\t"
    "ds_read_b64_tr_b16 v[190:191], v42 offset:16896\n\t"
    "buffer_load_dwordx4 v13, s[40:43], s88 offen offset:1024 lds\n\t"
    "v_mfma_f32_16x16x32_bf16 v[92:95], v[200:203], a[108:111], v[92:95]\n\t"
    "ds_read_b64_tr_b16 v[192:193], v43 offset:16384\n\t"
    "s_waitcnt lgkmcnt(7)\n\t"
    "buffer_load_dwordx4 v[144:147], v24, s[44:47], s79 offen offset:0 sc0 sc1\n\t"
    "v_mfma_f32_16x16x32_bf16 v[80:83], v[204:207], a[112:115], v[80:83]\n\t"
    "ds_read_b64_tr_b16 v[194:195], v43 offset:16896\n\t"
    "buffer_load_dwordx4 v14, s[40:43], s88 offen offset:2048 lds\n\t"
    "v_mfma_f32_16x16x32_bf16 v[84:87], v[204:207], a[116:119], v[84:87]\n\t"
    "ds_read_b64_tr_b16 v[196:197], v42 offset:20480\n\t"
    "v_mfma_f32_16x16x32_bf16 v[88:91], v[208:211], a[112:115], v[88:91]\n\t"
    "ds_read_b64_tr_b16 v[198:199], v42 offset:20992\n\t"
    "buffer_load_dwordx4 v15, s[40:43], s88 offen offset:3072 lds\n\t"
    "v_mfma_f32_16x16x32_bf16 v[92:95], v[208:211], a[116:119], v[92:95]\n\t"
    "ds_read_b64_tr_b16 v[200:201], v43 offset:20480\n\t"
    "s_waitcnt lgkmcnt(7)\n\t"
    "s_add_u32 m0, s80, 16384\n\t"
    "s_add_u32 s88, s71, 768\n\t"
    "buffer_load_dwordx4 v[148:151], v24, s[44:47], s79 offen offset:1024 sc0 sc1\n\t"
    "v_mfma_f32_16x16x32_bf16 v[80:83], v[212:215], a[120:123], v[80:83]\n\t"
    "ds_read_b64_tr_b16 v[202:203], v43 offset:20992\n\t"
    "buffer_load_dwordx4 v12, s[40:43], s88 offen offset:0 lds\n\t"
    "v_mfma_f32_16x16x32_bf16 v[84:87], v[212:215], a[124:127], v[84:87]\n\t"
    "ds_read_b64_tr_b16 v[204:205], v42 offset:24576\n\t"
    "buffer_load_dwordx4 v13, s[40:43], s88 offen offset:1024 lds\n\t"
    "v_mfma_f32_16x16x32_bf16 v[88:91], v[216:219], a[120:123], v[88:91]\n\t"
    "ds_read_b64_tr_b16 v[206:207], v42 offset:25088\n\t"
    "v_mfma_f32_16x16x32_bf16 v[92:95], v[216:219], a[124:127], v[92:95]\n\t"
    "s_cmp_lt_u32 s60, 10000\n\t"
    "s_cbranch_scc0 LP832%=\n\t"
    "s_sleep 13\n\t"
    "LP832%=:\n\t"
    "ds_read_b64_tr_b16 v[208:209], v43 offset:24576\n\t"
    "s_waitcnt lgkmcnt(7)\n\t"
    "v_mfma_f32_16x16x32_bf16 v[80:83], v[188:191], a[128:131], v[80:83]\n\t"
    "ds_read_b64_tr_b16 v[210:211], v43 offset:25088\n\t"
    "buffer_load_dwordx4 v14, s[40:43], s88 offen offset:2048 lds\n\t"
    "v_mfma_f32_16x16x32_bf16 v[84:87], v[188:191], a[132:135], v[84:87]\n\t"
    "ds_read_b64_tr_b16 v[212:213], v42 offset:28672\n\t"
    "buffer_load_dwordx4 v15, s[40:43], s88 offen offset:3072 lds\n\t"
    "v_mfma_f32_16x16x32_bf16 v[88:91], v[192:195], a[128:131], v[88:91]\n\t"
    "ds_read_b64_tr_b16 v[214:215], v42 offset:29184\n\t"
    "v_mfma_f32_16x16x32_bf16 v[92:95], v[192:195], a[132:135], v[92:95]\n\t"
    "s_waitcnt vmcnt(21)\n\t"
    "s_barrier\n\t"
    "ds_read_b64_tr_b16 v[216:217], v43 offset:28672\n\t"
    "s_waitcnt lgkmcnt(7)\n\t"
    "v_mfma_f32_16x16x32_bf16 v[80:83], v[196:199], a[136:139], v[80:83]\n\t"
    "ds_read_b64_tr_b16 v[218:219], v43 offset:29184\n\t"
    "v_mfma_f32_16x16x32_bf16 v[84:87], v[196:199], a[140:143], v[84:87]\n\t"
    "ds_read_b64_tr_b16 v[188:189], v22 offset:49152\n\t"
    "v_mfma_f32_16x16x32_bf16 v[88:91], v[200:203], a[136:139], v[88:91]\n\t"
    "ds_read_b64_tr_b16 v[190:191], v22 offset:49664\n\t"
    "v_mfma_f32_16x16x32_bf16 v[92:95], v[200:203], a[140:143], v[92:95]\n\t"
    "ds_read_b64_tr_b16 v[192:193], v23 offset:49152\n\t"
    "s_waitcnt lgkmcnt(7)\n\t"
    "v_mfma_f32_16x16x32_bf16 v[80:83], v[204:207], a[144:147], v[80:83]\n\t"
    "ds_read_b64_tr_b16 v[194:195], v23 offset:49664\n\t"
    "v_mfma_f32_16x16x32_bf16 v[84:87], v[204:207], a[148:151], v[84:87]\n\t"
    "ds_read_b64_tr_b16 v[196:197], v22 offset:53248\n\t"
    "v_mfma_f32_16x16x32_bf16 v[88:91], v[208:211], a[144:147], v[88:91]\n\t"
    "ds_read_b64_tr_b16 v[198:199], v22 offset:53760\n\t"
    "v_mfma_f32_16x16x32_bf16 v[92:95], v[208:211], a[148:151], v[92:95]\n\t"
    "ds_read_b64_tr_b16 v[200:201], v23 offset:53248\n\t"
    "s_waitcnt lgkmcnt(7)\n\t"
    "v_mfma_f32_16x16x32_bf16 v[80:83], v[212:215], a[152:155], v[80:83]\n\t"
    "ds_read_b64_tr_b16 v[202:203], v23 offset:53760\n\t"
    "v_mfma_f32_16x16x32_bf16 v[84:87], v[212:215], a[156:159], v[84:87]\n\t"
    "ds_read_b64_tr_b16 v[204:205], v22 offset:57344\n\t"
    "v_mfma_f32_16x16x32_bf16 v[88:91], v[216:219], a[152:155], v[88:91]\n\t"
    "ds_read_b64_tr_b16 v[206:207], v22 offset:57856\n\t"
    "v_mfma_f32_16x16x32_bf16 v[92:95], v[216:219], a[156:159], v[92:95]\n\t"
    "ds_read_b64_tr_b16 v[208:209], v23 offset:57344\n\t"
    "s_waitcnt lgkmcnt(7)\n\t"
    "v_mfma_f32_16x16x32_bf16 v[80:83], v[188:191], a[160:163], v[80:83]\n\t"
    "ds_read_b64_tr_b16 v[210:211], v23 offset:57856\n\t"
    "v_mfma_f32_16x16x32_bf16 v[84:87], v[188:191], a[164:167], v[84:87]\n\t"
    "ds_read_b64_tr_b16 v[212:213], v22 offset:61440\n\t"
    "v_mfma_f32_16x16x32_bf16 v[88:91], v[192:195], a[160:163], v[88:91]\n\t"
    "ds_read_b64_tr_b16 v[214:215], v22 offset:61952\n\t"
    "v_mfma_f32_16x16x32_bf16 v[92:95], v[192:195], a[164:167], v[92:95]\n\t"
    "s_waitcnt vmcnt(5)\n\t"
    "s_barrier\n\t"
    "ds_read_b64_tr_b16 v[216:217], v23 offset:61440\n\t"
    "s_waitcnt lgkmcnt(7)\n\t"
    "v_mfma_f32_16x16x32_bf16 v[80:83], v[196:199], a[168:171], v[80:83]\n\t"
    "ds_read_b64_tr_b16 v[218:219], v23 offset:61952\n\t"
    "v_mfma_f32_16x16x32_bf16 v[84:87], v[196:199], a[172:175], v[84:87]\n\t"
    "ds_read_b64_tr_b16 v[188:189], v42 offset:32768\n\t"
    "v_mfma_f32_16x16x32_bf16 v[88:91], v[200:203], a[168:171], v[88:91]\n\t"
    "ds_read_b64_tr_b16 v[190:191], v42 offset:33280\n\t"
    "v_mfma_f32_16x16x32_bf16 v[92:95], v[200:203], a[172:175], v[92:95]\n\t"
    "ds_read_b64_tr_b16 v[192:193], v43 offset:32768\n\t"
    "s_waitcnt lgkmcnt(7)\n\t"
    "v_mfma_f32_16x16x32_bf16 v[80:83], v[204:207], a[176:179], v[80:83]\n\t"
    "ds_read_b64_tr_b16 v[194:195], v43 offset:33280\n\t"
    "v_mfma_f32_16x16x32_bf16 v[84:87], v[204:207], a[180:183], v[84:87]\n\t"
    "ds_read_b64_tr_b16 v[196:197], v42 offset:36864\n\t"
    "s_waitcnt vmcnt(4)\n\t"
    "v_bitop3_b32 v54, v128, v129, v48 bitop3:0x7e\n\t"
    "v_bitop3_b32 v55, v130, v131, v48 bitop3:0x7e\n\t"
    "v_or3_b32 v49, v54, v55, v51\n\t"
    "v_mfma_f32_16x16x32_bf16 v[88:91], v[208:211], a[176:179], v[88:91]\n\t"
    "ds_read_b64_tr_b16 v[198:199], v42 offset:37376\n\t"
    "v_bitop3_b32 v54, v132, v133, v48 bitop3:0x7e\n\t"
    "v_bitop3_b32 v55, v134, v135, v48 bitop3:0x7e\n\t"
    "v_or3_b32 v49, v54, v55, v49\n\t"
    "v_bitop3_b32 v54, v136, v137, v48 bitop3:0x7e\n\t"
    "v_mfma_f32_16x16x32_bf16 v[92:95], v[208:211], a[180:183], v[92:95]\n\t"
    "ds_read_b64_tr_b16 v[200:201], v43 offset:36864\n\t"
    "s_waitcnt lgkmcnt(7)\n\t"
    "v_bitop3_b32 v55, v138, v139, v48 bitop3:0x7e\n\t"
    "v_or3_b32 v49, v54, v55, v49\n\t"
    "v_bitop3_b32 v54, v140, v141, v48 bitop3:0x7e\n\t"
    "v_mfma_f32_16x16x32_bf16 v[80:83], v[212:215], a[184:187], v[80:83]\n\t"
    "ds_read_b64_tr_b16 v[202:203], v43 offset:37376\n\t"
    "v_bitop3_b32 v55, v142, v143, v48 bitop3:0x7e\n\t"
    "v_or3_b32 v49, v54, v55, v49\n\t"
    "v_bitop3_b32 v54, v144, v145, v48 bitop3:0x7e\n\t"
    "v_bitop3_b32 v55, v146, v147, v48 bitop3:0x7e\n\t"
    "v_mfma_f32_16x16x32_bf16 v[84:87], v[212:215], a[188:191], v[84:87]\n\t"
    "ds_read_b64_tr_b16 v[204:205], v42 offset:40960\n\t"
    "v_or3_b32 v49, v54, v55, v49\n\t"
    "v_bitop3_b32 v54, v148, v149, v48 bitop3:0x7e\n\t"
    "v_bitop3_b32 v55, v150, v151, v48 bitop3:0x7e\n\t"
    "v_or3_b32 v49, v54, v55, v49\n\t"
    "v_mfma_f32_16x16x32_bf16 v[88:91], v[216:219], a[184:187], v[88:91]\n\t"
    "ds_read_b64_tr_b16 v[206:207], v42 offset:41472\n\t"
    "v_and_b32 v49, 63, v49\n\t"
    "v_cmp_ne_u32 vcc, 0, v49\n\t"
    "s_nop 0\n\t"
    "s_cbranch_vccnz LS2%=\n\t"
    "LB2%=:\n\t"
    "v_add_f32 v96, v96, v128\n\t"
    "v_mfma_f32_16x16x32_bf16 v[92:95], v[216:219], a[188:191], v[92:95]\n\t"
    "s_cmp_lt_u32 s60, 10000\n\t"
    "s_cbranch_scc0 LP864%=\n\t"
    "s_sleep 13\n\t"
    "LP864%=:\n\t"
    "ds_read_b64_tr_b16 v[208:209], v43 offset:40960\n\t"
    "s_waitcnt lgkmcnt(7)\n\t"
    "v_add_f32 v97, v97, v129\n\t"
    "v_add_f32 v98, v98, v130\n\t"
    "v_add_f32 v99, v99, v131\n\t"
    "v_mfma_f32_16x16x32_bf16 v[80:83], v[188:191], a[192:195], v[80:83]\n\t"
    "ds_read_b64_tr_b16 v[210:211], v43 offset:41472\n\t"
    "v_add_f32 v100, v100, v132\n\t"
    "v_add_f32 v101, v101, v133\n\t"
    "v_add_f32 v102, v102, v134\n\t"
    "v_add_f32 v103, v103, v135\n\t"
    "v_mfma_f32_16x16x32_bf16 v[84:87], v[188:191], a[196:199], v[84:87]\n\t"
    "ds_read_b64_tr_b16 v[212:213], v42 offset:45056\n\t"
    "v_add_f32 v96, v96, v136\n\t"
    "v_add_f32 v97, v97, v137\n\t"
    "v_add_f32 v98, v98, v138\n\t"
    "v_add_f32 v99, v99, v139\n\t"
    "v_mfma_f32_16x16x32_bf16 v[88:91], v[192:195], a[192:195], v[88:91]\n\t"
    "ds_read_b64_tr_b16 v[214:215], v42 offset:45568\n\t"
    "v_add_f32 v100, v100, v140\n\t"
    "v_add_f32 v101, v101, v141\n\t"
    "v_add_f32 v102, v102, v142\n\t"
    "v_add_f32 v103, v103, v143\n\t"
    "v_mfma_f32_16x16x32_bf16 v[92:95], v[192:195], a[196:199], v[92:95]\n\t"
    "s_waitcnt vmcnt(0)\n\t"
    "s_barrier\n\t"
    "ds_read_b64_tr_b16 v[216:217], v43 offset:45056\n\t"
    "s_waitcnt lgkmcnt(7)\n\t"
    "v_add_f32 v96, v96, v144\n\t"
    "v_add_f32 v97, v97, v145\n\t"
    "v_add_f32 v98, v98, v146\n\t"
    "v_mfma_f32_16x16x32_bf16 v[80:83], v[196:199], a[200:203], v[80:83]\n\t"
    "ds_read_b64_tr_b16 v[218:219], v43 offset:45568\n\t"
    "v_add_f32 v99, v99, v147\n\t"
    "v_add_f32 v100, v100, v148\n\t"
    "v_add_f32 v101, v101, v149\n\t"
    "v_add_f32 v102, v102, v150\n\t"
    "v_mfma_f32_16x16x32_bf16 v[84:87], v[196:199], a[204:207], v[84:87]\n\t"
    "ds_read_b64_tr_b16 v[188:189], v22 offset:16384\n\t"
    "v_add_f32 v103, v103, v151\n\t"
    "v_mul_f32 v152, v96, v96\n\t"
    "v_mul_f32 v153, v97, v97\n\t"
    "v_mul_f32 v154, v98, v98\n\t"
    "v_mfma_f32_16x16x32_bf16 v[88:91], v[200:203], a[200:203], v[88:91]\n\t"
    "ds_read_b64_tr_b16 v[190:191], v22 offset:16896\n\t"
    "v_mul_f32 v155, v99, v99\n\t"
    "v_mul_f32 v156, v100, v100\n\t"
    "v_mul_f32 v157, v101, v101\n\t"
    "v_mul_f32 v158, v102, v102\n\t"
    "v_mfma_f32_16x16x32_bf16 v[92:95], v[200:203], a[204:207], v[92:95]\n\t"
    "ds_read_b64_tr_b16 v[192:193], v23 offset:16384\n\t"
    "s_waitcnt lgkmcnt(7)\n\t"
    "v_mul_f32 v159, v103, v103\n\t"
    "v_fma_f32 v152, v152, v27, v26\n\t"
    "v_fma_f32 v153, v153, v27, v26\n\t"
    "v_mfma_f32_16x16x32_bf16 v[80:83], v[204:207], a[208:211], v[80:83]\n\t"
    "ds_read_b64_tr_b16 v[194:195], v23 offset:16896\n\t"
    "v_fma_f32 v154, v154, v27, v26\n\t"
    "v_fma_f32 v155, v155, v27, v26\n\t"
    "v_fma_f32 v156, v156, v27, v26\n\t"
    "v_fma_f32 v157, v157, v27, v26\n\t"
    "v_mfma_f32_16x16x32_bf16 v[84:87], v[204:207], a[212:215], v[84:87]\n\t"
    "ds_read_b64_tr_b16 v[196:197], v22 offset:20480\n\t"
    "v_fma_f32 v158, v158, v27, v26\n\t"
    "v_fma_f32 v159, v159, v27, v26\n\t"
    "v_mul_f32 v152, v152, v96\n\t"
    "v_mul_f32 v153, v153, v97\n\t"
    "v_mfma_f32_16x16x32_bf16 v[88:91], v[208:211], a[208:211], v[88:91]\n\t"
    "ds_read_b64_tr_b16 v[198:199], v22 offset:20992\n\t"
    "v_mul_f32 v154, v154, v98\n\t"
    "v_mul_f32 v155, v155, v99\n\t"
    "v_mul_f32 v156, v156, v100\n\t"
    "v_mul_f32 v157, v157, v101\n\t"
    "v_mfma_f32_16x16x32_bf16 v[92:95], v[208:211], a[212:215], v[92:95]\n\t"
    "ds_read_b64_tr_b16 v[200:201], v23 offset:20480\n\t"
    "s_waitcnt lgkmcnt(7)\n\t"
    "v_mul_f32 v158, v158, v102\n\t"
    "v_mul_f32 v159, v159, v103\n\t"
    "v_exp_f32 v152, v152\n\t"
    "v_mfma_f32_16x16x32_bf16 v[80:83], v[212:215], a[216:219], v[80:83]\n\t"
    "ds_read_b64_tr_b16 v[202:203], v23 offset:20992\n\t"
    "v_exp_f32 v153, v153\n\t"
    "v_exp_f32 v154, v154\n\t"
    "v_exp_f32 v155, v155\n\t"
    "v_exp_f32 v156, v156\n\t"
    "v_mfma_f32_16x16x32_bf16 v[84:87], v[212:215], a[220:223], v[84:87]\n\t"
    "ds_read_b64_tr_b16 v[204:205], v22 offset:24576\n\t"
    "v_exp_f32 v157, v157\n\t"
    "v_exp_f32 v158, v158\n\t"
    "v_exp_f32 v159, v159\n\t"
    "v_add_f32 v152, 1.0, v152\n\t"
    "v_mfma_f32_16x16x32_bf16 v[88:91], v[216:219], a[216:219], v[88:91]\n\t"
    "ds_read_b64_tr_b16 v[206:207], v22 offset:25088\n\t"
    "v_add_f32 v153, 1.0, v153\n\t"
    "v_add_f32 v154, 1.0, v154\n\t"
    "v_add_f32 v155, 1.0, v155\n\t"
    "v_add_f32 v156, 1.0, v156\n\t"
    "v_mfma_f32_16x16x32_bf16 v[92:95], v[216:219], a[220:223], v[92:95]\n\t"
    "ds_read_b64_tr_b16 v[208:209], v23 offset:24576\n\t"
    "s_waitcnt lgkmcnt(7)\n\t"
    "v_add_f32 v157, 1.0, v157\n\t"
    "v_add_f32 v158, 1.0, v158\n\t"
    "v_add_f32 v159, 1.0, v159\n\t"
    "v_mfma_f32_16x16x32_bf16 v[80:83], v[188:191], a[224:227], v[80:83]\n\t"
    "ds_read_b64_tr_b16 v[210:211], v23 offset:25088\n\t"
    "v_rcp_f32 v152, v152\n\t"
    "v_rcp_f32 v153, v153\n\t"
    "v_rcp_f32 v154, v154\n\t"
    "v_rcp_f32 v155, v155\n\t"
    "v_mfma_f32_16x16x32_bf16 v[84:87], v[188:191], a[228:231], v[84:87]\n\t"
    "ds_read_b64_tr_b16 v[212:213], v22 offset:28672\n\t"
    "v_rcp_f32 v156, v156\n\t"
    "v_rcp_f32 v157, v157\n\t"
    "v_rcp_f32 v158, v158\n\t"
    "v_rcp_f32 v159, v159\n\t"
    "v_mfma_f32_16x16x32_bf16 v[88:91], v[192:195], a[224:227], v[88:91]\n\t"
    "ds_read_b64_tr_b16 v[214:215], v22 offset:29184\n\t"
    "v_mul_f32 v152, v152, v96\n\t"
    "v_mul_f32 v153, v153, v97\n\t"
    "v_mul_f32 v154, v154, v98\n\t"
    "v_mul_f32 v155, v155, v99\n\t"
    "v_mfma_f32_16x16x32_bf16 v[92:95], v[192:195], a[228:231], v[92:95]\n\t"
    "ds_read_b64_tr_b16 v[216:217], v23 offset:28672\n\t"
    "s_waitcnt lgkmcnt(7)\n\t"
    "v_mul_f32 v156, v156, v100\n\t"
    "v_mul_f32 v157, v157, v101\n\t"
    "v_mul_f32 v158, v158, v102\n\t"
    "v_mfma_f32_16x16x32_bf16 v[80:83], v[196:199], a[232:235], v[80:83]\n\t"
    "ds_read_b64_tr_b16 v[218:219], v23 offset:29184\n\t"
    "v_mul_f32 v159, v159, v103\n\t"
    "v_cvt_pk_bf16_f32 v160, v152, v153\n\t"
    "v_cvt_pk_bf16_f32 v161, v154, v155\n\t"
    "v_cvt_pk_bf16_f32 v162, v156, v157\n\t"
    "v_mfma_f32_16x16x32_bf16 v[84:87], v[196:199], a[236:239], v[84:87]\n\t"
    "ds_read_b64_tr_b16 v[188:189], v40 offset:49152\n\t"
    "v_cvt_pk_bf16_f32 v163, v158, v159\n\t"
    "s_nop 1\n\t"
    "v_permlane16_swap_b32 v160, v162\n\t"
    "v_permlane16_swap_b32 v161, v163\n\t"
    "global_store_dwordx4 v25, v[160:163], s[52:53] offset:512 sc0 sc1\n\t"
    "v_mfma_f32_16x16x32_bf16 v[88:91], v[200:203], a[232:235], v[88:91]\n\t"
    "ds_read_b64_tr_b16 v[190:191], v40 offset:49664\n\t"
    "v_mfma_f32_16x16x32_bf16 v[92:95], v[200:203], a[236:239], v[92:95]\n\t"
    "ds_read_b64_tr_b16 v[192:193], v41 offset:49152\n\t"
    "s_waitcnt lgkmcnt(7)\n\t"
    "v_mfma_f32_16x16x32_bf16 v[80:83], v[204:207], a[240:243], v[80:83]\n\t"
    "ds_read_b64_tr_b16 v[194:195], v41 offset:49664\n\t"
    "v_mfma_f32_16x16x32_bf16 v[84:87], v[204:207], a[244:247], v[84:87]\n\t"
    "ds_read_b64_tr_b16 v[196:197], v40 offset:53248\n\t"
    "v_mfma_f32_16x16x32_bf16 v[88:91], v[208:211], a[240:243], v[88:91]\n\t"
    "ds_read_b64_tr_b16 v[198:199], v40 offset:53760\n\t"
    "v_mfma_f32_16x16x32_bf16 v[92:95], v[208:211], a[244:247], v[92:95]\n\t"
    "ds_read_b64_tr_b16 v[200:201], v41 offset:53248\n\t"
    "s_waitcnt lgkmcnt(7)\n\t"
    "v_mfma_f32_16x16x32_bf16 v[80:83], v[212:215], a[248:251], v[80:83]\n\t"
    "ds_read_b64_tr_b16 v[202:203], v41 offset:53760\n\t"
    "v_mfma_f32_16x16x32_bf16 v[84:87], v[212:215], a[252:255], v[84:87]\n\t"
    "ds_read_b64_tr_b16 v[204:205], v40 offset:57344\n\t"
    "v_mfma_f32_16x16x32_bf16 v[88:91], v[216:219], a[248:251], v[88:91]\n\t"
    "ds_read_b64_tr_b16 v[206:207], v40 offset:57856\n\t"
    "v_mfma_f32_16x16x32_bf16 v[92:95], v[216:219], a[252:255], v[92:95]\n\t"
    "s_cmp_lt_u32 s60, 10000\n\t"
    "s_cbranch_scc0 LP896%=\n\t"
    "s_sleep 13\n\t"
    "LP896%=:\n\t"
    "ds_read_b64_tr_b16 v[208:209], v41 offset:57344\n\t"
    "s_waitcnt lgkmcnt(7)\n\t"
    "v_mfma_f32_16x16x32_bf16 v[64:67], v[188:191], a[0:3], 0\n\t"
    "ds_read_b64_tr_b16 v[210:211], v41 offset:57856\n\t"
    "v_mfma_f32_16x16x32_bf16 v[68:71], v[188:191], a[4:7], 0\n\t"
    "ds_read_b64_tr_b16 v[212:213], v40 offset:61440\n\t"
    "v_mfma_f32_16x16x32_bf16 v[72:75], v[192:195], a[0:3], 0\n\t"
    "ds_read_b64_tr_b16 v[214:215], v40 offset:61952\n\t"
    "v_mfma_f32_16x16x32_bf16 v[76:79], v[192:195], a[4:7], 0\n\t"
    "ds_read_b64_tr_b16 v[216:217], v41 offset:61440\n\t"
    "s_waitcnt lgkmcnt(7)\n\t"
    "v_bfi_b32 v104, 63, v48, v80\n\t"
    "v_bfi_b32 v105, 63, v48, v81\n\t"
    "v_bfi_b32 v106, 63, v48, v82\n\t"
    "v_mfma_f32_16x16x32_bf16 v[64:67], v[196:199], a[8:11], v[64:67]\n\t"
    "ds_read_b64_tr_b16 v[218:219], v41 offset:61952\n\t"
    "v_bfi_b32 v107, 63, v48, v83\n\t"
    "v_bfi_b32 v120, 63, v48, v84\n\t"
    "v_bfi_b32 v121, 63, v48, v85\n\t"
    "v_mfma_f32_16x16x32_bf16 v[68:71], v[196:199], a[12:15], v[68:71]\n\t"
    "buffer_store_dwordx4 v[104:107], v52, s[44:47], s74 offen offset:0 sc0 sc1\n\t"
    "ds_read_b64_tr_b16 v[188:189], v44 offset:0\n\t"
    "v_bfi_b32 v122, 63, v48, v86\n\t"
    "v_bfi_b32 v123, 63, v48, v87\n\t"
    "v_bfi_b32 v108, 63, v48, v88\n\t"
    "v_mfma_f32_16x16x32_bf16 v[72:75], v[200:203], a[8:11], v[72:75]\n\t"
    "ds_read_b64_tr_b16 v[190:191], v44 offset:512\n\t"
    "v_bfi_b32 v109, 63, v48, v89\n\t"
    "v_bfi_b32 v110, 63, v48, v90\n\t"
    "v_bfi_b32 v111, 63, v48, v91\n\t"
    "v_mfma_f32_16x16x32_bf16 v[76:79], v[200:203], a[12:15], v[76:79]\n\t"
    "ds_read_b64_tr_b16 v[192:193], v45 offset:0\n\t"
    "s_waitcnt lgkmcnt(7)\n\t"
    "v_bfi_b32 v124, 63, v48, v92\n\t"
    "v_bfi_b32 v125, 63, v48, v93\n\t"
    "v_bfi_b32 v126, 63, v48, v94\n\t"
    "v_mfma_f32_16x16x32_bf16 v[64:67], v[204:207], a[16:19], v[64:67]\n\t"
    "buffer_store_dwordx4 v[120:123], v52, s[44:47], s76 offen offset:0 sc0 sc1\n\t"
    "ds_read_b64_tr_b16 v[194:195], v45 offset:512\n\t"
    "v_bfi_b32 v127, 63, v48, v95\n\t"
    "v_mfma_f32_16x16x32_bf16 v[68:71], v[204:207], a[20:23], v[68:71]\n\t"
    "ds_read_b64_tr_b16 v[196:197], v44 offset:4096\n\t"
    "v_mfma_f32_16x16x32_bf16 v[72:75], v[208:211], a[16:19], v[72:75]\n\t"
    "ds_read_b64_tr_b16 v[198:199], v44 offset:4608\n\t"
    "v_mfma_f32_16x16x32_bf16 v[76:79], v[208:211], a[20:23], v[76:79]\n\t"
    "buffer_store_dwordx4 v[108:111], v52, s[44:47], s74 offen offset:1024 sc0 sc1\n\t"
    "ds_read_b64_tr_b16 v[200:201], v45 offset:4096\n\t"
    "s_waitcnt lgkmcnt(7)\n\t"
    "v_mfma_f32_16x16x32_bf16 v[64:67], v[212:215], a[24:27], v[64:67]\n\t"
    "ds_read_b64_tr_b16 v[202:203], v45 offset:4608\n\t"
    "v_mfma_f32_16x16x32_bf16 v[68:71], v[212:215], a[28:31], v[68:71]\n\t"
    "ds_read_b64_tr_b16 v[204:205], v44 offset:8192\n\t"
    "v_mfma_f32_16x16x32_bf16 v[72:75], v[216:219], a[24:27], v[72:75]\n\t"
    "buffer_store_dwordx4 v[124:127], v52, s[44:47], s76 offen offset:1024 sc0 sc1\n\t"
    "ds_read_b64_tr_b16 v[206:207], v44 offset:8704\n\t"
    "v_mfma_f32_16x16x32_bf16 v[76:79], v[216:219], a[28:31], v[76:79]\n\t"
    "ds_read_b64_tr_b16 v[208:209], v45 offset:8192\n\t"
    "s_waitcnt lgkmcnt(7)\n\t"
    "v_mfma_f32_16x16x32_bf16 v[64:67], v[188:191], a[32:35], v[64:67]\n\t"
    "ds_read_b64_tr_b16 v[210:211], v45 offset:8704\n\t"
    "v_mfma_f32_16x16x32_bf16 v[68:71], v[188:191], a[36:39], v[68:71]\n\t"
    "ds_read_b64_tr_b16 v[212:213], v44 offset:12288\n\t"
    "v_mfma_f32_16x16x32_bf16 v[72:75], v[192:195], a[32:35], v[72:75]\n\t"
    "ds_read_b64_tr_b16 v[214:215], v44 offset:12800\n\t"
    "v_mfma_f32_16x16x32_bf16 v[76:79], v[192:195], a[36:39], v[76:79]\n\t"
    "ds_read_b64_tr_b16 v[216:217], v45 offset:12288\n\t"
    "s_waitcnt lgkmcnt(7)\n\t"
    "v_mfma_f32_16x16x32_bf16 v[64:67], v[196:199], a[40:43], v[64:67]\n\t"
    "ds_read_b64_tr_b16 v[218:219], v45 offset:12800\n\t"
    "v_mfma_f32_16x16x32_bf16 v[68:71], v[196:199], a[44:47], v[68:71]\n\t"
    "ds_read_b64_tr_b16 v[188:189], v44 offset:16384\n\t"
    "v_mfma_f32_16x16x32_bf16 v[72:75], v[200:203], a[40:43], v[72:75]\n\t"
    "ds_read_b64_tr_b16 v[190:191], v44 offset:16896\n\t"
    "v_mfma_f32_16x16x32_bf16 v[76:79], v[200:203], a[44:47], v[76:79]\n\t"
    "ds_read_b64_tr_b16 v[192:193], v45 offset:16384\n\t"
    "s_waitcnt lgkmcnt(7)\n\t"
    "v_mfma_f32_16x16x32_bf16 v[64:67], v[204:207], a[48:51], v[64:67]\n\t"
    "ds_read_b64_tr_b16 v[194:195], v45 offset:16896\n\t"
    "v_mfma_f32_16x16x32_bf16 v[68:71], v[204:207], a[52:55], v[68:71]\n\t"
    "ds_read_b64_tr_b16 v[196:197], v44 offset:20480\n\t"
    "v_mfma_f32_16x16x32_bf16 v[72:75], v[208:211], a[48:51], v[72:75]\n\t"
    "ds_read_b64_tr_b16 v[198:199], v44 offset:20992\n\t"
    "v_mfma_f32_16x16x32_bf16 v[76:79], v[208:211], a[52:55], v[76:79]\n\t"
    "ds_read_b64_tr_b16 v[200:201], v45 offset:20480\n\t"
    "s_waitcnt lgkmcnt(7)\n\t"
    "v_mfma_f32_16x16x32_bf16 v[64:67], v[212:215], a[56:59], v[64:67]\n\t"
    "ds_read_b64_tr_b16 v[202:203], v45 offset:20992\n\t"
    "v_mfma_f32_16x16x32_bf16 v[68:71], v[212:215], a[60:63], v[68:71]\n\t"
    "ds_read_b64_tr_b16 v[204:205], v44 offset:24576\n\t"
    "v_mfma_f32_16x16x32_bf16 v[72:75], v[216:219], a[56:59], v[72:75]\n\t"
    "ds_read_b64_tr_b16 v[206:207], v44 offset:25088\n\t"
    "v_mfma_f32_16x16x32_bf16 v[76:79], v[216:219], a[60:63], v[76:79]\n\t"
    "s_cmp_lt_u32 s60, 10000\n\t"
    "s_cbranch_scc0 LP928%=\n\t"
    "s_sleep 13\n\t"
    "LP928%=:\n\t"
    "ds_read_b64_tr_b16 v[208:209], v45 offset:24576\n\t"
    "ds_read_b64_tr_b16 v[210:211], v45 offset:25088\n\t"
    "s_waitcnt lgkmcnt(8)\n\t"
    "v_mfma_f32_16x16x32_bf16 v[68:71], v[188:191], a[68:71], v[68:71]\n\t"
    "ds_read_b64_tr_b16 v[212:213], v44 offset:28672\n\t"
    "ds_read_b64_tr_b16 v[214:215], v44 offset:29184\n\t"
    "v_mfma_f32_16x16x32_bf16 v[76:79], v[192:195], a[68:71], v[76:79]\n\t"
    "ds_read_b64_tr_b16 v[216:217], v45 offset:28672\n\t"
    "ds_read_b64_tr_b16 v[218:219], v45 offset:29184\n\t"
    "s_waitcnt lgkmcnt(8)\n\t"
    "v_mfma_f32_16x16x32_bf16 v[68:71], v[196:199], a[76:79], v[68:71]\n\t"
    "ds_read_b64_tr_b16 v[188:189], v20 offset:0\n\t"
    "ds_read_b64_tr_b16 v[190:191], v20 offset:512\n\t"
    "v_mfma_f32_16x16x32_bf16 v[76:79], v[200:203], a[76:79], v[76:79]\n\t"
    "ds_read_b64_tr_b16 v[192:193], v21 offset:0\n\t"
    "ds_read_b64_tr_b16 v[194:195], v21 offset:512\n\t"
    "s_waitcnt lgkmcnt(8)\n\t"
    "v_mfma_f32_16x16x32_bf16 v[68:71], v[204:207], a[84:87], v[68:71]\n\t"
    "ds_read_b64_tr_b16 v[196:197], v20 offset:4096\n\t"
    "ds_read_b64_tr_b16 v[198:199], v20 offset:4608\n\t"
    "v_mfma_f32_16x16x32_bf16 v[76:79], v[208:211], a[84:87], v[76:79]\n\t"
    "ds_read_b64_tr_b16 v[200:201], v21 offset:4096\n\t"
    "ds_read_b64_tr_b16 v[202:203], v21 offset:4608\n\t"
    "s_waitcnt lgkmcnt(8)\n\t"
    "v_mfma_f32_16x16x32_bf16 v[68:71], v[212:215], a[92:95], v[68:71]\n\t"
    "ds_read_b64_tr_b16 v[204:205], v20 offset:8192\n\t"
    "ds_read_b64_tr_b16 v[206:207], v20 offset:8704\n\t"
    "v_mfma_f32_16x16x32_bf16 v[76:79], v[216:219], a[92:95], v[76:79]\n\t"
    "ds_read_b64_tr_b16 v[208:209], v21 offset:8192\n\t"
    "ds_read_b64_tr_b16 v[210:211], v21 offset:8704\n\t"
    "s_waitcnt lgkmcnt(8)\n\t"
    "v_mfma_f32_16x16x32_bf16 v[68:71], v[188:191], a[100:103], v[68:71]\n\t"
    "ds_read_b64_tr_b16 v[212:213], v20 offset:12288\n\t"
    "ds_read_b64_tr_b16 v[214:215], v20 offset:12800\n\t"
    "v_mfma_f32_16x16x32_bf16 v[76:79], v[192:195], a[100:103], v[76:79]\n\t"
    "ds_read_b64_tr_b16 v[216:217], v21 offset:12288\n\t"
    "ds_read_b64_tr_b16 v[218:219], v21 offset:12800\n\t"
    "s_waitcnt lgkmcnt(8)\n\t"
    "v_mfma_f32_16x16x32_bf16 v[68:71], v[196:199], a[108:111], v[68:71]\n\t"
    "ds_read_b64_tr_b16 v[188:189], v40 offset:16384\n\t"
    "ds_read_b64_tr_b16 v[190:191], v40 offset:16896\n\t"
    "v_mfma_f32_16x16x32_bf16 v[76:79], v[200:203], a[108:111], v[76:79]\n\t"
    "ds_read_b64_tr_b16 v[192:193], v41 offset:16384\n\t"
    "ds_read_b64_tr_b16 v[194:195], v41 offset:16896\n\t"
    "s_waitcnt lgkmcnt(8)\n\t"
    "v_mfma_f32_16x16x32_bf16 v[68:71], v[204:207], a[116:119], v[68:71]\n\t"
    "ds_read_b64_tr_b16 v[196:197], v40 offset:20480\n\t"
    "ds_read_b64_tr_b16 v[198:199], v40 offset:20992\n\t"
    "v_mfma_f32_16x16x32_bf16 v[76:79], v[208:211], a[116:119], v[76:79]\n\t"
    "ds_read_b64_tr_b16 v[200:201], v41 offset:20480\n\t"
    "ds_read_b64_tr_b16 v[202:203], v41 offset:20992\n\t"
    "s_waitcnt lgkmcnt(8)\n\t"
    "v_mfma_f32_16x16x32_bf16 v[68:71], v[212:215], a[124:127], v[68:71]\n\t"
    "ds_read_b64_tr_b16 v[204:205], v40 offset:24576\n\t"
    "ds_read_b64_tr_b16 v[206:207], v40 offset:25088\n\t"
    "v_mfma_f32_16x16x32_bf16 v[76:79], v[216:219], a[124:127], v[76:79]\n\t"
    "ds_read_b64_tr_b16 v[208:209], v41 offset:24576\n\t"
    "ds_read_b64_tr_b16 v[210:211], v41 offset:25088\n\t"
    "s_waitcnt lgkmcnt(8)\n\t"
    "v_mfma_f32_16x16x32_bf16 v[68:71], v[188:191], a[132:135], v[68:71]\n\t"
    "ds_read_b64_tr_b16 v[212:213], v40 offset:28672\n\t"
    "ds_read_b64_tr_b16 v[214:215], v40 offset:29184\n\t"
    "v_mfma_f32_16x16x32_bf16 v[76:79], v[192:195], a[132:135], v[76:79]\n\t"
    "ds_read_b64_tr_b16 v[216:217], v41 offset:28672\n\t"
    "ds_read_b64_tr_b16 v[218:219], v41 offset:29184\n\t"
    "s_waitcnt lgkmcnt(8)\n\t"
    "v_mfma_f32_16x16x32_bf16 v[68:71], v[196:199], a[140:143], v[68:71]\n\t"
    "ds_read_b64_tr_b16 v[188:189], v20 offset:49152\n\t"
    "ds_read_b64_tr_b16 v[190:191], v20 offset:49664\n\t"
    "v_mfma_f32_16x16x32_bf16 v[76:79], v[200:203], a[140:143], v[76:79]\n\t"
    "ds_read_b64_tr_b16 v[192:193], v21 offset:49152\n\t"
    "ds_read_b64_tr_b16 v[194:195], v21 offset:49664\n\t"
    "s_waitcnt lgkmcnt(8)\n\t"
    "v_mfma_f32_16x16x32_bf16 v[68:71], v[204:207], a[148:151], v[68:71]\n\t"
    "ds_read_b64_tr_b16 v[196:197], v20 offset:53248\n\t"
    "ds_read_b64_tr_b16 v[198:199], v20 offset:53760\n\t"
    "v_mfma_f32_16x16x32_bf16 v[76:79], v[208:211], a[148:151], v[76:79]\n\t"
    "ds_read_b64_tr_b16 v[200:201], v21 offset:53248\n\t"
    "ds_read_b64_tr_b16 v[202:203], v21 offset:53760\n\t"
    "s_waitcnt lgkmcnt(8)\n\t"
    "v_mfma_f32_16x16x32_bf16 v[68:71], v[212:215], a[156:159], v[68:71]\n\t"
    "ds_read_b64_tr_b16 v[204:205], v20 offset:57344\n\t"
    "ds_read_b64_tr_b16 v[206:207], v20 offset:57856\n\t"
    "v_mfma_f32_16x16x32_bf16 v[76:79], v[216:219], a[156:159], v[76:79]\n\t"
    "ds_read_b64_tr_b16 v[208:209], v21 offset:57344\n\t"
    "ds_read_b64_tr_b16 v[210:211], v21 offset:57856\n\t"
    "s_waitcnt lgkmcnt(8)\n\t"
    "v_mfma_f32_16x16x32_bf16 v[68:71], v[188:191], a[164:167], v[68:71]\n\t"
    "ds_read_b64_tr_b16 v[212:213], v20 offset:61440\n\t"
    "ds_read_b64_tr_b16 v[214:215], v20 offset:61952\n\t"
    "v_mfma_f32_16x16x32_bf16 v[76:79], v[192:195], a[164:167], v[76:79]\n\t"
    "ds_read_b64_tr_b16 v[216:217], v21 offset:61440\n\t"
    "ds_read_b64_tr_b16 v[218:219], v21 offset:61952\n\t"
    "s_waitcnt lgkmcnt(8)\n\t"
    "v_mfma_f32_16x16x32_bf16 v[68:71], v[196:199], a[172:175], v[68:71]\n\t"
    "ds_read_b64_tr_b16 v[188:189], v40 offset:32768\n\t"
    "ds_read_b64_tr_b16 v[190:191], v40 offset:33280\n\t"
    "v_mfma_f32_16x16x32_bf16 v[76:79], v[200:203], a[172:175], v[76:79]\n\t"
    "ds_read_b64_tr_b16 v[192:193], v41 offset:32768\n\t"
    "ds_read_b64_tr_b16 v[194:195], v41 offset:33280\n\t"
    "s_waitcnt lgkmcnt(8)\n\t"
    "v_mfma_f32_16x16x32_bf16 v[68:71], v[204:207], a[180:183], v[68:71]\n\t"
    "ds_read_b64_tr_b16 v[196:197], v40 offset:36864\n\t"
    "ds_read_b64_tr_b16 v[198:199], v40 offset:37376\n\t"
    "v_mfma_f32_16x16x32_bf16 v[76:79], v[208:211], a[180:183], v[76:79]\n\t"
    "ds_read_b64_tr_b16 v[200:201], v41 offset:36864\n\t"
    "ds_read_b64_tr_b16 v[202:203], v41 offset:37376\n\t"
    "s_waitcnt lgkmcnt(8)\n\t"
    "v_mfma_f32_16x16x32_bf16 v[68:71], v[212:215], a[188:191], v[68:71]\n\t"
    "ds_read_b64_tr_b16 v[204:205], v40 offset:40960\n\t"
    "ds_read_b64_tr_b16 v[206:207], v40 offset:41472\n\t"
    "v_mfma_f32_16x16x32_bf16 v[76:79], v[216:219], a[188:191], v[76:79]\n\t"
    "s_cmp_lt_u32 s60, 10000\n\t"
    "s_cbranch_scc0 LP960%=\n\t"
    "s_sleep 13\n\t"
    "LP960%=:\n\t"
    "ds_read_b64_tr_b16 v[208:209], v41 offset:40960\n\t"
    "ds_read_b64_tr_b16 v[210:211], v41 offset:41472\n\t"
    "s_waitcnt lgkmcnt(8)\n\t"
    "v_mfma_f32_16x16x32_bf16 v[68:71], v[188:191], a[196:199], v[68:71]\n\t"
    "ds_read_b64_tr_b16 v[212:213], v40 offset:45056\n\t"
    "ds_read_b64_tr_b16 v[214:215], v40 offset:45568\n\t"
    "v_mfma_f32_16x16x32_bf16 v[76:79], v[192:195], a[196:199], v[76:79]\n\t"
    "ds_read_b64_tr_b16 v[216:217], v41 offset:45056\n\t"
    "ds_read_b64_tr_b16 v[218:219], v41 offset:45568\n\t"
    "s_waitcnt lgkmcnt(8)\n\t"
    "v_mfma_f32_16x16x32_bf16 v[68:71], v[196:199], a[204:207], v[68:71]\n\t"
    "ds_read_b64_tr_b16 v[188:189], v20 offset:16384\n\t"
    "ds_read_b64_tr_b16 v[190:191], v20 offset:16896\n\t"
    "v_mfma_f32_16x16x32_bf16 v[76:79], v[200:203], a[204:207], v[76:79]\n\t"
    "ds_read_b64_tr_b16 v[192:193], v21 offset:16384\n\t"
    "ds_read_b64_tr_b16 v[194:195], v21 offset:16896\n\t"
    "s_waitcnt lgkmcnt(8)\n\t"
    "v_mfma_f32_16x16x32_bf16 v[68:71], v[204:207], a[212:215], v[68:71]\n\t"
    "ds_read_b64_tr_b16 v[196:197], v20 offset:20480\n\t"
    "ds_read_b64_tr_b16 v[198:199], v20 offset:20992\n\t"
    "v_mfma_f32_16x16x32_bf16 v[76:79], v[208:211], a[212:215], v[76:79]\n\t"
    "ds_read_b64_tr_b16 v[200:201], v21 offset:20480\n\t"
    "ds_read_b64_tr_b16 v[202:203], v21 offset:20992\n\t"
    "s_waitcnt lgkmcnt(8)\n\t"
    "v_mfma_f32_16x16x32_bf16 v[68:71], v[212:215], a[220:223], v[68:71]\n\t"
    "ds_read_b64_tr_b16 v[204:205], v20 offset:24576\n\t"
    "ds_read_b64_tr_b16 v[206:207], v20 offset:25088\n\t"
    "v_mfma_f32_16x16x32_bf16 v[76:79], v[216:219], a[220:223], v[76:79]\n\t"
    "ds_read_b64_tr_b16 v[208:209], v21 offset:24576\n\t"
    "ds_read_b64_tr_b16 v[210:211], v21 offset:25088\n\t"
    "s_waitcnt lgkmcnt(8)\n\t"
    "v_mfma_f32_16x16x32_bf16 v[68:71], v[188:191], a[228:231], v[68:71]\n\t"
    "ds_read_b64_tr_b16 v[212:213], v20 offset:28672\n\t"
    "ds_read_b64_tr_b16 v[214:215], v20 offset:29184\n\t"
    "v_mfma_f32_16x16x32_bf16 v[76:79], v[192:195], a[228:231], v[76:79]\n\t"
    "ds_read_b64_tr_b16 v[216:217], v21 offset:28672\n\t"
    "ds_read_b64_tr_b16 v[218:219], v21 offset:29184\n\t"
    "s_waitcnt lgkmcnt(8)\n\t"
    "v_mfma_f32_16x16x32_bf16 v[68:71], v[196:199], a[236:239], v[68:71]\n\t"
    "ds_read_b64_tr_b16 v[188:189], v44 offset:16384\n\t"
    "ds_read_b64_tr_b16 v[190:191], v44 offset:16896\n\t"
    "v_mfma_f32_16x16x32_bf16 v[76:79], v[200:203], a[236:239], v[76:79]\n\t"
    "ds_read_b64_tr_b16 v[192:193], v45 offset:16384\n\t"
    "ds_read_b64_tr_b16 v[194:195], v45 offset:16896\n\t"
    "s_waitcnt lgkmcnt(8)\n\t"
    "v_mfma_f32_16x16x32_bf16 v[68:71], v[204:207], a[244:247], v[68:71]\n\t"
    "ds_read_b64_tr_b16 v[196:197], v44 offset:20480\n\t"
    "ds_read_b64_tr_b16 v[198:199], v44 offset:20992\n\t"
    "v_mfma_f32_16x16x32_bf16 v[76:79], v[208:211], a[244:247], v[76:79]\n\t"
    "ds_read_b64_tr_b16 v[200:201], v45 offset:20480\n\t"
    "ds_read_b64_tr_b16 v[202:203], v45 offset:20992\n\t"
    "s_waitcnt lgkmcnt(8)\n\t"
    "v_mfma_f32_16x16x32_bf16 v[68:71], v[212:215], a[252:255], v[68:71]\n\t"
    "ds_read_b64_tr_b16 v[204:205], v44 offset:24576\n\t"
    "ds_read_b64_tr_b16 v[206:207], v44 offset:25088\n\t"
    "v_mfma_f32_16x16x32_bf16 v[76:79], v[216:219], a[252:255], v[76:79]\n\t"
    "ds_read_b64_tr_b16 v[208:209], v45 offset:24576\n\t"
    "ds_read_b64_tr_b16 v[210:211], v45 offset:25088\n\t"
    "s_waitcnt lgkmcnt(8)\n\t"
    "v_mfma_f32_16x16x32_bf16 v[64:67], v[188:191], a[64:67], v[64:67]\n\t"
    "ds_read_b64_tr_b16 v[212:213], v44 offset:28672\n\t"
    "ds_read_b64_tr_b16 v[214:215], v44 offset:29184\n\t"
    "v_mfma_f32_16x16x32_bf16 v[72:75], v[192:195], a[64:67], v[72:75]\n\t"
    "ds_read_b64_tr_b16 v[216:217], v45 offset:28672\n\t"
    "ds_read_b64_tr_b16 v[218:219], v45 offset:29184\n\t"
    "s_waitcnt lgkmcnt(8)\n\t"
    "v_mfma_f32_16x16x32_bf16 v[64:67], v[196:199], a[72:75], v[64:67]\n\t"
    "ds_read_b64_tr_b16 v[188:189], v20 offset:0\n\t"
    "ds_read_b64_tr_b16 v[190:191], v20 offset:512\n\t"
    "v_mfma_f32_16x16x32_bf16 v[72:75], v[200:203], a[72:75], v[72:75]\n\t"
    "ds_read_b64_tr_b16 v[192:193], v21 offset:0\n\t"
    "ds_read_b64_tr_b16 v[194:195], v21 offset:512\n\t"
    "s_waitcnt lgkmcnt(8)\n\t"
    "v_bfi_b32 v112, 63, v48, v68\n\t"
    "v_bfi_b32 v113, 63, v48, v69\n\t"
    "v_bfi_b32 v114, 63, v48, v70\n\t"
    "v_mfma_f32_16x16x32_bf16 v[64:67], v[204:207], a[80:83], v[64:67]\n\t"
    "ds_read_b64_tr_b16 v[196:197], v20 offset:4096\n\t"
    "ds_read_b64_tr_b16 v[198:199], v20 offset:4608\n\t"
    "v_bfi_b32 v115, 63, v48, v71\n\t"
    "v_bfi_b32 v116, 63, v48, v76\n\t"
    "v_bfi_b32 v117, 63, v48, v77\n\t"
    "v_mfma_f32_16x16x32_bf16 v[72:75], v[208:211], a[80:83], v[72:75]\n\t"
    "buffer_store_dwordx4 v[112:115], v52, s[44:47], s75 offen offset:0 sc0 sc1\n\t"
    "ds_read_b64_tr_b16 v[200:201], v21 offset:4096\n\t"
    "ds_read_b64_tr_b16 v[202:203], v21 offset:4608\n\t"
    "s_waitcnt lgkmcnt(8)\n\t"
    "v_bfi_b32 v118, 63, v48, v78\n\t"
    "v_bfi_b32 v119, 63, v48, v79\n\t"
    "v_mfma_f32_16x16x32_bf16 v[64:67], v[212:215], a[88:91], v[64:67]\n\t"
    "ds_read_b64_tr_b16 v[204:205], v20 offset:8192\n\t"
    "ds_read_b64_tr_b16 v[206:207], v20 offset:8704\n\t"
    "v_mfma_f32_16x16x32_bf16 v[72:75], v[216:219], a[88:91], v[72:75]\n\t"
    "ds_read_b64_tr_b16 v[208:209], v21 offset:8192\n\t"
    "ds_read_b64_tr_b16 v[210:211], v21 offset:8704\n\t"
    "s_waitcnt lgkmcnt(8)\n\t"
    "v_mfma_f32_16x16x32_bf16 v[64:67], v[188:191], a[96:99], v[64:67]\n\t"
    "buffer_store_dwordx4 v[116:119], v52, s[44:47], s75 offen offset:1024 sc0 sc1\n\t"
    "ds_read_b64_tr_b16 v[212:213], v20 offset:12288\n\t"
    "ds_read_b64_tr_b16 v[214:215], v20 offset:12800\n\t"
    "v_mfma_f32_16x16x32_bf16 v[72:75], v[192:195], a[96:99], v[72:75]\n\t"
    "ds_read_b64_tr_b16 v[216:217], v21 offset:12288\n\t"
    "ds_read_b64_tr_b16 v[218:219], v21 offset:12800\n\t"
    "s_waitcnt lgkmcnt(8)\n\t"
    "v_mfma_f32_16x16x32_bf16 v[64:67], v[196:199], a[104:107], v[64:67]\n\t"
    "ds_read_b64_tr_b16 v[188:189], v40 offset:16384\n\t"
    "ds_read_b64_tr_b16 v[190:191], v40 offset:16896\n\t"
    "v_mfma_f32_16x16x32_bf16 v[72:75], v[200:203], a[104:107], v[72:75]\n\t"
    "ds_read_b64_tr_b16 v[192:193], v41 offset:16384\n\t"
    "ds_read_b64_tr_b16 v[194:195], v41 offset:16896\n\t"
    "s_waitcnt lgkmcnt(8)\n\t"
    "v_mfma_f32_16x16x32_bf16 v[64:67], v[204:207], a[112:115], v[64:67]\n\t"
    "ds_read_b64_tr_b16 v[196:197], v40 offset:20480\n\t"
    "ds_read_b64_tr_b16 v[198:199], v40 offset:20992\n\t"
    "v_mfma_f32_16x16x32_bf16 v[72:75], v[208:211], a[112:115], v[72:75]\n\t"
    "ds_read_b64_tr_b16 v[200:201], v41 offset:20480\n\t"
    "ds_read_b64_tr_b16 v[202:203], v41 offset:20992\n\t"
    "s_waitcnt lgkmcnt(8)\n\t"
    "v_mfma_f32_16x16x32_bf16 v[64:67], v[212:215], a[120:123], v[64:67]\n\t"
    "ds_read_b64_tr_b16 v[204:205], v40 offset:24576\n\t"
    "ds_read_b64_tr_b16 v[206:207], v40 offset:25088\n\t"
    "v_mfma_f32_16x16x32_bf16 v[72:75], v[216:219], a[120:123], v[72:75]\n\t"
    "s_cmp_lt_u32 s60, 10000\n\t"
    "s_cbranch_scc0 LP992%=\n\t"
    "s_sleep 13\n\t"
    "LP992%=:\n\t"
    "ds_read_b64_tr_b16 v[208:209], v41 offset:24576\n\t"
    "ds_read_b64_tr_b16 v[210:211], v41 offset:25088\n\t"
    "s_waitcnt lgkmcnt(8)\n\t"
    "v_mfma_f32_16x16x32_bf16 v[64:67], v[188:191], a[128:131], v[64:67]\n\t"
    "ds_read_b64_tr_b16 v[212:213], v40 offset:28672\n\t"
    "ds_read_b64_tr_b16 v[214:215], v40 offset:29184\n\t"
    "v_mfma_f32_16x16x32_bf16 v[72:75], v[192:195], a[128:131], v[72:75]\n\t"
    "ds_read_b64_tr_b16 v[216:217], v41 offset:28672\n\t"
    "ds_read_b64_tr_b16 v[218:219], v41 offset:29184\n\t"
    "s_waitcnt lgkmcnt(8)\n\t"
    "v_mfma_f32_16x16x32_bf16 v[64:67], v[196:199], a[136:139], v[64:67]\n\t"
    "ds_read_b64_tr_b16 v[188:189], v20 offset:49152\n\t"
    "ds_read_b64_tr_b16 v[190:191], v20 offset:49664\n\t"
    "v_mfma_f32_16x16x32_bf16 v[72:75], v[200:203], a[136:139], v[72:75]\n\t"
    "ds_read_b64_tr_b16 v[192:193], v21 offset:49152\n\t"
    "ds_read_b64_tr_b16 v[194:195], v21 offset:49664\n\t"
    "s_waitcnt lgkmcnt(8)\n\t"
    "v_mfma_f32_16x16x32_bf16 v[64:67], v[204:207], a[144:147], v[64:67]\n\t"
    "ds_read_b64_tr_b16 v[196:197], v20 offset:53248\n\t"
    "ds_read_b64_tr_b16 v[198:199], v20 offset:53760\n\t"
    "v_mfma_f32_16x16x32_bf16 v[72:75], v[208:211], a[144:147], v[72:75]\n\t"
    "ds_read_b64_tr_b16 v[200:201], v21 offset:53248\n\t"
    "ds_read_b64_tr_b16 v[202:203], v21 offset:53760\n\t"
    "s_waitcnt lgkmcnt(8)\n\t"
    "v_mfma_f32_16x16x32_bf16 v[64:67], v[212:215], a[152:155], v[64:67]\n\t"
    "ds_read_b64_tr_b16 v[204:205], v20 offset:57344\n\t"
    "ds_read_b64_tr_b16 v[206:207], v20 offset:57856\n\t"
    "v_mfma_f32_16x16x32_bf16 v[72:75], v[216:219], a[152:155], v[72:75]\n\t"
    "ds_read_b64_tr_b16 v[208:209], v21 offset:57344\n\t"
    "ds_read_b64_tr_b16 v[210:211], v21 offset:57856\n\t"
    "s_waitcnt lgkmcnt(8)\n\t"
    "buffer_load_dwordx4 v[164:167], v52, s[44:47], s77 offen offset:0 sc0 sc1\n\t"
    "v_mfma_f32_16x16x32_bf16 v[64:67], v[188:191], a[160:163], v[64:67]\n\t"
    "ds_read_b64_tr_b16 v[212:213], v20 offset:61440\n\t"
    "ds_read_b64_tr_b16 v[214:215], v20 offset:61952\n\t"
    "v_mfma_f32_16x16x32_bf16 v[72:75], v[192:195], a[160:163], v[72:75]\n\t"
    "ds_read_b64_tr_b16 v[216:217], v21 offset:61440\n\t"
    "ds_read_b64_tr_b16 v[218:219], v21 offset:61952\n\t"
    "s_waitcnt lgkmcnt(8)\n\t"
    "v_mfma_f32_16x16x32_bf16 v[64:67], v[196:199], a[168:171], v[64:67]\n\t"
    "ds_read_b64_tr_b16 v[188:189], v40 offset:32768\n\t"
    "ds_read_b64_tr_b16 v[190:191], v40 offset:33280\n\t"
    "buffer_load_dwordx4 v[168:171], v52, s[44:47], s77 offen offset:1024 sc0 sc1\n\t"
    "v_mfma_f32_16x16x32_bf16 v[72:75], v[200:203], a[168:171], v[72:75]\n\t"
    "ds_read_b64_tr_b16 v[192:193], v41 offset:32768\n\t"
    "ds_read_b64_tr_b16 v[194:195], v41 offset:33280\n\t"
    "s_waitcnt lgkmcnt(8)\n\t"
    "v_mfma_f32_16x16x32_bf16 v[64:67], v[204:207], a[176:179], v[64:67]\n\t"
    "ds_read_b64_tr_b16 v[196:197], v40 offset:36864\n\t"
    "ds_read_b64_tr_b16 v[198:199], v40 offset:37376\n\t"
    "v_mfma_f32_16x16x32_bf16 v[72:75], v[208:211], a[176:179], v[72:75]\n\t"
    "ds_read_b64_tr_b16 v[200:201], v41 offset:36864\n\t"
    "ds_read_b64_tr_b16 v[202:203], v41 offset:37376\n\t"
    "s_waitcnt lgkmcnt(8)\n\t"
    "buffer_load_dwordx4 v[180:183], v52, s[44:47], s79 offen offset:0 sc0 sc1\n\t"
    "v_mfma_f32_16x16x32_bf16 v[64:67], v[212:215], a[184:187], v[64:67]\n\t"
    "ds_read_b64_tr_b16 v[204:205], v40 offset:40960\n\t"
    "ds_read_b64_tr_b16 v[206:207], v40 offset:41472\n\t"
    "v_mfma_f32_16x16x32_bf16 v[72:75], v[216:219], a[184:187], v[72:75]\n\t"
    "ds_read_b64_tr_b16 v[208:209], v41 offset:40960\n\t"
    "ds_read_b64_tr_b16 v[210:211], v41 offset:41472\n\t"
    "s_waitcnt lgkmcnt(8)\n\t"
    "v_mfma_f32_16x16x32_bf16 v[64:67], v[188:191], a[192:195], v[64:67]\n\t"
    "ds_read_b64_tr_b16 v[212:213], v40 offset:45056\n\t"
    "ds_read_b64_tr_b16 v[214:215], v40 offset:45568\n\t"
    "buffer_load_dwordx4 v[184:187], v52, s[44:47], s79 offen offset:1024 sc0 sc1\n\t"
    "v_mfma_f32_16x16x32_bf16 v[72:75], v[192:195], a[192:195], v[72:75]\n\t"
    "ds_read_b64_tr_b16 v[216:217], v41 offset:45056\n\t"
    "ds_read_b64_tr_b16 v[218:219], v41 offset:45568\n\t"
    "s_waitcnt lgkmcnt(8)\n\t"
    "v_mfma_f32_16x16x32_bf16 v[64:67], v[196:199], a[200:203], v[64:67]\n\t"
    "ds_read_b64_tr_b16 v[188:189], v20 offset:16384\n\t"
    "ds_read_b64_tr_b16 v[190:191], v20 offset:16896\n\t"
    "v_mfma_f32_16x16x32_bf16 v[72:75], v[200:203], a[200:203], v[72:75]\n\t"
    "ds_read_b64_tr_b16 v[192:193], v21 offset:16384\n\t"
    "ds_read_b64_tr_b16 v[194:195], v21 offset:16896\n\t"
    "s_waitcnt lgkmcnt(8)\n\t"
    "buffer_load_dwordx4 v[172:175], v52, s[44:47], s78 offen offset:0 sc0 sc1\n\t"
    "v_mfma_f32_16x16x32_bf16 v[64:67], v[204:207], a[208:211], v[64:67]\n\t"
    "ds_read_b64_tr_b16 v[196:197], v20 offset:20480\n\t"
    "ds_read_b64_tr_b16 v[198:199], v20 offset:20992\n\t"
    "v_mfma_f32_16x16x32_bf16 v[72:75], v[208:211], a[208:211], v[72:75]\n\t"
    "ds_read_b64_tr_b16 v[200:201], v21 offset:20480\n\t"
    "ds_read_b64_tr_b16 v[202:203], v21 offset:20992\n\t"
    "s_waitcnt lgkmcnt(8)\n\t"
    "v_mfma_f32_16x16x32_bf16 v[64:67], v[212:215], a[216:219], v[64:67]\n\t"
    "ds_read_b64_tr_b16 v[204:205], v20 offset:24576\n\t"
    "ds_read_b64_tr_b16 v[206:207], v20 offset:25088\n\t"
    "buffer_load_dwordx4 v[176:179], v52, s[44:47], s78 offen offset:1024 sc0 sc1\n\t"
    "v_mfma_f32_16x16x32_bf16 v[72:75], v[216:219], a[216:219], v[72:75]\n\t"
    "ds_read_b64_tr_b16 v[208:209], v21 offset:24576\n\t"
    "ds_read_b64_tr_b16 v[210:211], v21 offset:25088\n\t"
    "s_waitcnt lgkmcnt(8)\n\t"
    "v_mfma_f32_16x16x32_bf16 v[64:67], v[188:191], a[224:227], v[64:67]\n\t"
    "ds_read_b64_tr_b16 v[212:213], v20 offset:28672\n\t"
    "ds_read_b64_tr_b16 v[214:215], v20 offset:29184\n\t"
    "v_mfma_f32_16x16x32_bf16 v[72:75], v[192:195], a[224:227], v[72:75]\n\t"
    "ds_read_b64_tr_b16 v[216:217], v21 offset:28672\n\t"
    "ds_read_b64_tr_b16 v[218:219], v21 offset:29184\n\t"
    "s_waitcnt lgkmcnt(8)\n\t"
    "v_mfma_f32_16x16x32_bf16 v[64:67], v[196:199], a[232:235], v[64:67]\n\t"
    "v_mfma_f32_16x16x32_bf16 v[72:75], v[200:203], a[232:235], v[72:75]\n\t"
    "s_waitcnt lgkmcnt(4)\n\t"
    "v_mfma_f32_16x16x32_bf16 v[64:67], v[204:207], a[240:243], v[64:67]\n\t"
    "v_mfma_f32_16x16x32_bf16 v[72:75], v[208:211], a[240:243], v[72:75]\n\t"
    "s_waitcnt lgkmcnt(0)\n\t"
    "v_mfma_f32_16x16x32_bf16 v[64:67], v[212:215], a[248:251], v[64:67]\n\t"
    "v_mfma_f32_16x16x32_bf16 v[72:75], v[216:219], a[248:251], v[72:75]\n\t"
    "s_waitcnt vmcnt(0)\n\t"
    "v_bitop3_b32 v54, v164, v165, v48 bitop3:0x7e\n\t"
    "v_bitop3_b32 v55, v166, v167, v48 bitop3:0x7e\n\t"
    "v_or3_b32 v49, v54, v55, v51\n\t"
    "v_bitop3_b32 v54, v168, v169, v48 bitop3:0x7e\n\t"
    "v_bitop3_b32 v55, v170, v171, v48 bitop3:0x7e\n\t"
    "v_or3_b32 v49, v54, v55, v49\n\t"
    "v_bitop3_b32 v54, v172, v173, v48 bitop3:0x7e\n\t"
    "v_bitop3_b32 v55, v174, v175, v48 bitop3:0x7e\n\t"
    "v_or3_b32 v49, v54, v55, v49\n\t"
    "v_bitop3_b32 v54, v176, v177, v48 bitop3:0x7e\n\t"
    "v_bitop3_b32 v55, v178, v179, v48 bitop3:0x7e\n\t"
    "v_or3_b32 v49, v54, v55, v49\n\t"
    "v_bitop3_b32 v54, v180, v181, v48 bitop3:0x7e\n\t"
    "v_bitop3_b32 v55, v182, v183, v48 bitop3:0x7e\n\t"
    "v_or3_b32 v49, v54, v55, v49\n\t"
    "v_bitop3_b32 v54, v184, v185, v48 bitop3:0x7e\n\t"
    "v_bitop3_b32 v55, v186, v187, v48 bitop3:0x7e\n\t"
    "v_or3_b32 v49, v54, v55, v49\n\t"
    "v_and_b32 v49, 63, v49\n\t"
    "v_cmp_ne_u32 vcc, 0, v49\n\t"
    "s_nop 1\n\t"
    "s_cbranch_vccnz LS3%=\n\t"
    "LB3%=:\n\t"
    "v_add_f32 v96, v64, v164\n\t"
    "v_add_f32 v97, v65, v165\n\t"
    "v_add_f32 v98, v66, v166\n\t"
    "v_add_f32 v99, v67, v167\n\t"
    "v_add_f32 v100, v72, v168\n\t"
    "v_add_f32 v101, v73, v169\n\t"
    "v_add_f32 v102, v74, v170\n\t"
    "v_add_f32 v103, v75, v171\n\t"
    "v_add_f32 v96, v96, v172\n\t"
    "v_add_f32 v97, v97, v173\n\t"
    "v_add_f32 v98, v98, v174\n\t"
    "v_add_f32 v99, v99, v175\n\t"
    "v_add_f32 v100, v100, v176\n\t"
    "v_add_f32 v101, v101, v177\n\t"
    "v_add_f32 v102, v102, v178\n\t"
    "v_add_f32 v103, v103, v179\n\t"
    "v_add_f32 v96, v96, v180\n\t"
    "v_add_f32 v97, v97, v181\n\t"
    "v_add_f32 v98, v98, v182\n\t"
    "v_add_f32 v99, v99, v183\n\t"
    "v_add_f32 v100, v100, v184\n\t"
    "v_add_f32 v101, v101, v185\n\t"
    "v_add_f32 v102, v102, v186\n\t"
    "v_add_f32 v103, v103, v187\n\t"
    "v_mul_f32 v152, v96, v96\n\t"
    "v_mul_f32 v153, v97, v97\n\t"
    "v_mul_f32 v154, v98, v98\n\t"
    "v_mul_f32 v155, v99, v99\n\t"
    "v_mul_f32 v156, v100, v100\n\t"
    "v_mul_f32 v157, v101, v101\n\t"
    "v_mul_f32 v158, v102, v102\n\t"
    "v_mul_f32 v159, v103, v103\n\t"
    "v_fma_f32 v152, v152, v27, v26\n\t"
    "v_fma_f32 v153, v153, v27, v26\n\t"
    "v_fma_f32 v154, v154, v27, v26\n\t"
    "v_fma_f32 v155, v155, v27, v26\n\t"
    "v_fma_f32 v156, v156, v27, v26\n\t"
    "v_fma_f32 v157, v157, v27, v26\n\t"
    "v_fma_f32 v158, v158, v27, v26\n\t"
    "v_fma_f32 v159, v159, v27, v26\n\t"
    "v_mul_f32 v152, v152, v96\n\t"
    "v_mul_f32 v153, v153, v97\n\t"
    "v_mul_f32 v154, v154, v98\n\t"
    "v_mul_f32 v155, v155, v99\n\t"
    "v_mul_f32 v156, v156, v100\n\t"
    "v_mul_f32 v157, v157, v101\n\t"
    "v_mul_f32 v158, v158, v102\n\t"
    "v_mul_f32 v159, v159, v103\n\t"
    "v_exp_f32 v152, v152\n\t"
    "v_exp_f32 v153, v153\n\t"
    "v_exp_f32 v154, v154\n\t"
    "v_exp_f32 v155, v155\n\t"
    "v_exp_f32 v156, v156\n\t"
    "v_exp_f32 v157, v157\n\t"
    "v_exp_f32 v158, v158\n\t"
    "v_exp_f32 v159, v159\n\t"
    "v_add_f32 v152, 1.0, v152\n\t"
    "v_add_f32 v153, 1.0, v153\n\t"
    "v_add_f32 v154, 1.0, v154\n\t"
    "v_add_f32 v155, 1.0, v155\n\t"
    "v_add_f32 v156, 1.0, v156\n\t"
    "v_add_f32 v157, 1.0, v157\n\t"
    "v_add_f32 v158, 1.0, v158\n\t"
    "v_add_f32 v159, 1.0, v159\n\t"
    "v_rcp_f32 v152, v152\n\t"
    "v_rcp_f32 v153, v153\n\t"
    "v_rcp_f32 v154, v154\n\t"
    "v_rcp_f32 v155, v155\n\t"
    "v_rcp_f32 v156, v156\n\t"
    "v_rcp_f32 v157, v157\n\t"
    "v_rcp_f32 v158, v158\n\t"
    "v_rcp_f32 v159, v159\n\t"
    "v_mul_f32 v152, v152, v96\n\t"
    "v_mul_f32 v153, v153, v97\n\t"
    "v_mul_f32 v154, v154, v98\n\t"
    "v_mul_f32 v155, v155, v99\n\t"
    "v_mul_f32 v156, v156, v100\n\t"
    "v_mul_f32 v157, v157, v101\n\t"
    "v_mul_f32 v158, v158, v102\n\t"
    "v_mul_f32 v159, v159, v103\n\t"
    "v_cvt_pk_bf16_f32 v160, v152, v153\n\t"
    "v_cvt_pk_bf16_f32 v161, v154, v155\n\t"
    "v_cvt_pk_bf16_f32 v162, v156, v157\n\t"
    "v_cvt_pk_bf16_f32 v163, v158, v159\n\t"
    "s_nop 1\n\t"
    "v_permlane16_swap_b32 v160, v162\n\t"
    "v_permlane16_swap_b32 v161, v163\n\t"
    "s_barrier\n\t"
    "global_store_dwordx4 v25, v[160:163], s[52:53] offset:768 sc0 sc1\n\t"
    "s_waitcnt vmcnt(0)\n\t"
    "s_barrier\n\t"
    "s_cmp_lg_u32 s81, 0\n\t"
    "s_cbranch_scc1 LEND%=\n\t"
    "s_mov_b64 exec, 1\n\t"
    "global_store_dword v51, v50, s[54:55] sc0 sc1\n\t"
    "s_mov_b64 exec, -1\n\t"
    "s_branch LEND%=\n\t"
    "LS0%=:\n\t"
    "s_cmp_lg_u32 s90, 0\n\t"
    "s_cbranch_scc1 LS0_bad%=\n\t"
    "LS0_loop%=:\n\t"
    "buffer_load_dword v56, v29, s[56:59], 0 offen offset:0 sc0 sc1\n\t"
    "s_memrealtime s[34:35]\n\t"
    "s_waitcnt vmcnt(0) lgkmcnt(0)\n\t"
    "v_cmp_gt_u32 vcc, v50, v56\n\t"
    "s_nop 1\n\t"
    "s_cbranch_vccz LB0%=\n\t"
    "s_sub_u32 s34, s62, s34\n\t"
    "s_subb_u32 s35, s63, s35\n\t"
    "s_cbranch_scc0 LS0_loop%=\n\t"
    "LS0_bad%=:\n\t"
    "s_mov_b32 s90, 1\n\t"
    "v_mov_b32 v96, 0x7fc00000\n\t"
    "v_mov_b32 v97, 0x7fc00000\n\t"
    "v_mov_b32 v98, 0x7fc00000\n\t"
    "v_mov_b32 v99, 0x7fc00000\n\t"
    "v_mov_b32 v100, 0x7fc00000\n\t"
    "v_mov_b32 v101, 0x7fc00000\n\t"
    "v_mov_b32 v102, 0x7fc00000\n\t"
    "v_mov_b32 v103, 0x7fc00000\n\t"
    "s_branch LB0%=\n\t"
    "LS1%=:\n\t"
    "s_cmp_lg_u32 s90, 0\n\t"
    "s_cbranch_scc1 LS1_bad%=\n\t"
    "LS1_loop%=:\n\t"
    "buffer_load_dword v56, v29, s[56:59], 0 offen offset:128 sc0 sc1\n\t"
    "s_memrealtime s[34:35]\n\t"
    "s_waitcnt vmcnt(0) lgkmcnt(0)\n\t"
    "v_cmp_gt_u32 vcc, v50, v56\n\t"
    "s_nop 1\n\t"
    "s_cbranch_vccz LB1%=\n\t"
    "s_sub_u32 s34, s62, s34\n\t"
    "s_subb_u32 s35, s63, s35\n\t"
    "s_cbranch_scc0 LS1_loop%=\n\t"
    "LS1_bad%=:\n\t"
    "s_mov_b32 s90, 1\n\t"
    "v_mov_b32 v96, 0x7fc00000\n\t"
    "v_mov_b32 v97, 0x7fc00000\n\t"
    "v_mov_b32 v98, 0x7fc00000\n\t"
    "v_mov_b32 v99, 0x7fc00000\n\t"
    "v_mov_b32 v100, 0x7fc00000\n\t"
    "v_mov_b32 v101, 0x7fc00000\n\t"
    "v_mov_b32 v102, 0x7fc00000\n\t"
    "v_mov_b32 v103, 0x7fc00000\n\t"
    "s_branch LB1%=\n\t"
    "LS2%=:\n\t"
    "s_cmp_lg_u32 s90, 0\n\t"
    "s_cbranch_scc1 LS2_bad%=\n\t"
    "LS2_loop%=:\n\t"
    "buffer_load_dwordx4 v[128:131], v24, s[44:47], s77 offen offset:0 sc0 sc1\n\t"
    "buffer_load_dwordx4 v[132:135], v24, s[44:47], s77 offen offset:1024 sc0 sc1\n\t"
    "buffer_load_dwordx4 v[136:139], v24, s[44:47], s78 offen offset:0 sc0 sc1\n\t"
    "buffer_load_dwordx4 v[140:143], v24, s[44:47], s78 offen offset:1024 sc0 sc1\n\t"
    "buffer_load_dwordx4 v[144:147], v24, s[44:47], s79 offen offset:0 sc0 sc1\n\t"
    "buffer_load_dwordx4 v[148:151], v24, s[44:47], s79 offen offset:1024 sc0 sc1\n\t"
    "s_memrealtime s[34:35]\n\t"
    "s_waitcnt vmcnt(0) lgkmcnt(0)\n\t"
    "v_bitop3_b32 v54, v128, v129, v48 bitop3:0x7e\n\t"
    "v_bitop3_b32 v55, v130, v131, v48 bitop3:0x7e\n\t"
    "v_or3_b32 v49, v54, v55, v51\n\t"
    "v_bitop3_b32 v54, v132, v133, v48 bitop3:0x7e\n\t"
    "v_bitop3_b32 v55, v134, v135, v48 bitop3:0x7e\n\t"
    "v_or3_b32 v49, v54, v55, v49\n\t"
    "v_bitop3_b32 v54, v136, v137, v48 bitop3:0x7e\n\t"
    "v_bitop3_b32 v55, v138, v139, v48 bitop3:0x7e\n\t"
    "v_or3_b32 v49, v54, v55, v49\n\t"
    "v_bitop3_b32 v54, v140, v141, v48 bitop3:0x7e\n\t"
    "v_bitop3_b32 v55, v142, v143, v48 bitop3:0x7e\n\t"
    "v_or3_b32 v49, v54, v55, v49\n\t"
    "v_bitop3_b32 v54, v144, v145, v48 bitop3:0x7e\n\t"
    "v_bitop3_b32 v55, v146, v147, v48 bitop3:0x7e\n\t"
    "v_or3_b32 v49, v54, v55, v49\n\t"
    "v_bitop3_b32 v54, v148, v149, v48 bitop3:0x7e\n\t"
    "v_bitop3_b32 v55, v150, v151, v48 bitop3:0x7e\n\t"
    "v_or3_b32 v49, v54, v55, v49\n\t"
    "v_and_b32 v49, 63, v49\n\t"
    "v_cmp_ne_u32 vcc, 0, v49\n\t"
    "s_nop 1\n\t"
    "s_cbranch_vccz LB2%=\n\t"
    "s_sub_u32 s34, s62, s34\n\t"
    "s_subb_u32 s35, s63, s35\n\t"
    "s_cbranch_scc0 LS2_loop%=\n\t"
    "LS2_bad%=:\n\t"
    "s_mov_b32 s90, 1\n\t"
    "v_mov_b32 v128, 0x7fc00000\n\t"
    "v_mov_b32 v129, 0x7fc00000\n\t"
    "v_mov_b32 v130, 0x7fc00000\n\t"
    "v_mov_b32 v131, 0x7fc00000\n\t"
    "v_mov_b32 v132, 0x7fc00000\n\t"
    "v_mov_b32 v133, 0x7fc00000\n\t"
    "v_mov_b32 v134, 0x7fc00000\n\t"
    "v_mov_b32 v135, 0x7fc00000\n\t"
    "s_branch LB2%=\n\t"
    "LS3%=:\n\t"
    "s_cmp_lg_u32 s90, 0\n\t"
    "s_cbranch_scc1 LS3_bad%=\n\t"
    "LS3_loop%=:\n\t"
    "buffer_load_dwordx4 v[164:167], v52, s[44:47], s77 offen offset:0 sc0 sc1\n\t"
    "buffer_load_dwordx4 v[168:171], v52, s[44:47], s77 offen offset:1024 sc0 sc1\n\t"
    "buffer_load_dwordx4 v[172:175], v52, s[44:47], s78 offen offset:0 sc0 sc1\n\t"
    "buffer_load_dwordx4 v[176:179], v52, s[44:47], s78 offen offset:1024 sc0 sc1\n\t"
    "buffer_load_dwordx4 v[180:183], v52, s[44:47], s79 offen offset:0 sc0 sc1\n\t"
    "buffer_load_dwordx4 v[184:187], v52, s[44:47], s79 offen offset:1024 sc0 sc1\n\t"
    "s_memrealtime s[34:35]\n\t"
    "s_waitcnt vmcnt(0) lgkmcnt(0)\n\t"
    "v_bitop3_b32 v54, v164, v165, v48 bitop3:0x7e\n\t"
    "v_bitop3_b32 v55, v166, v167, v48 bitop3:0x7e\n\t"
    "v_or3_b32 v49, v54, v55, v51\n\t"
    "v_bitop3_b32 v54, v168, v169, v48 bitop3:0x7e\n\t"
    "v_bitop3_b32 v55, v170, v171, v48 bitop3:0x7e\n\t"
    "v_or3_b32 v49, v54, v55, v49\n\t"
    "v_bitop3_b32 v54, v172, v173, v48 bitop3:0x7e\n\t"
    "v_bitop3_b32 v55, v174, v175, v48 bitop3:0x7e\n\t"
    "v_or3_b32 v49, v54, v55, v49\n\t"
    "v_bitop3_b32 v54, v176, v177, v48 bitop3:0x7e\n\t"
    "v_bitop3_b32 v55, v178, v179, v48 bitop3:0x7e\n\t"
    "v_or3_b32 v49, v54, v55, v49\n\t"
    "v_bitop3_b32 v54, v180, v181, v48 bitop3:0x7e\n\t"
    "v_bitop3_b32 v55, v182, v183, v48 bitop3:0x7e\n\t"
    "v_or3_b32 v49, v54, v55, v49\n\t"
    "v_bitop3_b32 v54, v184, v185, v48 bitop3:0x7e\n\t"
    "v_bitop3_b32 v55, v186, v187, v48 bitop3:0x7e\n\t"
    "v_or3_b32 v49, v54, v55, v49\n\t"
    "v_and_b32 v49, 63, v49\n\t"
    "v_cmp_ne_u32 vcc, 0, v49\n\t"
    "s_nop 1\n\t"
    "s_cbranch_vccz LB3%=\n\t"
    "s_sub_u32 s34, s62, s34\n\t"
    "s_subb_u32 s35, s63, s35\n\t"
    "s_cbranch_scc0 LS3_loop%=\n\t"
    "LS3_bad%=:\n\t"
    "s_mov_b32 s90, 1\n\t"
    "v_mov_b32 v164, 0x7fc00000\n\t"
    "v_mov_b32 v165, 0x7fc00000\n\t"
    "v_mov_b32 v166, 0x7fc00000\n\t"
    "v_mov_b32 v167, 0x7fc00000\n\t"
    "v_mov_b32 v168, 0x7fc00000\n\t"
    "v_mov_b32 v169, 0x7fc00000\n\t"
    "v_mov_b32 v170, 0x7fc00000\n\t"
    "v_mov_b32 v171, 0x7fc00000\n\t"
    "s_branch LB3%=\n\t"
    "LEND%=:\n\t"
    "s_nop 1\n\t"
    : "={s90}"(bad), "={s91}"(call), "+{v[24:31]}"(v2)
    : "{s[36:39]}"(sx), "{s[40:43]}"(sw), "{s[44:47]}"(sp), "{s[48:51]}"(sc), "{s[56:59]}"(sf), "{s[52:53]}"(pr.out), "{s[54:55]}"(fout), "{s[62:63]}"(dl), "{s[64:79]}"(s1), "{s[80:81]}"(s2),
      "{v[8:15]}"(v0), "{v[16:23]}"(v1)
    : "v32", "v33", "v34", "v35", "v36", "v37", "v38", "v39", "v40", "v41", "v42", "v43", "v44", "v45", "v46", "v47", "v48", "v49", "v50", "v51", "v52", "v53", "v54", "v55", "v56", "v57", "v58", "v59", "v60", "v61", "v62", "v63", "v64", "v65", "v66", "v67", "v68", "v69", "v70", "v71", "v72", "v73", "v74", "v75", "v76", "v77", "v78", "v79", "v80", "v81", "v82", "v83", "v84", "v85", "v86", "v87", "v88", "v89", "v90", "v91", "v92", "v93", "v94", "v95", "v96", "v97", "v98", "v99", "v100", "v101", "v102", "v103", "v104", "v105", "v106", "v107", "v108", "v109", "v110", "v111", "v112", "v113", "v114", "v115", "v116", "v117", "v118", "v119", "v120", "v121", "v122", "v123", "v124", "v125", "v126", "v127", "v128", "v129", "v130", "v131", "v132", "v133", "v134", "v135", "v136", "v137", "v138", "v139", "v140", "v141", "v142", "v143", "v144", "v145", "v146", "v147", "v148", "v149", "v150", "v151", "v152", "v153", "v154", "v155", "v156", "v157", "v158", "v159", "v160", "v161", "v162", "v163", "v164", "v165", "v166", "v167", "v168", "v169", "v170", "v171", "v172", "v173", "v174", "v175", "v176", "v177", "v178", "v179", "v180", "v181", "v182", "v183", "v184", "v185", "v186", "v187", "v188", "v189", "v190", "v191", "v192", "v193", "v194", "v195", "v196", "v197", "v198", "v199", "v200", "v201", "v202", "v203", "v204", "v205", "v206", "v207", "v208", "v209", "v210", "v211", "v212", "v213", "v214", "v215", "v216", "v217", "v218", "v219", "a0", "a1", "a2", "a3", "a4", "a5", "a6", "a7", "a8", "a9", "a10", "a11", "a12", "a13", "a14", "a15", "a16", "a17", "a18", "a19", "a20", "a21", "a22", "a23", "a24", "a25", "a26", "a27", "a28", "a29", "a30", "a31", "a32", "a33", "a34", "a35", "a36", "a37", "a38", "a39", "a40", "a41", "a42", "a43", "a44", "a45", "a46", "a47", "a48", "a49", "a50", "a51", "a52", "a53", "a54", "a55", "a56", "a57", "a58", "a59", "a60", "a61", "a62", "a63", "a64", "a65", "a66", "a67", "a68", "a69", "a70", "a71", "a72", "a73", "a74", "a75", "a76", "a77", "a78", "a79", "a80", "a81", "a82", "a83", "a84", "a85", "a86", "a87", "a88", "a89", "a90", "a91", "a92", "a93", "a94", "a95", "a96", "a97", "a98", "a99", "a100", "a101", "a102", "a103", "a104", "a105", "a106", "a107", "a108", "a109", "a110", "a111", "a112", "a113", "a114", "a115", "a116", "a117", "a118", "a119", "a120", "a121", "a122", "a123", "a124", "a125", "a126", "a127", "a128", "a129", "a130", "a131", "a132", "a133", "a134", "a135", "a136", "a137", "a138", "a139", "a140", "a141", "a142", "a143", "a144", "a145", "a146", "a147", "a148", "a149", "a150", "a151", "a152", "a153", "a154", "a155", "a156", "a157", "a158", "a159", "a160", "a161", "a162", "a163", "a164", "a165", "a166", "a167", "a168", "a169", "a170", "a171", "a172", "a173", "a174", "a175", "a176", "a177", "a178", "a179", "a180", "a181", "a182", "a183", "a184", "a185", "a186", "a187", "a188", "a189", "a190", "a191", "a192", "a193", "a194", "a195", "a196", "a197", "a198", "a199", "a200", "a201", "a202", "a203", "a204", "a205", "a206", "a207", "a208", "a209", "a210", "a211", "a212", "a213", "a214", "a215", "a216", "a217", "a218", "a219", "a220", "a221", "a222", "a223", "a224", "a225", "a226", "a227", "a228", "a229", "a230", "a231", "a232", "a233", "a234", "a235", "a236", "a237", "a238", "a239", "a240", "a241", "a242", "a243", "a244", "a245", "a246", "a247", "a248", "a249", "a250", "a251", "a252", "a253", "a254", "a255", "s60", "s82", "s83", "s84", "s85", "s86", "s87", "s88", "s89", "s34", "s35", "vcc", "scc", "memory");
  unsigned* tmo = (unsigned*)(wsl + WS_TMO);
  if (bad && lane == 0) __hip_atomic_store(tmo, call, __ATOMIC_RELAXED, __HIP_MEMORY_SCOPE_AGENT);
  if (w == 0) {
    if (lane == 0) __hip_atomic_store((unsigned long long*)(wsl + WS_CTR + b * 128), (dl << 32) | call, __ATOMIC_RELAXED, __HIP_MEMORY_SCOPE_SYSTEM);
    if (dst == rank && lane < NR && lane != rank) {
      unsigned* fl = (unsigned*)(wsl + WS_FLAG + (lane * 32 + slot) * 128);
      bool late = false;
      for (unsigned it = 1; __hip_atomic_load(fl, __ATOMIC_RELAXED, __HIP_MEMORY_SCOPE_SYSTEM) < call; ++it) {
        if ((it & 15) == 0) {
          if (__hip_atomic_load(tmo, __ATOMIC_RELAXED, __HIP_MEMORY_SCOPE_AGENT) == call || __builtin_amdgcn_s_memrealtime() > dl) {
            __hip_atomic_store(tmo, call, __ATOMIC_RELAXED, __HIP_MEMORY_SCOPE_AGENT); late = true; break; }
          __builtin_amdgcn_s_sleep(1);
        }
      }
      if (late) { asm volatile("s_waitcnt vmcnt(0)" ::: "memory"); *(unsigned short*)(pr.out + oo) = 0x7fc0; }
    }
  }
}

}

extern "C" void kernel_launch(int rank, int nranks, ncclComm_t comm, hipStream_t stream,
                              void* const* d_in, const int* in_sizes, int n_in, void* d_out, int out_size,
                              const int* in_dtypes, int out_dtype, void* const* d_out_all, void* const* d_ws_all, size_t ws_bytes) {
  static bool init = false;
  if (!init) { HIP_CHECK(hipFuncSetAttribute((const void*)gemm_a2a, hipFuncAttributeMaxDynamicSharedMemorySize, LDS_BYTES)); init = true; }
  if (nranks != NR || (unsigned)rank >= (unsigned)NR || ws_bytes < (size_t)WS_PART + PART_BYTES) { fprintf(stderr, "unexpected configuration\n"); return; }
  Args a; a.x = (const char*)d_in[0]; a.w = (const char*)d_in[1]; a.rank = rank; a.wsl = (char*)d_ws_all[rank];
  a.pad = 0;
  for (int p = 0; p < NR; ++p) { a.p[p].out = (char*)d_out_all[p] + (size_t)rank * ML * NS * 2; a.p[p].flag = (char*)d_ws_all[p] + WS_FLAG + rank * 32 * 128; }
  hipLaunchKernelGGL(gemm_a2a, dim3(NWG), dim3(256), LDS_BYTES, stream, a);
}
```
